```python
import numpy as np

kernel_name = "gpu_gemm_ar_m4096_k4096_n16384_bf16_relu"


ABSMAX_RTOL = 0.008

COMPUTE_DTYPE = "bfloat16"
EPILOGUE = "relu"
IN_QUANT = None
OUT_QUANT = None
IN_DTYPES = {"x": "bfloat16", "w_mat": "bfloat16"}
WORLD_SIZE = 8

def _to_bf16(x):
    u = np.ascontiguousarray(x, dtype=np.float32).view(np.uint32)
    return ((u + 0x7FFF + ((u >> 16) & 1)) & 0xFFFF0000).view(np.float32)

def setup_inputs():

    x = _to_bf16(np.random.randn(4096, 4096).astype(np.float32))
    w_mat = _to_bf16((np.random.randn(4096, 16384) / np.sqrt(4096)).astype(np.float32))
    return {"x": x, "w_mat": w_mat}

def _epilogue(y):
    return np.maximum(y, 0.0).astype(np.float32)

def reference(x, w_mat):
    y = (x @ w_mat).astype(np.float32)
    return _epilogue(y)
```

```cpp
#include <hip/hip_runtime.h>
#include <rccl/rccl.h>
#include <cstdio>

typedef unsigned u32;
typedef unsigned long long u64;
typedef unsigned char u8;
typedef u32 u32x2 __attribute__((ext_vector_type(2)));
typedef u32 u32x4 __attribute__((ext_vector_type(4)));
typedef float f32x2 __attribute__((ext_vector_type(2)));
typedef float f32x4 __attribute__((ext_vector_type(4)));
typedef short s16x2 __attribute__((ext_vector_type(2)));
typedef short s16x4 __attribute__((ext_vector_type(4)));
typedef __bf16 bf16x2 __attribute__((ext_vector_type(2)));
typedef __bf16 bf16x8 __attribute__((ext_vector_type(8)));
#define LAS __attribute__((address_space(3)))
#define GAS __attribute__((address_space(1)))
typedef __amdgpu_buffer_rsrc_t rsrc_t;

namespace {

constexpr int NR = 8;
constexpr int MM = 4096, NN = 16384, KSH = 512, NSH = 2048;
constexpr int NWG = 256, NTHR = 512;
constexpr int NL = 18;
constexpr int NENT = 768 + 2048;
constexpr int FROW = 192;
constexpr int NTASK = 2560;
#define WINDOW 6
constexpr u64 DEADLINE = 30000000ull;

constexpr u32 OFF_CALL = 0;
constexpr u32 OFF_TMO = 16384;
constexpr u32 OFF_CNT = 20480;
constexpr u32 OFF_FLAG = 32768;
constexpr u32 OFF_DBG = 512u << 10;
constexpr u32 OFF_EX = 1u << 20;
constexpr u32 OFF_EW = 5u << 20;
constexpr u32 OFF_EO = 21u << 20;
constexpr u32 OFF_AT = 37u << 20;
constexpr u32 OFF_BT = 69u << 20;
constexpr u32 WS_NEED = 85u << 20;

constexpr int L_STAGE = 49152;
constexpr int L_HS = 14 * 8192;
constexpr int L_HS_STRIDE = 4160;
constexpr int L_MISC = 3 * L_STAGE;
constexpr int L_LUT = L_MISC;
constexpr int L_META = L_MISC + 128;
constexpr int L_FREE = L_MISC + 384;
constexpr int L_LANDED = L_MISC + 448;
constexpr int L_DEAD = L_MISC + 452;
constexpr int L_CALL = L_MISC + 456;
constexpr int L_FLAGS = L_MISC + 512;
constexpr int L_EXC = L_MISC + 512 + 4 * FROW;
constexpr int L_TOTAL = L_EXC + 8 * 256;
static_assert(L_HS + 7 * L_HS_STRIDE <= L_MISC, "lds");
static_assert(L_TOTAL <= 163840, "lds");

constexpr u32 LO_OFF = 528;
constexpr u32 UB_IN = LO_OFF + 4096;

struct Params {
  const void* x;
  const void* w;
  void* out;
  void* ws[NR];
  int rank;
  int pad;
};

#define CFENCE() asm volatile("" ::: "memory")
#define WAIT_VM(n) asm volatile("s_waitcnt vmcnt(" #n ")" ::: "memory")
#define WAIT_LGKM0() asm volatile("s_waitcnt lgkmcnt(0)" ::: "memory")
#define STR2(x) #x
#define STR(x) STR2(x)

__device__ __forceinline__ u32 rfl(u32 v) { return __builtin_amdgcn_readfirstlane(v); }
__device__ __forceinline__ u32 vperm(u32 hi, u32 lo, u32 sel) { return __builtin_amdgcn_perm(hi, lo, sel); }
__device__ __forceinline__ u32 bsum(u32 v, u32 acc) { return __builtin_amdgcn_sad_u8(v, 0u, acc); }
__device__ __forceinline__ u64 now() { return __builtin_amdgcn_s_memrealtime(); }
__device__ __forceinline__ u32 mbcnt(u64 m) {
  return __builtin_amdgcn_mbcnt_hi((u32)(m >> 32), __builtin_amdgcn_mbcnt_lo((u32)m, 0u));
}
__device__ __forceinline__ u32 scan_incl(u32 x) {
  x += __builtin_amdgcn_update_dpp(0u, x, 0x111, 0xf, 0xf, true);
  x += __builtin_amdgcn_update_dpp(0u, x, 0x112, 0xf, 0xf, true);
  x += __builtin_amdgcn_update_dpp(0u, x, 0x114, 0xf, 0xf, true);
  x += __builtin_amdgcn_update_dpp(0u, x, 0x118, 0xf, 0xf, true);
  x += __builtin_amdgcn_update_dpp(0u, x, 0x142, 0xa, 0xf, false);
  x += __builtin_amdgcn_update_dpp(0u, x, 0x143, 0xc, 0xf, false);
  return x;
}
__device__ __forceinline__ rsrc_t make_rsrc(const void* p, u32 bytes) {
  return __builtin_amdgcn_make_buffer_rsrc((void*)p, 0, bytes, 0x00020000);
}
template <int SYS>
__device__ __forceinline__ void dma16(rsrc_t r, u32 lds, u32 voff, u32 soff) {
  u32 keep;
  if (SYS == 2)
    asm volatile("s_mov_b32 %0, m0\n\ts_mov_b32 m0, %1\n\ts_nop 2\n\tbuffer_load_dwordx4 %2, %3, %4 offen sc0 sc1 lds\n\ts_mov_b32 m0, %0"
                 : "=&s"(keep) : "s"(lds), "v"(voff), "s"(r), "s"(soff) : "memory");
  else if (SYS == 1)
    asm volatile("s_mov_b32 %0, m0\n\ts_mov_b32 m0, %1\n\ts_nop 2\n\tbuffer_load_dwordx4 %2, %3, %4 offen sc1 lds\n\ts_mov_b32 m0, %0"
                 : "=&s"(keep) : "s"(lds), "v"(voff), "s"(r), "s"(soff) : "memory");
  else
    asm volatile("s_mov_b32 %0, m0\n\ts_mov_b32 m0, %1\n\ts_nop 2\n\tbuffer_load_dwordx4 %2, %3, %4 offen lds\n\ts_mov_b32 m0, %0"
                 : "=&s"(keep) : "s"(lds), "v"(voff), "s"(r), "s"(soff) : "memory");
}
__device__ __forceinline__ void dma4(rsrc_t r, u32 lds, u32 voff, u32 soff) {
  u32 keep;
  asm volatile("s_mov_b32 %0, m0\n\ts_mov_b32 m0, %1\n\ts_nop 2\n\tbuffer_load_dword %2, %3, %4 offen sc0 sc1 lds\n\ts_mov_b32 m0, %0"
               : "=&s"(keep) : "s"(lds), "v"(voff), "s"(r), "s"(soff) : "memory");
}
__device__ __forceinline__ u32 lds_ld(LAS u8* lds, int off) {
  return rfl(*(volatile LAS u32*)(lds + off));
}
__device__ __forceinline__ void lds_st(LAS u8* lds, int off, u32 v) { *(volatile LAS u32*)(lds + off) = v; }

#define STAMP(ws, slot) do {} while (0)

template <bool OUT>
__device__ __forceinline__ u32 encode_chunk(const u32 (&P)[32], LAS u8* bld, LAS u32* exs, const LAS u32* lutc, rsrc_t ws,
                                            u32 goff, u32 lane) {
  u32 lo[16], hi[16], nzm[16], len[16];
  u32 pl[2] = {0u, 0u};
  u32 sum = 0, sgn = 0;
#pragma unroll
  for (int g = 0; g < 16; ++g) {
    lo[g] = vperm(P[2 * g + 1], P[2 * g], 0x06040200u);
    hi[g] = vperm(P[2 * g + 1], P[2 * g], 0x07050301u);
    if (OUT) {
      const s16x2 z = {0, 0};
      const u32 n0 = __builtin_bit_cast(u32, z - __builtin_bit_cast(s16x2, P[2 * g]));
      const u32 n1 = __builtin_bit_cast(u32, z - __builtin_bit_cast(s16x2, P[2 * g + 1]));
      nzm[g] = vperm(n1, n0, 0x0b0a0908u);
      pl[g >> 3] |= bsum(nzm[g] & 0x08040201u, 0u) << (4 * (g & 7));
      sgn |= hi[g];
    } else {
      pl[g >> 3] |= (hi[g] >> (7 - (g & 7))) & (0x01010101u << (g & 7));
      hi[g] &= 0x7f7f7f7fu;
      nzm[g] = 0xffffffffu;
    }
    sum = bsum(hi[g], sum);
  }
  const bool signed_out = OUT && __builtin_amdgcn_ballot_w64((sgn & 0x80808080u) != 0u) != 0ull;
  const u32 cnt = OUT ? (u32)(__builtin_popcount(pl[0]) + __builtin_popcount(pl[1])) : 64u;
  const u32 sc0 = scan_incl((sum & 0x1fffu) | (cnt << 19));
  const u32 tot0 = __builtin_amdgcn_readlane(sc0, 63);
  const u32 nnz = tot0 >> 19;
  const u32 pbase = (sc0 >> 19) - cnt;
  const float mean = (float)(tot0 & 0x7ffffu) / (float)(nnz ? nnz : 1u);
  int b = (int)(mean + 0.78f);
  const float fr = mean - (float)(b - 1);
  const u32 k = rfl(fr < 0.32f ? 5u : fr < 0.41f ? 4u : fr < 0.60f ? 3u : fr < 0.87f ? 2u : 1u);
  b = b < 6 ? 6 : b > 126 ? 126 : b;
  const u32 base4 = rfl((u32)(b - 6)) * 0x01010101u;
  const u64 below = (1ull << (8 * k)) - 1ull;
  const u64 dec = (0x0001020304050607ull & ~below & ~(0xffull << (8 * k))) | (7ull << (8 * k)) |
                  (0x0000010203040506ull & below);
  const u64 inv = (0x0001020304050607ull + (0x0101010101010101ull >> (8 * (k + 1)))) | ((u64)(k + 1) << 56);
  const u32 dec_lo = (u32)dec + base4, dec_hi = (u32)(dec >> 32) + base4;
  const u32 inv_lo = (u32)inv, inv_hi = (u32)(inv >> 32);

  u32 bits = 0, nexc = 0;
#pragma unroll
  for (int g = 0; g < 16; ++g) {
    const u32 e4 = ((hi[g] | 0x80808080u) - base4) ^ 0x80808080u;
    const u32 bad = e4 & 0xf8f8f8f8u & nzm[g];
    len[g] = vperm(inv_hi, inv_lo, e4) & nzm[g];
    if (__builtin_amdgcn_ballot_w64(bad != 0u) != 0ull) {
      u32 idx0 = pbase;
      if (OUT) {
        idx0 += g >= 8 ? __builtin_popcount(pl[0]) : 0;
        idx0 += __builtin_popcount(pl[g >> 3] & ((1u << (4 * (g & 7))) - 1u));
      } else {
        idx0 += 4 * g;
      }
#pragma unroll
      for (int i = 0; i < 4; ++i) {
        const bool bi = ((bad >> (8 * i)) & 0xffu) != 0u;
        const u64 m = __builtin_amdgcn_ballot_w64(bi);
        const u32 slot = nexc + mbcnt(m);
        const u32 idx = idx0 + __builtin_popcount(nzm[g] & 0x01010101u & ((1u << (8 * i)) - 1u));
        if (bi) {
          len[g] = (len[g] & ~(0xffu << (8 * i))) | (1u << (8 * i));
          if (slot < 64u) exs[slot] = (idx << 8) | ((hi[g] >> (8 * i)) & 0xffu);
        }
        nexc += __builtin_popcountll(m);
      }
    }
    bits = bsum(len[g], bits);
  }
  const u32 sc1 = scan_incl(bits);
  const u32 tbits = __builtin_amdgcn_readlane(sc1, 63);
  const u32 nw = (tbits + 31u) >> 5;
  const u32 ubase = OUT ? LO_OFF + ((nnz + 3u) & ~3u) : UB_IN;
  const u32 xbase = ubase + 4u * nw;
  const u32 size16 = (xbase + 4u * nexc + 15u) >> 4;
  if (size16 >= 512u || nexc > 64u || signed_out) {
#pragma unroll
    for (int v = 0; v < 8; ++v) {
      const u32x4 q = {P[4 * v], P[4 * v + 1], P[4 * v + 2], P[4 * v + 3]};
      __builtin_amdgcn_raw_buffer_store_b128(q, ws, lane * 16u + v * 1024u, goff, 17);
    }
    return 512u;
  }
  {
    const u32x4 z = {0u, 0u, 0u, 0u};
    for (u32 o = (OUT ? LO_OFF : UB_IN) + lane * 16u; o < size16 * 16u; o += 1024u) *(LAS u32x4*)(bld + o) = z;
  }
  CFENCE();
  u32 q = sc1 - bits, p = pbase;
#pragma unroll
  for (int g = 0; g < 16; ++g) {
    u32 t = len[g] + 0x1fu;
    t += t << 8;
    t += t << 16;
    const u32 one = nzm[g] & 0x01010101u;
    const u32 b4 = ((one & 0xffu) << (t & 31u)) | (((one >> 8) & 0xffu) << ((t >> 8) & 31u)) |
                   (((one >> 16) & 0xffu) << ((t >> 16) & 31u)) | ((one >> 24) << ((t >> 24) & 31u));
    const u64 v = (u64)b4 << (q & 31u);
    LAS u32* d = (LAS u32*)(bld + ubase + ((q >> 5) << 2));
    __hip_atomic_fetch_or(d, (u32)v, __ATOMIC_RELAXED, __HIP_MEMORY_SCOPE_WAVEFRONT);
    __hip_atomic_fetch_or(d + 1, (u32)(v >> 32), __ATOMIC_RELAXED, __HIP_MEMORY_SCOPE_WAVEFRONT);
    q = bsum(len[g], q);
    if (OUT) {
      const u32 m4 = (pl[g >> 3] >> (4 * (g & 7))) & 15u;
      const u32 c4 = vperm(0u, lo[g], lutc[m4]);
      const u64 u = (u64)c4 << ((p & 3u) * 8u);
      LAS u32* e = (LAS u32*)(bld + LO_OFF + (p & ~3u));
      __hip_atomic_fetch_or(e, (u32)u, __ATOMIC_RELAXED, __HIP_MEMORY_SCOPE_WAVEFRONT);
      __hip_atomic_fetch_or(e + 1, (u32)(u >> 32), __ATOMIC_RELAXED, __HIP_MEMORY_SCOPE_WAVEFRONT);
      p += __builtin_popcount(m4);
    }
  }
  if (!OUT) {
#pragma unroll
    for (int j = 0; j < 4; ++j) {
      const u32x4 t = {lo[4 * j], lo[4 * j + 1], lo[4 * j + 2], lo[4 * j + 3]};
      *(LAS u32x4*)(bld + LO_OFF + j * 1024 + lane * 16u) = t;
    }
  }
  {
    const u32x2 t = {pl[0], pl[1]};
    *(LAS u32x2*)(bld + 16 + lane * 8u) = t;
    if (lane == 0) {
      const u32x4 h = {dec_lo, dec_hi, nw | (nexc << 16), nnz};
      *(LAS u32x4*)bld = h;
    }
  }
  CFENCE();
  if (lane < nexc) *(LAS u32*)(bld + xbase + 4u * lane) = exs[lane];
  CFENCE();
  for (u32 o = lane * 16u; o < size16 * 16u; o += 1024u) {
    const u32x4 t = *(const LAS u32x4*)(bld + o);
    __builtin_amdgcn_raw_buffer_store_b128(t, ws, o, goff, 17);
  }
  CFENCE();
  return size16;
}

template <bool OUT>
__device__ __forceinline__ void decode_chunk(const LAS u8* slot, LAS u8* hs, const LAS u32x2* lutx, u32 size16, u32 lane,
                                             u32 (&P)[32]) {
  if (size16 >= 512u) {
#pragma unroll
    for (int v = 0; v < 8; ++v) {
      const u32x4 t = *(const LAS u32x4*)(slot + v * 1024 + lane * 16u);
      P[4 * v] = t.x; P[4 * v + 1] = t.y; P[4 * v + 2] = t.z; P[4 * v + 3] = t.w;
    }
    return;
  }
  const u32x4 h = *(const LAS u32x4*)slot;
  const u32 tab_lo = rfl(h.x), tab_hi = rfl(h.y), hz = rfl(h.z), nnz = rfl(h.w) & 0x1fffu;
  const u32 nw = hz & 0x7ffu, nexc = (hz >> 16) & 0x7fu;
  const u32 ubase = OUT ? LO_OFF + ((nnz + 3u) & ~3u) : UB_IN;
  const LAS u32* U = (const LAS u32*)(slot + ubase);
  u32 run = 0;
  for (u32 w0 = 0; w0 < nw; w0 += 64u) {
    const u32 i = w0 + lane;
    u32 w = U[i], prev = U[(int)i - 1];
    w = i < nw ? w : 0u;
    prev = i > 0u ? prev : 0x80000000u;
    const u32 c = __builtin_popcount(w);
    const u32 incl = scan_incl(c);
    LAS u8* dst = hs + ((run + incl - c) & 4095u);
    run += __builtin_amdgcn_readlane(incl, 63);
    u32 nq = __builtin_clz(prev | 1u);
    while (w) {
      const u32 pos = __builtin_ctz(w);
      w &= w - 1u;
      *dst++ = (u8)vperm(tab_hi, tab_lo, nq + pos);
      nq = ~pos;
    }
  }
  CFENCE();
  if (lane < nexc) {
    const u32 r = U[nw + lane];
    hs[(r >> 8) & 4095u] = (u8)r;
  }
  CFENCE();
  const u32x2 pl = *(const LAS u32x2*)(slot + 16 + lane * 8u);
  if (OUT) {
    const u32 c = __builtin_popcount(pl.x) + __builtin_popcount(pl.y);
    u32 p = scan_incl(c) - c;
#pragma unroll
    for (int g = 0; g < 16; ++g) {
      const u32 m4 = ((g < 8 ? pl.x : pl.y) >> (4 * (g & 7))) & 15u;
      const u32x2 sel = lutx[m4];
      const u32 a = p & ~3u;
      const LAS u32* lp = (const LAS u32*)(slot + LO_OFF + a);
      const LAS u32* hp = (const LAS u32*)(hs + a);
      const u32 l4 = __builtin_amdgcn_alignbyte(lp[1], lp[0], p);
      const u32 h4 = __builtin_amdgcn_alignbyte(hp[1], hp[0], p);
      P[2 * g] = vperm(h4, l4, sel.x);
      P[2 * g + 1] = vperm(h4, l4, sel.y);
      p += __builtin_popcount(m4);
    }
  } else {
#pragma unroll
    for (int j = 0; j < 4; ++j) {
      const u32x4 L = *(const LAS u32x4*)(slot + LO_OFF + j * 1024 + lane * 16u);
      const u32x4 H = *(const LAS u32x4*)(hs + lane * 64u + j * 16);
#pragma unroll
      for (int i = 0; i < 4; ++i) {
        const int g = 4 * j + i;
        const u32 hh = H[i] | (((g < 8 ? pl.x : pl.y) << (7 - (g & 7))) & 0x80808080u);
        P[2 * g] = vperm(hh, L[i], 0x05010400u);
        P[2 * g + 1] = vperm(hh, L[i], 0x07030602u);
      }
    }
  }
}

__device__ __forceinline__ u32 entry_w(u32 wl) { const u32 q = wl >> 6; return (q == 0 ? 0u : q == 1 ? 192u : q == 2 ? 384u : 448u) + (wl & 63u); }
__device__ __forceinline__ u32 entry_x(u32 xc) { const u32 q = xc >> 7; return (q == 0 ? 64u : q == 1 ? 256u : q == 2 ? 512u : 640u) + (xc & 127u); }
__device__ __forceinline__ void entry_inv(u32 e, u32& kind, u32& id) {
  if (e >= 768u) { kind = 2; id = e - 768u; }
  else if (e < 64u) { kind = 0; id = e; }
  else if (e < 192u) { kind = 1; id = e - 64u; }
  else if (e < 256u) { kind = 0; id = e - 128u; }
  else if (e < 384u) { kind = 1; id = e - 128u; }
  else if (e < 512u) { kind = 0; id = e - 256u; }
  else { kind = 1; id = e - 256u; }
}
__device__ __forceinline__ void st_coord(u32 seq, u32& i, u32& j) {
  if (seq >= 8u) { i = seq >> 2; j = seq & 3u; }
  else if (seq < 4u) { i = seq >> 1; j = seq & 1u; }
  else { i = seq & 1u; j = seq >> 1; }
}
__device__ __forceinline__ u32 flag_off(u32 src, u32 e) { return OFF_FLAG + ((src * NL + e % NL) * FROW + e / NL) * 4u; }

__device__ __forceinline__ bool timed_out(const GAS u32* tmo, u32 call) {
  return rfl(__hip_atomic_load(tmo, __ATOMIC_RELAXED, __HIP_MEMORY_SCOPE_AGENT)) == call;
}

}

__global__ __launch_bounds__(NTHR, 2) void gemm_ar_kernel(Params prm) {
  __shared__ __attribute__((aligned(16))) u8 smem_raw[L_TOTAL];
  LAS u8* smem = (LAS u8*)smem_raw;
  const u32 tid = threadIdx.x, lane = tid & 63u, wid = rfl(tid >> 6);
  const u32 rank = (u32)prm.rank;
  char* wsl = (char*)prm.ws[rank & 7u];
  const rsrc_t ws = make_rsrc(wsl, WS_NEED);
  GAS u32* tmo = (GAS u32*)(wsl + OFF_TMO);
  const u32 xcd = blockIdx.x & 7u, pos = blockIdx.x >> 3;
  STAMP(wsl, 0);

  if (tid == 0) {
    GAS u32* cc = (GAS u32*)(wsl + OFF_CALL + blockIdx.x * 64u);
    const u32 c = __hip_atomic_load(cc, __ATOMIC_RELAXED, __HIP_MEMORY_SCOPE_AGENT) + 1u;
    __hip_atomic_store(cc, c, __ATOMIC_RELAXED, __HIP_MEMORY_SCOPE_AGENT);
    lds_st(smem, L_CALL, c);
  }
  if (tid >= 64u && tid < 80u) {
    const u32 m = tid - 64u;
    u32 cs = 0x0c0c0c0cu, r = 0;
    u64 xs = 0x0c0c0c0c0c0c0c0cull;
    for (u32 i = 0; i < 4; ++i)
      if ((m >> i) & 1u) {
        cs = (cs & ~(0xffu << (8 * r))) | (i << (8 * r));
        xs = (xs & ~(0xffffull << (16 * i))) | ((u64)(r | ((4u + r) << 8)) << (16 * i));
        ++r;
      }
    if (pos < 16u) *(LAS u32*)(smem + L_LUT + 4 * m) = cs;
    else *(LAS u64*)(smem + L_LUT + 8 * m) = xs;
  }
  if (tid >= 128u && tid < 128u + (L_CALL - L_META) / 4u) *(LAS u32*)(smem + L_META + (tid - 128u) * 4u) = 0u;
  if (tid >= 256u && tid < 256u + FROW) *(LAS u32*)(smem + L_FLAGS + (tid - 256u) * 4u) = 0u;

  if (pos < 16u) {
    LAS u8* stg = smem + wid * 16384u;
    LAS u32* exs = (LAS u32*)(smem + L_EXC + wid * 256u);
    const LAS u32* lutc = (const LAS u32*)(smem + L_LUT);
    const rsrc_t rx = make_rsrc(prm.x, MM * KSH * 2), rw = make_rsrc(prm.w, KSH * NN * 2);
    u32 call = 0;
    {
      const u32 gw = (pos * 8u + xcd) * 8u + wid;
      for (u32 t = gw; t < (u32)NTASK; t += 1024u) {
        u32 isw, id, dest = 0;
        if (t < 512u) { isw = 1; id = t; }
        else if (t < 640u) { isw = 0; id = t - 512u; }
        else if (t < 1152u) { isw = 1; id = t - 128u; }
        else if (t < 1280u) { isw = 0; id = t - 1024u; }
        else if (t < 2304u) { isw = 1; id = t - 256u; }
        else { isw = 0; id = t - 2048u; }
        u32 P[32];
        if (isw) {
          dest = id & 7u;
          id >>= 3;
          const u32 cb = id >> 4, kb = id & 15u;
          const u32 voff = (lane >> 4) * (NN * 2) + (lane & 15u) * 16u;
          const u32 soff = kb * 32u * (NN * 2) + (dest * NSH + cb * 128u) * 2u;
#pragma unroll
          for (int u = 0; u < 8; ++u) dma16<0>(rw, (u32)(size_t)stg + u * 1024u, voff, soff + u * 4u * (NN * 2));
          WAIT_VM(0);
          const LAS u8* tb = stg + ((lane >> 4) * 8u + ((lane & 15u) >> 2)) * 256u + (lane & 3u) * 16u;
#pragma unroll
          for (int nf = 0; nf < 8; ++nf)
#pragma unroll
            for (int h = 0; h < 2; ++h) {
              const s16x4 r = __builtin_amdgcn_ds_read_tr16_b64_v4i16((LAS s16x4*)(tb + h * 1024 + (nf >> 1) * 64 + (nf & 1) * 8));
              const u32x2 r2 = __builtin_bit_cast(u32x2, r);
              P[4 * nf + 2 * h] = r2.x;
              P[4 * nf + 2 * h + 1] = r2.y;
            }
        } else {
          const u32 rb = id >> 5, ksl = (id >> 2) & 7u, mq = id & 3u;
          const u32 voff = (lane & 15u) * (KSH * 2) + (lane >> 4) * 16u;
          const u32 soff = (rb * 256u + mq * 64u) * (KSH * 2) + ksl * 128u;
#pragma unroll
          for (int v = 0; v < 8; ++v) {
            const u32x4 r = __builtin_amdgcn_raw_buffer_load_b128(rx, voff + (v & 3) * 16 * (KSH * 2) + (v >> 2) * 64, soff, 0);
            P[4 * v] = r.x; P[4 * v + 1] = r.y; P[4 * v + 2] = r.z; P[4 * v + 3] = r.w;
          }
        }
        if (call == 0) {
          __syncthreads();
          call = lds_ld(smem, L_CALL);
        }
        u32 size16 = 0;
        if (!isw || dest != rank) size16 = encode_chunk<false>(P, stg + 8192, exs, lutc, ws, isw ? OFF_EW + (dest * 256u + id) * 8192u : OFF_EX + id * 8192u, lane);
        if (!isw || dest == rank) {
          u32 o;
          if (isw) o = OFF_BT + (id >> 4) * (1u << 20) + (rank * 16u + (id & 15u)) * 8192u;
          else o = OFF_AT + (id >> 5) * (2u << 20) + (rank * 8u + ((id >> 2) & 7u)) * 32768u + (id & 3u) * 4096u;
#pragma unroll
          for (int v = 0; v < 8; ++v) {
            const u32x4 q = {P[4 * v], P[4 * v + 1], P[4 * v + 2], P[4 * v + 3]};
            __builtin_amdgcn_raw_buffer_store_b128(q, ws, lane * 16u + (isw ? v * 1024u : (v & 3) * 1024u + (v >> 2) * 16384u), o, 16);
          }
        }
        WAIT_VM(0);
        if (!isw || dest != rank) {
          const u32 e = isw ? entry_w(id) : entry_x(id);
          if (lane < (u32)NR && lane != rank && (!isw || lane == dest))
            __hip_atomic_store((GAS u32*)((char*)prm.ws[lane] + flag_off(rank, e)), (call << 10) | size16, __ATOMIC_RELAXED, __HIP_MEMORY_SCOPE_SYSTEM);
        }
        if ((!isw || dest == rank) && lane == 0)
          __hip_atomic_fetch_add((GAS u32*)(wsl + OFF_CNT + (isw ? 16u + (id >> 4) : (id >> 5)) * 128u), 1u, __ATOMIC_RELAXED, __HIP_MEMORY_SCOPE_AGENT);
      }
    }
    STAMP(wsl, 1);
    const rsrc_t ro = make_rsrc(prm.out, (u32)MM * NN * 2u);
    const u32 wm = wid & 3u, wn = wid >> 2;
    for (u32 round = 0; round < 2u; ++round) {
      const u32 seq = round * 8u + xcd;
      u32 si, sj;
      st_coord(seq, si, sj);
      const u32 rb = si * 4u + (pos >> 2), cb = sj * 4u + (pos & 3u);
      if (wid == 0) {
        const GAS u32* ca = (const GAS u32*)(wsl + OFF_CNT + rb * 128u);
        const GAS u32* cbp = (const GAS u32*)(wsl + OFF_CNT + (16u + cb) * 128u);
        const u64 t0 = now();
        u32 dead = 0;
        while (rfl(__hip_atomic_load(ca, __ATOMIC_RELAXED, __HIP_MEMORY_SCOPE_AGENT)) < call * 256u ||
               rfl(__hip_atomic_load(cbp, __ATOMIC_RELAXED, __HIP_MEMORY_SCOPE_AGENT)) < call * 128u) {
          if (timed_out(tmo, call) || now() - t0 > DEADLINE) {
            if (lane == 0) __hip_atomic_store(tmo, call, __ATOMIC_RELAXED, __HIP_MEMORY_SCOPE_AGENT);
            dead = 1;
            break;
          }
          __builtin_amdgcn_s_sleep(4);
        }
        if (timed_out(tmo, call)) dead = 1;
        __builtin_amdgcn_fence(__ATOMIC_ACQUIRE, "agent");
        if (lane == 0) lds_st(smem, L_DEAD, dead);
      }
      asm volatile("s_waitcnt vmcnt(0) lgkmcnt(0)" ::: "memory");
      __builtin_amdgcn_s_barrier();
      CFENCE();
      const u32 dead = lds_ld(smem, L_DEAD);
      STAMP(wsl, 2 + round * 3);

      f32x4 acc[4][4];
#pragma unroll
      for (int a = 0; a < 4; ++a)
#pragma unroll
        for (int c = 0; c < 4; ++c) acc[a][c] = (f32x4){0.f, 0.f, 0.f, 0.f};
      const u32 ga = OFF_AT + rb * (2u << 20) + wid * 4096u, gb = OFF_BT + cb * (1u << 20) + wid * 2048u;
      const u32 la = (u32)(size_t)smem + wid * 4096u, lb = (u32)(size_t)smem + 32768u + wid * 2048u;
      const u32 vo = lane * 16u;
      auto stage = [&](u32 ks, u32 buf) {
#pragma unroll
        for (int u = 0; u < 4; ++u) dma16<0>(ws, la + buf * L_STAGE + u * 1024u, vo, ga + ks * 32768u + u * 1024u);
#pragma unroll
        for (int u = 0; u < 2; ++u) dma16<0>(ws, lb + buf * L_STAGE + u * 1024u, vo, gb + ks * 16384u + u * 1024u);
      };
      stage(0, 0);
      stage(1, 1);
      u32 buf = 0;
      const LAS u8* ra = smem + wm * 4096u + lane * 16u;
      const LAS u8* rbp = smem + 32768u + wn * 4096u + lane * 16u;
      for (u32 ks = 0; ks < 64u; ++ks) {
        if (ks < 63u) WAIT_VM(6); else WAIT_VM(0);
        WAIT_LGKM0();
        __builtin_amdgcn_s_barrier();
        CFENCE();
        if (ks + 2u < 64u) stage(ks + 2u, buf == 0 ? 2u : buf - 1u);
        const LAS u8* pa = ra + buf * L_STAGE;
        const LAS u8* pb = rbp + buf * L_STAGE;
        bf16x8 fa[2][4], fb[2][4];
#pragma unroll
        for (int h = 0; h < 2; ++h)
#pragma unroll
          for (int i = 0; i < 4; ++i) {
            fb[h][i] = *(const LAS bf16x8*)(pb + h * 8192 + i * 1024);
            fa[h][i] = *(const LAS bf16x8*)(pa + h * 16384 + i * 1024);
          }
#pragma unroll
        for (int h = 0; h < 2; ++h)
#pragma unroll
          for (int a = 0; a < 4; ++a)
#pragma unroll
            for (int c = 0; c < 4; ++c) acc[a][c] = __builtin_amdgcn_mfma_f32_16x16x32_bf16(fb[h][c], fa[h][a], acc[a][c], 0, 0, 0);
        buf = buf == 2u ? 0u : buf + 1u;
      }
      STAMP(wsl, 3 + round * 3);
      u32 P[32];
#pragma unroll
      for (int a = 0; a < 4; ++a)
#pragma unroll
        for (int c = 0; c < 4; ++c)
#pragma unroll
          for (int j = 0; j < 2; ++j) {
            const f32x2 v = {__builtin_elementwise_maximum(acc[a][c][2 * j], 0.f), __builtin_elementwise_maximum(acc[a][c][2 * j + 1], 0.f)};
            const u32 pk = __builtin_bit_cast(u32, __builtin_convertvector(v, bf16x2));
            P[a * 8 + c * 2 + j] = dead ? 0x7fc07fc0u : pk;
          }
      WAIT_LGKM0();
      __builtin_amdgcn_s_barrier();
      CFENCE();
      const u32 oc = seq * 128u + pos * 8u + wid;
      const u32 size16 = encode_chunk<true>(P, stg + 8192, exs, lutc, ws, OFF_EO + oc * 8192u, lane);
      WAIT_VM(0);
      if (lane < (u32)NR && lane != rank)
        __hip_atomic_store((GAS u32*)((char*)prm.ws[lane] + flag_off(rank, 768u + oc)), (call << 10) | size16, __ATOMIC_RELAXED, __HIP_MEMORY_SCOPE_SYSTEM);
      {
        const u32 voff = (lane & 15u) * (NN * 2u) + (lane >> 4) * 16u;
        const u32 soff = (rb * 256u + wm * 64u) * (NN * 2u) + (rank * NSH + cb * 128u + wn * 64u) * 2u;
#pragma unroll
        for (int v = 0; v < 8; ++v) {
          const u32x4 q = {P[4 * v], P[4 * v + 1], P[4 * v + 2], P[4 * v + 3]};
          __builtin_amdgcn_raw_buffer_store_b128(q, ro, voff + (v & 1) * 64u, soff + (v >> 1) * 16u * (NN * 2u), 0);
        }
      }
      STAMP(wsl, 4 + round * 3);
    }
  } else {
    __syncthreads();
    const u32 call = lds_ld(smem, L_CALL);
    const u32 cid = (pos - 16u) * 8u + xcd;
    if (cid >= 7u * NL) return;
    const u32 jl = cid / 7u, src = (rank + 1u + cid % 7u) & 7u;
    const u32 nmine = (NENT - jl + NL - 1) / NL;
    if (wid == 0) {
      const rsrc_t rp = make_rsrc(prm.ws[src], WS_NEED);
      const u32 frow = OFF_FLAG + (src * NL + jl) * FROW * 4u;
      const u32 lbase = (u32)(size_t)smem;
      u32 issued = 0, pub = 0, dead = 0, slot = 0;
      for (u32 idx = 0; idx < nmine; ++idx) {
        const u32 e = jl + NL * idx;
        u32 f = lds_ld(smem, L_FLAGS + idx * 4u);
        const u32 need_free = idx >= 14u ? idx - 13u : 0u;
        if ((f >> 10) != call || lds_ld(smem, L_FREE + slot * 4u) < need_free) {
          const u64 t0 = now();
          for (;;) {
            if ((f >> 10) != call) dma4(ws, lbase + L_FLAGS + (idx & ~63u) * 4u, lane * 4u, frow + (idx & ~63u) * 4u);
            WAIT_VM(0);
            pub = issued;
            lds_st(smem, L_LANDED, issued);
            f = lds_ld(smem, L_FLAGS + idx * 4u);
            if ((f >> 10) == call && lds_ld(smem, L_FREE + slot * 4u) >= need_free) break;
            if (dead || timed_out(tmo, call) || now() - t0 > DEADLINE) {
              if (lane == 0) __hip_atomic_store(tmo, call, __ATOMIC_RELAXED, __HIP_MEMORY_SCOPE_AGENT);
              lds_st(smem, L_DEAD, 1u);
              dead = 1;
              break;
            }
            __builtin_amdgcn_s_sleep(1);
          }
          if (dead) break;
        }
        u32 size16 = f & 1023u;
        size16 = size16 > 512u ? 512u : size16;
        u32 kind, id;
        entry_inv(e, kind, id);
        const u32 go = kind == 0 ? OFF_EW + (rank * 256u + id) * 8192u : kind == 1 ? OFF_EX + id * 8192u : OFF_EO + id * 8192u;
        const u32 np = (size16 + 63u) >> 6;
        lds_st(smem, L_META + slot * 4u, (idx + 1u) | (size16 << 9) | ((issued + np) << 19));
        for (u32 pc = 0; pc < np; ++pc) {
          const u32 o16 = pc * 64u + lane;
          dma16<2>(rp, lbase + slot * 8192u + pc * 1024u, o16 < size16 ? o16 * 16u : 0x80000000u, go);
          ++issued;
          asm volatile("s_waitcnt vmcnt(" STR(WINDOW) " - 1)" ::: "memory");
          if (issued >= pub + WINDOW) {
            pub = issued - (WINDOW - 1);
            lds_st(smem, L_LANDED, pub);
          }
        }
        {
          const u32 nb = (idx + 2u) & ~63u;
          dma4(ws, lbase + L_FLAGS + nb * 4u, lane * 4u, frow + nb * 4u);
          ++issued;
        }
        slot = slot == 13u ? 0u : slot + 1u;
      }
      WAIT_VM(0);
      lds_st(smem, L_LANDED, issued);
      STAMP(wsl, 3);
    } else {
      const rsrc_t ro = make_rsrc(prm.out, (u32)MM * NN * 2u);
      LAS u8* hs = smem + L_HS + (wid - 1u) * L_HS_STRIDE;
      const LAS u32x2* lutx = (const LAS u32x2*)(smem + L_LUT);
      u32 dead = 0;
      for (u32 idx = wid - 1u; idx < nmine; idx += 7u) {
        const u32 slot = idx % 14u;
        u32 size16 = 0;
        if (!dead) {
          const u64 t0 = now();
          for (;;) {
            const u32 mt = lds_ld(smem, L_META + slot * 4u);
            if ((mt & 511u) == idx + 1u && lds_ld(smem, L_LANDED) >= (mt >> 19)) { size16 = (mt >> 9) & 1023u; break; }
            if (lds_ld(smem, L_DEAD) || now() - t0 > 2 * DEADLINE) { dead = 1; break; }
            __builtin_amdgcn_s_sleep(2);
          }
        }
        CFENCE();
        u32 kind, id;
        entry_inv(jl + NL * idx, kind, id);
        u32 P[32];
        if (dead) {
#pragma unroll
          for (int i = 0; i < 32; ++i) P[i] = 0x7fc07fc0u;
        } else if (kind == 2) {
          decode_chunk<true>(smem + slot * 8192u, hs, lutx, size16, lane, P);
        } else {
          decode_chunk<false>(smem + slot * 8192u, hs, lutx, size16, lane, P);
        }
#pragma unroll
        for (int i = 0; i < 32; ++i) asm volatile("" : "+v"(P[i]));
        WAIT_LGKM0();
        lds_st(smem, L_FREE + slot * 4u, idx + 1u);
        if (kind == 2) {
          u32 si, sj;
          st_coord(id >> 7, si, sj);
          const u32 t = (id >> 3) & 15u, w = id & 7u;
          const u32 voff = (lane & 15u) * (NN * 2u) + (lane >> 4) * 16u;
          const u32 soff = ((si * 4u + (t >> 2)) * 256u + (w & 3u) * 64u) * (NN * 2u) + (src * NSH + (sj * 4u + (t & 3u)) * 128u + (w >> 2) * 64u) * 2u;
#pragma unroll
          for (int v = 0; v < 8; ++v) {
            const u32x4 q = {P[4 * v], P[4 * v + 1], P[4 * v + 2], P[4 * v + 3]};
            __builtin_amdgcn_raw_buffer_store_b128(q, ro, voff + (v & 1) * 64u, soff + (v >> 1) * 16u * (NN * 2u), 0);
          }
        } else {
          u32 o;
          if (kind == 0) o = OFF_BT + (id >> 4) * (1u << 20) + (src * 16u + (id & 15u)) * 8192u;
          else o = OFF_AT + (id >> 5) * (2u << 20) + (src * 8u + ((id >> 2) & 7u)) * 32768u + (id & 3u) * 4096u;
          const u32 hstride = kind == 0 ? 4096u : 16384u;
#pragma unroll
          for (int v = 0; v < 8; ++v) {
            const u32x4 q = {P[4 * v], P[4 * v + 1], P[4 * v + 2], P[4 * v + 3]};
            __builtin_amdgcn_raw_buffer_store_b128(q, ws, lane * 16u + (v & 3) * 1024u, o + (v >> 2) * hstride, 16);
          }
          WAIT_VM(0);
          if (lane == 0)
            __hip_atomic_fetch_add((GAS u32*)(wsl + OFF_CNT + (kind == 0 ? 16u + (id >> 4) : (id >> 5)) * 128u), 1u, __ATOMIC_RELAXED, __HIP_MEMORY_SCOPE_AGENT);
        }
      }
      STAMP(wsl, 3);
    }
  }
}

extern "C" void kernel_launch(int rank, int nranks, ncclComm_t comm, hipStream_t stream, void* const* d_in, const int* in_sizes,
                              int n_in, void* d_out, int out_size, const int* in_dtypes, int out_dtype, void* const* d_out_all,
                              void* const* d_ws_all, size_t ws_bytes) {
  (void)comm; (void)in_sizes; (void)n_in; (void)out_size; (void)in_dtypes; (void)out_dtype; (void)d_out_all;
  if (nranks != NR || ws_bytes < WS_NEED || (unsigned)rank >= (unsigned)NR) {
    fprintf(stderr, "unsupported configuration\n");
    return;
  }
  Params p;
  p.x = d_in[0];
  p.w = d_in[1];
  p.out = d_out;
  for (int i = 0; i < NR; ++i) p.ws[i] = d_ws_all[i];
  p.rank = rank;
  p.pad = 0;
  gemm_ar_kernel<<<NWG, NTHR, 0, stream>>>(p);
}
```

```cpp
#include <hip/hip_runtime.h>
#include <rccl/rccl.h>
#include <cstdio>

typedef unsigned u32;
typedef unsigned long long u64;
typedef unsigned char u8;
typedef u32 u32x2 __attribute__((ext_vector_type(2)));
typedef u32 u32x4 __attribute__((ext_vector_type(4)));
typedef float f32x2 __attribute__((ext_vector_type(2)));
typedef float f32x4 __attribute__((ext_vector_type(4)));
typedef short s16x2 __attribute__((ext_vector_type(2)));
typedef short s16x4 __attribute__((ext_vector_type(4)));
typedef __bf16 bf16x2 __attribute__((ext_vector_type(2)));
typedef __bf16 bf16x8 __attribute__((ext_vector_type(8)));
#define LAS __attribute__((address_space(3)))
#define GAS __attribute__((address_space(1)))
typedef __amdgpu_buffer_rsrc_t rsrc_t;

namespace {

constexpr int NR = 8;
constexpr int MM = 4096, NN = 16384, KSH = 512, NSH = 2048;
constexpr int NWG = 256, NTHR = 512;
constexpr int NL = 18;
constexpr int NENT = 768 + 2048;
constexpr int NTK = 768 + 1024;
constexpr int NSLOT = 14, NDEC = 6, NQ = 32;
constexpr int NTASK = 2560;
#define WINDOW 4
#define LOOK 2
#define OAUX 16
constexpr u64 DEADLINE = 30000000ull;

constexpr u32 OFF_CALL = 0;
constexpr u32 OFF_TMO = 16384;
constexpr u32 OFF_TK = 17408;
constexpr u32 OFF_CNT = 20480;
constexpr u32 OFF_FLAG = 32768;
constexpr u32 OFF_DBG = 512u << 10;
constexpr u32 OFF_EX = 1u << 20;
constexpr u32 OFF_EW = 5u << 20;
constexpr u32 OFF_EO = 21u << 20;
constexpr u32 OFF_AT = 37u << 20;
constexpr u32 OFF_BT = 69u << 20;
constexpr u32 WS_NEED = 85u << 20;

constexpr int L_STAGE = 49152;
constexpr int L_HS = NSLOT * 8192;
constexpr int L_HS_STRIDE = 4160;
constexpr int L_MISC = 3 * L_STAGE;
constexpr int L_LUT = L_MISC;
constexpr int L_REQ = L_MISC + 128;
constexpr int L_LEND = L_REQ + 4 * NQ;
constexpr int L_FREE = L_LEND + 64;
constexpr int L_LANDED = L_FREE + 64;
constexpr int L_LPOS = L_LANDED + 4;
constexpr int L_NEND = L_LANDED + 8;
constexpr int L_DEAD = L_LANDED + 12;
constexpr int L_CALL = L_LANDED + 16;
constexpr int L_EXC = L_MISC + 512;
constexpr int L_TOTAL = L_EXC + 8 * 256;
static_assert(L_HS + NDEC * L_HS_STRIDE <= L_MISC, "lds");
static_assert(L_CALL + 4 <= L_EXC, "lds");
static_assert(L_TOTAL <= 163840, "lds");

constexpr u32 LO_OFF = 528;
constexpr u32 UB_IN = LO_OFF + 4096;

struct Params {
  const void* x;
  const void* w;
  void* out;
  void* ws[NR];
  int rank;
  int pad;
};

#define CFENCE() asm volatile("" ::: "memory")
#define WAIT_VM(n) asm volatile("s_waitcnt vmcnt(" #n ")" ::: "memory")
#define WAIT_LGKM0() asm volatile("s_waitcnt lgkmcnt(0)" ::: "memory")
#define STR2(x) #x
#define STR(x) STR2(x)

__device__ __forceinline__ u32 rfl(u32 v) { return __builtin_amdgcn_readfirstlane(v); }
__device__ __forceinline__ u32 vperm(u32 hi, u32 lo, u32 sel) { return __builtin_amdgcn_perm(hi, lo, sel); }
__device__ __forceinline__ u32 bsum(u32 v, u32 acc) { return __builtin_amdgcn_sad_u8(v, 0u, acc); }
__device__ __forceinline__ u64 now() { return __builtin_amdgcn_s_memrealtime(); }
__device__ __forceinline__ u32 mbcnt(u64 m) {
  return __builtin_amdgcn_mbcnt_hi((u32)(m >> 32), __builtin_amdgcn_mbcnt_lo((u32)m, 0u));
}
__device__ __forceinline__ u32 scan_incl(u32 x) {
  x += __builtin_amdgcn_update_dpp(0u, x, 0x111, 0xf, 0xf, true);
  x += __builtin_amdgcn_update_dpp(0u, x, 0x112, 0xf, 0xf, true);
  x += __builtin_amdgcn_update_dpp(0u, x, 0x114, 0xf, 0xf, true);
  x += __builtin_amdgcn_update_dpp(0u, x, 0x118, 0xf, 0xf, true);
  x += __builtin_amdgcn_update_dpp(0u, x, 0x142, 0xa, 0xf, false);
  x += __builtin_amdgcn_update_dpp(0u, x, 0x143, 0xc, 0xf, false);
  return x;
}
__device__ __forceinline__ rsrc_t make_rsrc(const void* p, u32 bytes) {
  return __builtin_amdgcn_make_buffer_rsrc((void*)p, 0, bytes, 0x00020000);
}
template <int IMM>
__device__ __forceinline__ void dma16(rsrc_t r, u32 lds, u32 voff, u32 soff) {
  asm volatile("s_mov_b32 m0, %0\n\ts_nop 0\n\tbuffer_load_dwordx4 %1, %2, %3 offen offset:%4 lds"
               :: "s"(lds), "v"(voff), "s"(r), "s"(soff), "i"(IMM) : "memory");
}
__device__ __forceinline__ void dma16_sys(rsrc_t r, u32 lds, u32 voff, u32 soff) {
  asm volatile("s_mov_b32 m0, %0\n\ts_nop 3\n\tbuffer_load_dwordx4 %1, %2, %3 offen sc0 sc1 lds"
               :: "s"(lds), "v"(voff), "s"(r), "s"(soff) : "memory");
}
__device__ __forceinline__ u32 lds_ld(LAS u8* lds, int off) {
  return rfl(*(volatile LAS u32*)(lds + off));
}
__device__ __forceinline__ void lds_st(LAS u8* lds, int off, u32 v) { *(volatile LAS u32*)(lds + off) = v; }

#define STAMP(ws, slot) do {} while (0)
#define STAMPV(ws, slot, v) do {} while (0)
#define DBG(x)

template <bool OUT>
__device__ __forceinline__ u32 encode_chunk(const u32 (&P)[32], LAS u8* bld, LAS u32* exs, const LAS u32* lutc, rsrc_t ws,
                                            u32 goff, u32 lane) {
  u32 lo[16], hi[16], nzm[16], len[16];
  u32 pl[2] = {0u, 0u};
  u32 sum = 0, sgn = 0;
#pragma unroll
  for (int g = 0; g < 16; ++g) {
    lo[g] = vperm(P[2 * g + 1], P[2 * g], 0x06040200u);
    hi[g] = vperm(P[2 * g + 1], P[2 * g], 0x07050301u);
    if (OUT) {
      const s16x2 z = {0, 0};
      const u32 n0 = __builtin_bit_cast(u32, z - __builtin_bit_cast(s16x2, P[2 * g]));
      const u32 n1 = __builtin_bit_cast(u32, z - __builtin_bit_cast(s16x2, P[2 * g + 1]));
      nzm[g] = vperm(n1, n0, 0x0b0a0908u);
      pl[g >> 3] |= bsum(nzm[g] & 0x08040201u, 0u) << (4 * (g & 7));
      sgn |= hi[g];
    } else {
      pl[g >> 3] |= (hi[g] >> (7 - (g & 7))) & (0x01010101u << (g & 7));
      hi[g] &= 0x7f7f7f7fu;
      nzm[g] = 0xffffffffu;
    }
    sum = bsum(hi[g], sum);
  }
  const bool signed_out = OUT && __builtin_amdgcn_ballot_w64((sgn & 0x80808080u) != 0u) != 0ull;
  const u32 cnt = OUT ? (u32)(__builtin_popcount(pl[0]) + __builtin_popcount(pl[1])) : 64u;
  const u32 sc0 = scan_incl((sum & 0x1fffu) | (cnt << 19));
  const u32 tot0 = __builtin_amdgcn_readlane(sc0, 63);
  const u32 nnz = tot0 >> 19;
  const u32 pbase = (sc0 >> 19) - cnt;
  const float mean = (float)(tot0 & 0x7ffffu) * __builtin_amdgcn_rcpf((float)(nnz ? nnz : 1u));
  int b = (int)(mean + 0.78f);
  const float fr = mean - (float)(b - 1);
  const u32 k = rfl(1u + (fr < 0.32f) + (fr < 0.41f) + (fr < 0.60f) + (fr < 0.87f));
  b = b < 6 ? 6 : b > 126 ? 126 : b;
  const u32 base4 = rfl((u32)(b - 6)) * 0x01010101u;
  const u32 cneg = 0x80808080u - base4;
  const u64 below = (1ull << (8 * k)) - 1ull;
  const u64 dec = (0x0001020304050607ull & ~below & ~(0xffull << (8 * k))) | (7ull << (8 * k)) |
                  (0x0000010203040506ull & below);
  const u64 inv = (0x0001020304050607ull + (0x0101010101010101ull >> (8 * (k + 1)))) | ((u64)(k + 1) << 56);
  const u32 dec_lo = (u32)dec + base4, dec_hi = (u32)(dec >> 32) + base4;
  u32 inv_lo = (u32)inv;
  const u32 inv_hi = (u32)(inv >> 32);
  asm volatile("" : "+v"(inv_lo));

  u32 anybad = 0, nexc = 0;
#pragma unroll
  for (int g = 0; g < 16; ++g) {
    const u32 e4 = (hi[g] + cneg) ^ 0x80808080u;
    anybad |= e4 & nzm[g];
    len[g] = vperm(inv_hi, inv_lo, e4) & nzm[g];
    if (OUT) lo[g] = vperm(0u, lo[g], lutc[(pl[g >> 3] >> (4 * (g & 7))) & 15u]);
  }
  if (__builtin_amdgcn_ballot_w64((anybad & 0xf8f8f8f8u) != 0u) != 0ull) {
    u32 bp[2] = {0u, 0u};
#pragma unroll
    for (int g = 0; g < 16; ++g) {
      const u32 bad = ((hi[g] + cneg) ^ 0x80808080u) & 0xf8f8f8f8u & nzm[g];
      const u32 t = ((bad >> 1) + 0x7c7c7c7cu) & 0x80808080u;
      const u32 fm = (t << 1) - (t >> 7);
      len[g] = (len[g] & ~fm) | (t >> 7);
      bp[g >> 3] |= bsum(fm & 0x08040201u, 0u) << (4 * (g & 7));
    }
#pragma unroll
    for (int j = 0; j < 4; ++j) {
      const u32x4 t = {hi[4 * j], hi[4 * j + 1], hi[4 * j + 2], hi[4 * j + 3]};
      *(LAS u32x4*)(bld + lane * 64u + j * 16) = t;
    }
    CFENCE();
    u64 bm = ((u64)bp[1] << 32) | bp[0];
    const u64 plm = ((u64)pl[1] << 32) | pl[0];
    for (;;) {
      const u64 m = __builtin_amdgcn_ballot_w64(bm != 0ull);
      if (m == 0ull || nexc > 64u) break;
      if (bm != 0ull) {
        const u32 j = __builtin_ctzll(bm);
        bm &= bm - 1ull;
        const u32 slot = nexc + mbcnt(m);
        const u32 idx = pbase + (OUT ? (u32)__builtin_popcountll(plm & ((1ull << j) - 1ull)) : j);
        if (slot < 64u) exs[slot] = (idx << 8) | bld[lane * 64u + j];
      }
      nexc += __builtin_popcountll(m);
    }
    CFENCE();
  }
  u32 bits = 0;
#pragma unroll
  for (int g = 0; g < 16; ++g) bits = bsum(len[g], bits);
  const u32 sc1 = scan_incl(bits);
  const u32 tbits = __builtin_amdgcn_readlane(sc1, 63);
  const u32 nw = (tbits + 31u) >> 5;
  const u32 ubase = OUT ? LO_OFF + ((nnz + 3u) & ~3u) : UB_IN;
  const u32 xbase = ubase + 4u * nw;
  const u32 size16 = (xbase + 4u * nexc + 15u) >> 4;
  if (size16 >= 512u || nexc > 64u || signed_out) {
#pragma unroll
    for (int v = 0; v < 8; ++v) {
      const u32x4 q = {P[4 * v], P[4 * v + 1], P[4 * v + 2], P[4 * v + 3]};
      __builtin_amdgcn_raw_buffer_store_b128(q, ws, lane * 16u + v * 1024u, goff, 17);
    }
    return 512u;
  }
  {
    const u32x4 z = {0u, 0u, 0u, 0u};
    for (u32 o = (OUT ? LO_OFF : UB_IN); o < size16 * 16u; o += 1024u) *(LAS u32x4*)(bld + o + lane * 16u) = z;
  }
  CFENCE();
  u32 q = sc1 - bits + ((u32)(size_t)bld + ubase) * 8u;
  u32 p = pbase + (u32)(size_t)bld + LO_OFF;
  u32 ones = 0x01010101u;
  asm volatile("" : "+v"(ones));
#pragma unroll
  for (int g = 0; g < 16; ++g) {
    u32 t = len[g] + 0x1fu;
    t += t << 8;
    asm volatile("" : "+v"(t));
    t += t << 16;
    const u32 one = OUT ? (nzm[g] & 0x01010101u) : ones;
    u32 b4, b5, b6;
    asm("v_lshlrev_b32_sdwa %0, %3, %4 dst_sel:DWORD dst_unused:UNUSED_PAD src0_sel:BYTE_0 src1_sel:BYTE_0\n\t"
        "v_lshlrev_b32_sdwa %1, %3, %4 dst_sel:DWORD dst_unused:UNUSED_PAD src0_sel:BYTE_1 src1_sel:BYTE_1\n\t"
        "v_lshlrev_b32_sdwa %2, %3, %4 dst_sel:DWORD dst_unused:UNUSED_PAD src0_sel:BYTE_2 src1_sel:BYTE_2\n\t"
        "v_or3_b32 %0, %0, %1, %2\n\t"
        "v_lshlrev_b32_sdwa %1, %3, %4 dst_sel:DWORD dst_unused:UNUSED_PAD src0_sel:BYTE_3 src1_sel:BYTE_3\n\t"
        "v_or_b32 %0, %0, %1"
        : "=&v"(b4), "=&v"(b5), "=&v"(b6) : "v"(t), "v"(one));
    const u64 v = (u64)b4 << (q & 31u);
    LAS u32* d = (LAS u32*)(size_t)((q >> 3) & ~3u);
    __hip_atomic_fetch_or(d, (u32)v, __ATOMIC_RELAXED, __HIP_MEMORY_SCOPE_WAVEFRONT);
    __hip_atomic_fetch_or(d + 1, (u32)(v >> 32), __ATOMIC_RELAXED, __HIP_MEMORY_SCOPE_WAVEFRONT);
    q = bsum(len[g], q);
    if (OUT) {
      const u64 u = (u64)lo[g] << ((p & 3u) * 8u);
      LAS u32* e = (LAS u32*)(size_t)(p & ~3u);
      __hip_atomic_fetch_or(e, (u32)u, __ATOMIC_RELAXED, __HIP_MEMORY_SCOPE_WAVEFRONT);
      __hip_atomic_fetch_or(e + 1, (u32)(u >> 32), __ATOMIC_RELAXED, __HIP_MEMORY_SCOPE_WAVEFRONT);
      p = bsum(one, p);
    }
  }
  if (!OUT) {
#pragma unroll
    for (int j = 0; j < 4; ++j) {
      const u32x4 t = {lo[4 * j], lo[4 * j + 1], lo[4 * j + 2], lo[4 * j + 3]};
      *(LAS u32x4*)(bld + LO_OFF + j * 1024 + lane * 16u) = t;
    }
  }
  {
    const u32x2 t = {pl[0], pl[1]};
    *(LAS u32x2*)(bld + 16 + lane * 8u) = t;
    if (lane == 0) {
      const u32x4 h = {dec_lo, dec_hi, nw | (nexc << 16), nnz};
      *(LAS u32x4*)bld = h;
    }
  }
  CFENCE();
  if (lane < nexc) *(LAS u32*)(bld + xbase + 4u * lane) = exs[lane];
  CFENCE();
  constexpr int NB = OUT ? 4 : 6;
  for (u32 o = 0; o < size16; o += 64u * NB) {
    u32x4 t[NB];
#pragma unroll
    for (int j = 0; j < NB; ++j) t[j] = *(const LAS u32x4*)(bld + ((o * 16u + j * 1024u) & 8191u) + lane * 16u);
#pragma unroll
    for (int j = 0; j < NB; ++j) {
      const u32 o16 = o + j * 64u + lane;
      __builtin_amdgcn_raw_buffer_store_b128(t[j], ws, o16 < size16 ? o16 * 16u : 0x80000000u, goff, 17);
    }
  }
  CFENCE();
  return size16;
}

template <bool OUT>
__device__ __forceinline__ void decode_chunk(const LAS u8* slot, LAS u8* hs, const LAS u32x2* lutx, u32 size16, u32 lane,
                                             u32 (&P)[32]) {
  if (size16 >= 512u) {
#pragma unroll
    for (int v = 0; v < 8; ++v) {
      const u32x4 t = *(const LAS u32x4*)(slot + v * 1024 + lane * 16u);
      P[4 * v] = t.x; P[4 * v + 1] = t.y; P[4 * v + 2] = t.z; P[4 * v + 3] = t.w;
    }
    return;
  }
  const u32x4 h = *(const LAS u32x4*)slot;
  const u32 tab_lo = h.x, tab_hi = rfl(h.y), hz = rfl(h.z), nnz = rfl(h.w) & 0x1fffu;
  const u32 nw = hz & 0x7ffu, nexc = (hz >> 16) & 0x7fu;
  const u32 ubase = OUT ? LO_OFF + ((nnz + 3u) & ~3u) : UB_IN;
  const LAS u32* U = (const LAS u32*)(slot + ubase);
  u32 run = 0;
  for (u32 w0 = 0; w0 < nw; w0 += 64u) {
    const u32 i = w0 + lane;
    u32 w = U[i], prev = U[(int)i - 1];
    w = i < nw ? w : 0u;
    prev = i > 0u ? prev : 0x80000000u;
    const u32 c = __builtin_popcount(w);
    const u32 incl = scan_incl(c);
    const u32 dst = (u32)(size_t)hs + ((run + incl - c) & 4095u);
    run += __builtin_amdgcn_readlane(incl, 63);
    u32 q0 = ~(u32)__builtin_clz(prev | 1u), q1, t, sy;
    u64 sv;
#define PSTEP(QN, QO, I) \
    "v_ffbl_b32 " QN ", %0\n\tv_add_u32 %3, -1, %0\n\tv_xad_u32 %4, " QO ", -1, " QN "\n\tv_and_b32 %0, %3, %0\n\t" \
    "v_perm_b32 %4, %7, %8, %4\n\tds_write_b8 %6, %4 offset:" #I "\n\tv_cmpx_ne_u32 vcc, 0, %0\n\t"
#define PSTEP2(I, J) PSTEP("%2", "%1", I) PSTEP("%1", "%2", J)
#define PEXIT "s_cbranch_execz 1f\n\t"
    asm volatile("s_mov_b64 %5, exec\n\tv_cmpx_ne_u32 vcc, 0, %0\n\t"
                 PSTEP2(0, 1) PSTEP2(2, 3) PSTEP2(4, 5) PSTEP2(6, 7) PEXIT PSTEP2(8, 9) PSTEP2(10, 11) PSTEP2(12, 13) PSTEP2(14, 15) PEXIT
                 PSTEP2(16, 17) PSTEP2(18, 19) PEXIT PSTEP2(20, 21) PEXIT PSTEP2(22, 23) PEXIT PSTEP2(24, 25) PEXIT PSTEP2(26, 27) PEXIT
                 PSTEP2(28, 29) PEXIT PSTEP2(30, 31)
                 "1:\n\ts_mov_b64 exec, %5\n\ts_nop 4"
                 : "+v"(w), "+v"(q0), "=&v"(q1), "=&v"(t), "=&v"(sy), "=&s"(sv)
                 : "v"(dst), "s"(tab_hi), "v"(tab_lo) : "vcc", "memory");
  }
  CFENCE();
  if (lane < nexc) {
    const u32 r = U[nw + lane];
    hs[(r >> 8) & 4095u] = (u8)r;
  }
  CFENCE();
  const u32x2 pl = *(const LAS u32x2*)(slot + 16 + lane * 8u);
  if (OUT) {
    const u32 c = __builtin_popcount(pl.x) + __builtin_popcount(pl.y);
    u32 p = scan_incl(c) - c;
#pragma unroll
    for (int g = 0; g < 16; ++g) {
      const u32 m4 = ((g < 8 ? pl.x : pl.y) >> (4 * (g & 7))) & 15u;
      const u32x2 sel = lutx[m4];
      const u32 a = p & ~3u;
      const LAS u32* lp = (const LAS u32*)(slot + LO_OFF + a);
      const LAS u32* hp = (const LAS u32*)(hs + a);
      const u32 l4 = __builtin_amdgcn_alignbyte(lp[1], lp[0], p);
      const u32 h4 = __builtin_amdgcn_alignbyte(hp[1], hp[0], p);
      P[2 * g] = vperm(h4, l4, sel.x);
      P[2 * g + 1] = vperm(h4, l4, sel.y);
      p += __builtin_popcount(m4);
    }
  } else {
#pragma unroll
    for (int j = 0; j < 4; ++j) {
      const u32x4 L = *(const LAS u32x4*)(slot + LO_OFF + j * 1024 + lane * 16u);
      const u32x4 H = *(const LAS u32x4*)(hs + lane * 64u + j * 16);
#pragma unroll
      for (int i = 0; i < 4; ++i) {
        const int g = 4 * j + i;
        const u32 hh = H[i] | (((g < 8 ? pl.x : pl.y) << (7 - (g & 7))) & 0x80808080u);
        P[2 * g] = vperm(hh, L[i], 0x05010400u);
        P[2 * g + 1] = vperm(hh, L[i], 0x07030602u);
      }
    }
  }
}

__device__ __forceinline__ u32 entry_w(u32 wl) { const u32 q = wl >> 6; return (q == 0 ? 0u : q == 1 ? 192u : q == 2 ? 384u : 448u) + (wl & 63u); }
__device__ __forceinline__ u32 entry_x(u32 xc) { const u32 q = xc >> 7; return (q == 0 ? 64u : q == 1 ? 256u : q == 2 ? 512u : 640u) + (xc & 127u); }
__device__ __forceinline__ void entry_inv(u32 e, u32& kind, u32& id) {
  if (e >= 768u) { kind = 2; id = e - 768u; }
  else if (e < 64u) { kind = 0; id = e; }
  else if (e < 192u) { kind = 1; id = e - 64u; }
  else if (e < 256u) { kind = 0; id = e - 128u; }
  else if (e < 384u) { kind = 1; id = e - 128u; }
  else if (e < 512u) { kind = 0; id = e - 256u; }
  else { kind = 1; id = e - 256u; }
}
__device__ __forceinline__ void st_coord(u32 seq, u32& i, u32& j) {
  if (seq >= 8u) { i = seq >> 2; j = seq & 3u; }
  else if (seq < 4u) { i = seq >> 1; j = seq & 1u; }
  else { i = seq & 1u; j = seq >> 1; }
}
__device__ __forceinline__ u32 flag_off(u32 src, u32 e) { return OFF_FLAG + (src * NENT + e) * 4u; }

__device__ __forceinline__ bool timed_out(const GAS u32* tmo, u32 call) {
  return rfl(__hip_atomic_load(tmo, __ATOMIC_RELAXED, __HIP_MEMORY_SCOPE_AGENT)) == call;
}

}

__global__ __launch_bounds__(NTHR, 2) void gemm_ar_kernel(Params prm) {
  __shared__ __attribute__((aligned(16))) u8 smem_raw[L_TOTAL];
  LAS u8* smem = (LAS u8*)smem_raw;
  const u32 tid = threadIdx.x, lane = tid & 63u, wid = rfl(tid >> 6);
  const u32 rank = (u32)prm.rank;
  char* wsl = (char*)prm.ws[rank & 7u];
  const rsrc_t ws = make_rsrc(wsl, WS_NEED);
  GAS u32* tmo = (GAS u32*)(wsl + OFF_TMO);
  const u32 xcd = blockIdx.x & 7u, pos = blockIdx.x >> 3;
  STAMP(wsl, 0);

  if (tid == 0) {
    GAS u32* cc = (GAS u32*)(wsl + OFF_CALL + blockIdx.x * 64u);
    const u32 c = __hip_atomic_load(cc, __ATOMIC_RELAXED, __HIP_MEMORY_SCOPE_AGENT) + 1u;
    __hip_atomic_store(cc, c, __ATOMIC_RELAXED, __HIP_MEMORY_SCOPE_AGENT);
    lds_st(smem, L_CALL, c);
  }
  if (tid >= 64u && tid < 80u) {
    const u32 m = tid - 64u;
    u32 cs = 0x0c0c0c0cu, r = 0;
    u64 xs = 0x0c0c0c0c0c0c0c0cull;
    for (u32 i = 0; i < 4; ++i)
      if ((m >> i) & 1u) {
        cs = (cs & ~(0xffu << (8 * r))) | (i << (8 * r));
        xs = (xs & ~(0xffffull << (16 * i))) | ((u64)(r | ((4u + r) << 8)) << (16 * i));
        ++r;
      }
    if (pos < 16u) *(LAS u32*)(smem + L_LUT + 4 * m) = cs;
    else *(LAS u64*)(smem + L_LUT + 8 * m) = xs;
  }
  if (tid >= 128u && tid < 128u + (L_CALL - L_REQ) / 4u) *(LAS u32*)(smem + L_REQ + (tid - 128u) * 4u) = 0u;

  if (pos < 16u) {
    LAS u8* bld = smem + wid * 18432u;
    LAS u8* stg = bld + 10240;
    LAS u32* exs = (LAS u32*)(smem + L_EXC + wid * 256u);
    const LAS u32* lutc = (const LAS u32*)(smem + L_LUT);
    const rsrc_t rx = make_rsrc(prm.x, MM * KSH * 2), rw = make_rsrc(prm.w, KSH * NN * 2);
    u32 call = 0;
    {
      const u32 gw = (pos * 8u + xcd) * 8u + wid;
      for (u32 t = gw; t < (u32)NTASK; t += 1024u) {
        u32 isw, id, dest = 0;
        if (t < 512u) { isw = 1; id = t; }
        else if (t < 640u) { isw = 0; id = t - 512u; }
        else if (t < 1152u) { isw = 1; id = t - 128u; }
        else if (t < 1280u) { isw = 0; id = t - 1024u; }
        else if (t < 2304u) { isw = 1; id = t - 256u; }
        else { isw = 0; id = t - 2048u; }
        u32 P[32];
        if (isw) {
          dest = id & 7u;
          id >>= 3;
          const u32 cb = id >> 4, kb = id & 15u;
          const u32 voff = (lane >> 4) * (NN * 2) + (lane & 15u) * 16u;
          const u32 soff = kb * 32u * (NN * 2) + (dest * NSH + cb * 128u) * 2u;
          asm volatile("s_nop 4");
#pragma unroll
          for (int u = 0; u < 8; ++u) dma16<0>(rw, (u32)(size_t)stg + u * 1024u, voff, soff + u * 4u * (NN * 2));
          WAIT_VM(0);
          const LAS u8* tb = stg + ((lane >> 4) * 8u + ((lane & 15u) >> 2)) * 256u + (lane & 3u) * 16u;
#pragma unroll
          for (int nf = 0; nf < 8; ++nf)
#pragma unroll
            for (int h = 0; h < 2; ++h) {
              const s16x4 r = __builtin_amdgcn_ds_read_tr16_b64_v4i16((LAS s16x4*)(tb + h * 1024 + (nf >> 1) * 64 + (nf & 1) * 8));
              const u32x2 r2 = __builtin_bit_cast(u32x2, r);
              P[4 * nf + 2 * h] = r2.x;
              P[4 * nf + 2 * h + 1] = r2.y;
            }
        } else {
          const u32 rb = id >> 5, ksl = (id >> 2) & 7u, mq = id & 3u;
          const u32 voff = (lane & 15u) * (KSH * 2) + (lane >> 4) * 16u;
          const u32 soff = (rb * 256u + mq * 64u) * (KSH * 2) + ksl * 128u;
#pragma unroll
          for (int v = 0; v < 8; ++v) {
            const u32x4 r = __builtin_amdgcn_raw_buffer_load_b128(rx, voff + (v & 3) * 16 * (KSH * 2) + (v >> 2) * 64, soff, 0);
            P[4 * v] = r.x; P[4 * v + 1] = r.y; P[4 * v + 2] = r.z; P[4 * v + 3] = r.w;
          }
        }
        if (call == 0) {
          __syncthreads();
          call = lds_ld(smem, L_CALL);
        }
        u32 size16 = 0;
        DBG(if (t == gw) { for (int i = 0; i < 32; ++i) asm volatile("" : "+v"(P[i])); STAMP(wsl, 12); })
        if (!isw || dest != rank) size16 = encode_chunk<false>(P, bld, exs, lutc, ws, isw ? OFF_EW + (dest * 256u + id) * 8192u : OFF_EX + id * 8192u, lane);
        if (!isw || dest == rank) {
          u32 o;
          if (isw) o = OFF_BT + (id >> 4) * (1u << 20) + (rank * 16u + (id & 15u)) * 8192u;
          else o = OFF_AT + (id >> 5) * (2u << 20) + (rank * 8u + ((id >> 2) & 7u)) * 32768u + (id & 3u) * 4096u;
#pragma unroll
          for (int v = 0; v < 8; ++v) {
            const u32x4 q = {P[4 * v], P[4 * v + 1], P[4 * v + 2], P[4 * v + 3]};
            __builtin_amdgcn_raw_buffer_store_b128(q, ws, lane * 16u + (isw ? v * 1024u : (v & 3) * 1024u + (v >> 2) * 16384u), o, 16);
          }
        }
        DBG(if (t == gw) STAMP(wsl, 13);)
        WAIT_VM(0);
        DBG(if (t == gw) STAMP(wsl, 14);)
        if (!isw || dest != rank) {
          const u32 e = isw ? entry_w(id) : entry_x(id);
          if (lane < (u32)NR && lane != rank && (!isw || lane == dest))
            __hip_atomic_store((GAS u32*)((char*)prm.ws[lane] + flag_off(rank, e)), (call << 10) | size16, __ATOMIC_RELAXED, __HIP_MEMORY_SCOPE_SYSTEM);
        }
        if ((!isw || dest == rank) && lane == 0)
          __hip_atomic_fetch_add((GAS u32*)(wsl + OFF_CNT + (isw ? 16u + (id >> 4) : (id >> 5)) * 128u), 1u, __ATOMIC_RELAXED, __HIP_MEMORY_SCOPE_AGENT);
      }
    }
    STAMP(wsl, 1);
    const rsrc_t ro = make_rsrc(prm.out, (u32)MM * NN * 2u);
    const u32 wm = wid & 3u, wn = wid >> 2;
    for (u32 round = 0; round < 2u; ++round) {
      const u32 seq = round * 8u + xcd;
      u32 si, sj;
      st_coord(seq, si, sj);
      const u32 rb = si * 4u + (pos >> 2), cb = sj * 4u + (pos & 3u);
      f32x4 acc[4][4];
#pragma unroll
      for (int a = 0; a < 4; ++a)
#pragma unroll
        for (int c = 0; c < 4; ++c) acc[a][c] = (f32x4){0.f, 0.f, 0.f, 0.f};
      u32 dead = 0;
      for (u32 seg = 0; seg < 2u; ++seg) {
        if (wid == 0) {
          const GAS u32* ca = (const GAS u32*)(wsl + OFF_CNT + (seg * 32u + rb) * 128u);
          const GAS u32* cbp = (const GAS u32*)(wsl + OFF_CNT + (seg * 32u + 16u + cb) * 128u);
          const u32 mul = seg ? call * 7u : call;
          const u64 t0 = now();
          u32 dd = 0;
          for (;;) {
            const u32 va = __hip_atomic_load(ca, __ATOMIC_RELAXED, __HIP_MEMORY_SCOPE_AGENT);
            const u32 vb = __hip_atomic_load(cbp, __ATOMIC_RELAXED, __HIP_MEMORY_SCOPE_AGENT);
            const u32 vt = __hip_atomic_load(tmo, __ATOMIC_RELAXED, __HIP_MEMORY_SCOPE_AGENT);
            if (rfl(vt) == call) { dd = 1; break; }
            if (rfl(va) >= mul * 32u && rfl(vb) >= mul * 16u) break;
            if (now() - t0 > DEADLINE) {
              if (lane == 0) __hip_atomic_store(tmo, call, __ATOMIC_RELAXED, __HIP_MEMORY_SCOPE_AGENT);
              dd = 1;
              break;
            }
            __builtin_amdgcn_s_sleep(2);
          }
          __builtin_amdgcn_fence(__ATOMIC_ACQUIRE, "agent");
          if (dd) lds_st(smem, L_DEAD, 1u);
        }
        asm volatile("s_waitcnt vmcnt(0) lgkmcnt(0)" ::: "memory");
        __builtin_amdgcn_s_barrier();
        CFENCE();
        dead = lds_ld(smem, L_DEAD);
        if (seg) STAMP(wsl, 2 + round * 3);

        const u32 nks = seg ? 56u : 8u;
        u32 kn = rank * 8u + seg * 8u;
        const u32 ga = OFF_AT + rb * (2u << 20) + wid * 4096u, gb = OFF_BT + cb * (1u << 20) + wid * 2048u;
        const u32 la = (u32)(size_t)smem + wid * 4096u, lb = (u32)(size_t)smem + 32768u + wid * 2048u;
        const u32 vo = lane * 16u;
#define SB() __builtin_amdgcn_sched_barrier(0)
#define STG_A(I) dma16<(I) * 1024>(ws, la + sbuf, vo, ga + (kn & 63u) * 32768u)
#define STG_B(I) dma16<(I) * 1024>(ws, lb + sbuf, vo, gb + (kn & 63u) * 16384u)
        asm volatile("s_nop 4");
#pragma unroll
        for (int i = 0; i < 3; ++i) {
          const u32 sbuf = i * L_STAGE;
          STG_B(0); STG_B(1); STG_A(0); STG_A(1); STG_A(2); STG_A(3);
          ++kn;
        }
        const LAS u8* ra = smem + wm * 4096u + lane * 16u;
        const LAS u8* rbp = smem + 32768u + wn * 4096u + lane * 16u;
        bf16x8 a0[4], b0[4], a1[4], b1[4];
        WAIT_VM(12);
        __builtin_amdgcn_s_barrier();
        CFENCE();
#pragma unroll
        for (int i = 0; i < 4; ++i) {
          b0[i] = *(const LAS bf16x8*)(rbp + i * 1024);
          a0[i] = *(const LAS bf16x8*)(ra + i * 1024);
        }
        u32 sbuf = 0;
#define MM_(FA, FB, A, C) acc[A][C] = __builtin_amdgcn_mfma_f32_16x16x32_bf16(FB[C], FA[A], acc[A][C], 0, 0, 0)
#define RD(DST, PTR) DST = *(const LAS bf16x8*)(PTR)
#define KSTEP(MODE) do { \
          const LAS u8* pa = ra + sbuf; const LAS u8* pb = rbp + sbuf; \
          SB(); MM_(a0, b0, 0, 0); SB(); RD(b1[0], pb + 8192); SB(); MM_(a0, b0, 0, 1); SB(); RD(a1[0], pa + 16384); \
          SB(); MM_(a0, b0, 0, 2); SB(); RD(b1[1], pb + 9216); SB(); MM_(a0, b0, 0, 3); SB(); RD(b1[2], pb + 10240); \
          SB(); MM_(a0, b0, 1, 0); SB(); RD(b1[3], pb + 11264); SB(); MM_(a0, b0, 1, 1); SB(); RD(a1[1], pa + 17408); \
          SB(); MM_(a0, b0, 1, 2); SB(); RD(a1[2], pa + 18432); SB(); MM_(a0, b0, 1, 3); SB(); RD(a1[3], pa + 19456); \
          SB(); MM_(a0, b0, 2, 0); MM_(a0, b0, 2, 1); MM_(a0, b0, 2, 2); MM_(a0, b0, 2, 3); \
          MM_(a0, b0, 3, 0); MM_(a0, b0, 3, 1); MM_(a0, b0, 3, 2); MM_(a0, b0, 3, 3); SB(); \
          const u32 nbuf = sbuf == 2u * L_STAGE ? 0u : sbuf + L_STAGE; \
          if (MODE < 3) { \
            if (MODE < 2) asm volatile("s_waitcnt vmcnt(6) lgkmcnt(0)" ::: "memory"); \
            else asm volatile("s_waitcnt vmcnt(0) lgkmcnt(0)" ::: "memory"); \
            __builtin_amdgcn_s_barrier(); CFENCE(); \
            pa = ra + nbuf; pb = rbp + nbuf; \
          } \
          SB(); MM_(a1, b1, 0, 0); SB(); if (MODE < 3) RD(b0[0], pb); SB(); MM_(a1, b1, 0, 1); SB(); if (MODE < 3) RD(a0[0], pa); \
          SB(); MM_(a1, b1, 0, 2); SB(); if (MODE < 3) RD(b0[1], pb + 1024); SB(); MM_(a1, b1, 0, 3); SB(); if (MODE < 3) RD(b0[2], pb + 2048); \
          SB(); MM_(a1, b1, 1, 0); SB(); if (MODE < 3) RD(b0[3], pb + 3072); SB(); MM_(a1, b1, 1, 1); SB(); if (MODE < 3) RD(a0[1], pa + 1024); \
          SB(); MM_(a1, b1, 1, 2); SB(); if (MODE < 3) RD(a0[2], pa + 2048); SB(); MM_(a1, b1, 1, 3); SB(); if (MODE < 3) RD(a0[3], pa + 3072); \
          SB(); MM_(a1, b1, 2, 0); SB(); if (MODE == 0) STG_B(0); SB(); MM_(a1, b1, 2, 1); SB(); if (MODE == 0) STG_B(1); \
          SB(); MM_(a1, b1, 2, 2); SB(); if (MODE == 0) STG_A(0); SB(); MM_(a1, b1, 2, 3); SB(); if (MODE == 0) STG_A(1); \
          SB(); MM_(a1, b1, 3, 0); SB(); if (MODE == 0) STG_A(2); SB(); MM_(a1, b1, 3, 1); SB(); if (MODE == 0) STG_A(3); \
          SB(); MM_(a1, b1, 3, 2); MM_(a1, b1, 3, 3); SB(); \
          ++kn; sbuf = nbuf; \
        } while (0)
        for (u32 ks = 3; ks < nks; ++ks) KSTEP(0);
        KSTEP(1);
        KSTEP(2);
        KSTEP(3);
      }
      STAMP(wsl, 3 + round * 3);
      u32 P[32];
#pragma unroll
      for (int a = 0; a < 4; ++a)
#pragma unroll
        for (int c = 0; c < 4; ++c)
#pragma unroll
          for (int j = 0; j < 2; ++j) {
            const f32x2 v = {__builtin_elementwise_maximum(acc[a][c][2 * j], 0.f), __builtin_elementwise_maximum(acc[a][c][2 * j + 1], 0.f)};
            const u32 pk = __builtin_bit_cast(u32, __builtin_convertvector(v, bf16x2));
            P[a * 8 + c * 2 + j] = dead ? 0x7fc07fc0u : pk;
          }
      WAIT_LGKM0();
      __builtin_amdgcn_s_barrier();
      CFENCE();
      const u32 oc = seq * 128u + pos * 8u + wid;
      DBG(const u64 te0 = now();)
      const u32 size16 = encode_chunk<true>(P, bld, exs, lutc, ws, OFF_EO + oc * 8192u, lane);
      DBG(const u64 te1 = now();)
      WAIT_VM(0);
      DBG(STAMPV(wsl, 10, te1 - te0); STAMPV(wsl, 11, now() - te1);)
      if (lane < (u32)NR && lane != rank)
        __hip_atomic_store((GAS u32*)((char*)prm.ws[lane] + flag_off(rank, 768u + oc)), (call << 10) | size16, __ATOMIC_RELAXED, __HIP_MEMORY_SCOPE_SYSTEM);
      {
        const u32 voff = (lane & 15u) * (NN * 2u) + (lane >> 4) * 16u;
        const u32 soff = (rb * 256u + wm * 64u) * (NN * 2u) + (rank * NSH + cb * 128u + wn * 64u) * 2u;
#pragma unroll
        for (int v = 0; v < 8; ++v) {
          const u32x4 q = {P[4 * v], P[4 * v + 1], P[4 * v + 2], P[4 * v + 3]};
          __builtin_amdgcn_raw_buffer_store_b128(q, ro, voff + (v & 1) * 64u, soff + (v >> 1) * 16u * (NN * 2u), OAUX);
        }
      }
      STAMP(wsl, 4 + round * 3);
    }
  } else {
    __syncthreads();
    const u32 call = lds_ld(smem, L_CALL);
    const u32 cid = (pos - 16u) * 8u + xcd;
    if (cid >= 7u * NL) return;
    const u32 src = (rank + 1u + cid % 7u) & 7u;
    if (wid == 1) {
      GAS u32* tk = (GAS u32*)(wsl + OFF_TK + src * 128u);
      const u32 tbase = (call - 1u) * (u32)(NTK + NL);
      u32 tnext = 0;
      if (lane == 0) tnext = __hip_atomic_fetch_add(tk, 1u, __ATOMIC_RELAXED, __HIP_MEMORY_SCOPE_AGENT);
      u32 n = 0, dead = 0, nprev[LOOK + 1];
#pragma unroll
      for (int i = 0; i <= LOOK; ++i) nprev[i] = 0;
      DBG(u64 tw[4] = {};)
      for (;;) {
        const u32 t = rfl(tnext) - tbase;
        if (t >= (u32)NTK) break;
        const u64 t0 = now();
        while (lds_ld(smem, L_LPOS) < nprev[LOOK] && now() - t0 < 4 * DEADLINE) __builtin_amdgcn_s_sleep(1);
        if (lane == 0) tnext = __hip_atomic_fetch_add(tk, 1u, __ATOMIC_RELAXED, __HIP_MEMORY_SCOPE_AGENT);
        const u32 e = t < 768u ? t : 2u * t - 768u, cnt = t < 768u ? 1u : 2u;
        const GAS u32* fp = (const GAS u32*)(wsl + flag_off(src, e)) + (lane & (cnt - 1u));
        u32 posted = 0;
        DBG(const u64 t1 = now(); u32 polls = 0;)
#pragma unroll
        for (int i = LOOK; i > 0; --i) nprev[i] = nprev[i - 1];
        nprev[0] = n + 1u;
        for (;;) {
          const u32 f = dead ? 0u : __hip_atomic_load(fp, __ATOMIC_RELAXED, __HIP_MEMORY_SCOPE_SYSTEM);
          while (posted < cnt) {
            const u32 fi = __builtin_amdgcn_readlane(f, posted);
            if (!dead && (fi >> 10) != call) break;
            u32 size16 = fi & 1023u;
            size16 = size16 > 512u ? 512u : size16;
            lds_st(smem, L_REQ + (n % NQ) * 4u, ((n + 1u) & 1023u) | (size16 << 10) | ((e + posted) << 20));
            ++n;
            ++posted;
          }
          if (posted == cnt) break;
          DBG(++polls;)
          if (timed_out(tmo, call) || now() - t0 > DEADLINE) {
            if (lane == 0) __hip_atomic_store(tmo, call, __ATOMIC_RELAXED, __HIP_MEMORY_SCOPE_AGENT);
            dead = 1;
          }
          __builtin_amdgcn_s_sleep(1);
        }
        DBG(if (polls && n > 1) tw[t < 768u ? 0 : 2] += now() - t1;)
      }
      lds_st(smem, L_NEND, n + 1u);
      DBG(for (int i = 0; i < 4; ++i) STAMPV(wsl, 10 + i, tw[i]);)
    } else if (wid == 0) {
      const rsrc_t rp = make_rsrc(prm.ws[src], WS_NEED);
      const u32 lbase = (u32)(size_t)smem;
      u32 issued = 0, pub = 0, slot = 0;
      DBG(u64 tw[4] = {}; u32 seen = 0;)
      for (u32 n = 0;; ++n) {
        const u32 need_free = n >= (u32)NSLOT ? n - (NSLOT - 1u) : 0u;
        u32 rq = lds_ld(smem, L_REQ + (n % NQ) * 4u);
        bool stop = false;
        if ((rq & 1023u) != ((n + 1u) & 1023u) || lds_ld(smem, L_FREE + slot * 4u) < need_free) {
          const u64 t0 = now();
          DBG(const bool wf = (rq & 1023u) != ((n + 1u) & 1023u);)
          for (;;) {
            if (issued > pub) {
              switch (issued - pub) {
                case 1: WAIT_VM(0); break;
                case 2: WAIT_VM(1); break;
                case 3: WAIT_VM(2); break;
                case 4: WAIT_VM(3); break;
                case 5: WAIT_VM(4); break;
                case 6: WAIT_VM(5); break;
                case 7: WAIT_VM(6); break;
                case 8: WAIT_VM(7); break;
                case 9: WAIT_VM(8); break;
                case 10: WAIT_VM(9); break;
                default: WAIT_VM(0); break;
              }
              ++pub;
              lds_st(smem, L_LANDED, pub);
            } else {
              const u32 ne = lds_ld(smem, L_NEND);
              if ((ne != 0u && n + 1u >= ne) || now() - t0 > 4 * DEADLINE) { stop = true; break; }
              __builtin_amdgcn_s_sleep(1);
            }
            rq = lds_ld(smem, L_REQ + (n % NQ) * 4u);
            if ((rq & 1023u) == ((n + 1u) & 1023u) && lds_ld(smem, L_FREE + slot * 4u) >= need_free) break;
          }
          DBG(if (n > 0 && !stop) { u64 dt = now() - t0; if ((rq >> 20) < 768u) { if (wf) tw[0] += dt; else tw[1] += dt; } else { if (wf) tw[2] += dt; else tw[3] += dt; } })
        }
        if (stop) break;
        lds_st(smem, L_LPOS, n + 1u);
        const u32 e = rq >> 20, size16 = (rq >> 10) & 1023u;
        u32 kind, id;
        entry_inv(e, kind, id);
        const u32 go = kind == 0 ? OFF_EW + (rank * 256u + id) * 8192u : kind == 1 ? OFF_EX + id * 8192u : OFF_EO + id * 8192u;
        const u32 np = (size16 + 63u) >> 6;
        lds_st(smem, L_LEND + slot * 4u, ((n + 1u) & 0xffffu) | ((issued + np) << 16));
        for (u32 pc = 0; pc < np; ++pc) {
          const u32 o16 = pc * 64u + lane;
          dma16_sys(rp, lbase + slot * 8192u + pc * 1024u, o16 < size16 ? o16 * 16u : 0x80000000u, go);
          ++issued;
          if (issued >= pub + WINDOW) {
            asm volatile("s_waitcnt vmcnt(" STR(WINDOW) " - 1)" ::: "memory");
            pub = issued - (WINDOW - 1);
            lds_st(smem, L_LANDED, pub);
          }
        }
        slot = slot == NSLOT - 1u ? 0u : slot + 1u;
      }
      STAMP(wsl, 3);
      DBG(for (int i = 0; i < 4; ++i) STAMPV(wsl, 10 + i, tw[i]);)
    } else {
      const rsrc_t ro = make_rsrc(prm.out, (u32)MM * NN * 2u);
      LAS u8* hs = smem + L_HS + (wid - 2u) * L_HS_STRIDE;
      const LAS u32x2* lutx = (const LAS u32x2*)(smem + L_LUT);
      DBG(u64 td[4] = {};)
      for (u32 n = wid - 2u;; n += NDEC) {
        const u32 slot = n % NSLOT;
        bool stop = false;
        {
          const u64 t0 = now();
          for (;;) {
            const u32 le = lds_ld(smem, L_LEND + slot * 4u);
            if ((le & 0xffffu) == n + 1u && lds_ld(smem, L_LANDED) >= (le >> 16)) break;
            const u32 ne = lds_ld(smem, L_NEND);
            if (ne != 0u && n + 1u >= ne) { stop = true; break; }
            if (now() - t0 > 5 * DEADLINE) {
              if (lane == 0) {
                __hip_atomic_store(tmo, call, __ATOMIC_RELAXED, __HIP_MEMORY_SCOPE_AGENT);
                ((GAS unsigned short*)prm.out)[src * NSH] = 0x7fc0;
              }
              stop = true;
              break;
            }
            __builtin_amdgcn_s_sleep(1);
          }
        }
        if (stop) break;
        CFENCE();
        DBG(const u64 td0 = now(); if (n == wid - 2u) STAMP(wsl, 1);)
        const u32 rq = lds_ld(smem, L_REQ + (n % NQ) * 4u);
        const u32 size16 = (rq >> 10) & 1023u;
        const bool dead = size16 == 0u;
        u32 kind, id;
        entry_inv(rq >> 20, kind, id);
        u32 P[32];
        if (dead) {
#pragma unroll
          for (int i = 0; i < 32; ++i) P[i] = 0x7fc07fc0u;
        } else if (kind == 2) {
          decode_chunk<true>(smem + slot * 8192u, hs, lutx, size16, lane, P);
        } else {
          decode_chunk<false>(smem + slot * 8192u, hs, lutx, size16, lane, P);
        }
#pragma unroll
        for (int i = 0; i < 32; ++i) asm volatile("" : "+v"(P[i]));
        WAIT_LGKM0();
        lds_st(smem, L_FREE + slot * 4u, n + 1u);
        DBG(td[kind == 2 ? 2 : 0] += now() - td0;)
        if (kind == 2) {
          u32 si, sj;
          st_coord(id >> 7, si, sj);
          const u32 t = (id >> 3) & 15u, w = id & 7u;
          const u32 voff = (lane & 15u) * (NN * 2u) + (lane >> 4) * 16u;
          const u32 soff = ((si * 4u + (t >> 2)) * 256u + (w & 3u) * 64u) * (NN * 2u) + (src * NSH + (sj * 4u + (t & 3u)) * 128u + (w >> 2) * 64u) * 2u;
#pragma unroll
          for (int v = 0; v < 8; ++v) {
            const u32x4 q = {P[4 * v], P[4 * v + 1], P[4 * v + 2], P[4 * v + 3]};
            __builtin_amdgcn_raw_buffer_store_b128(q, ro, voff + (v & 1) * 64u, soff + (v >> 1) * 16u * (NN * 2u), OAUX);
          }
        } else {
          u32 o;
          if (kind == 0) o = OFF_BT + (id >> 4) * (1u << 20) + (src * 16u + (id & 15u)) * 8192u;
          else o = OFF_AT + (id >> 5) * (2u << 20) + (src * 8u + ((id >> 2) & 7u)) * 32768u + (id & 3u) * 4096u;
          const u32 hstride = kind == 0 ? 4096u : 16384u;
#pragma unroll
          for (int v = 0; v < 8; ++v) {
            const u32x4 q = {P[4 * v], P[4 * v + 1], P[4 * v + 2], P[4 * v + 3]};
            __builtin_amdgcn_raw_buffer_store_b128(q, ws, lane * 16u + (v & 3) * 1024u, o + (v >> 2) * hstride, 16);
          }
          WAIT_VM(0);
          if (lane == 0)
            __hip_atomic_fetch_add((GAS u32*)(wsl + OFF_CNT + (32u + (kind == 0 ? 16u + (id >> 4) : (id >> 5))) * 128u), 1u, __ATOMIC_RELAXED, __HIP_MEMORY_SCOPE_AGENT);
        }
        DBG(td[kind == 2 ? 3 : 1] += now() - td0; if (n == wid - 2u) STAMP(wsl, 2);)
      }
      STAMP(wsl, 3);
      DBG(for (int i = 0; i < 4; ++i) STAMPV(wsl, 10 + i, td[i]);)
    }
  }
}

extern "C" void kernel_launch(int rank, int nranks, ncclComm_t comm, hipStream_t stream, void* const* d_in, const int* in_sizes,
                              int n_in, void* d_out, int out_size, const int* in_dtypes, int out_dtype, void* const* d_out_all,
                              void* const* d_ws_all, size_t ws_bytes) {
  (void)comm; (void)in_sizes; (void)n_in; (void)out_size; (void)in_dtypes; (void)out_dtype; (void)d_out_all;
  if (nranks != NR || ws_bytes < WS_NEED || (unsigned)rank >= (unsigned)NR) {
    fprintf(stderr, "unsupported configuration\n");
    return;
  }
  Params p;
  p.x = d_in[0];
  p.w = d_in[1];
  p.out = d_out;
  for (int i = 0; i < NR; ++i) p.ws[i] = d_ws_all[i];
  p.rank = rank;
  p.pad = 0;
  gemm_ar_kernel<<<NWG, NTHR, 0, stream>>>(p);
}
```

```cpp
#include <hip/hip_runtime.h>
#include <rccl/rccl.h>
#include <cstdio>

typedef unsigned u32;
typedef unsigned long long u64;
typedef unsigned char u8;
typedef u32 u32x2 __attribute__((ext_vector_type(2)));
typedef u32 u32x4 __attribute__((ext_vector_type(4)));
typedef float f32x2 __attribute__((ext_vector_type(2)));
typedef float f32x4 __attribute__((ext_vector_type(4)));
typedef short s16x2 __attribute__((ext_vector_type(2)));
typedef short s16x4 __attribute__((ext_vector_type(4)));
typedef __bf16 bf16x2 __attribute__((ext_vector_type(2)));
typedef __bf16 bf16x8 __attribute__((ext_vector_type(8)));
#define LAS __attribute__((address_space(3)))
#define GAS __attribute__((address_space(1)))
typedef __amdgpu_buffer_rsrc_t rsrc_t;

namespace {

constexpr int NR = 8;
constexpr int MM = 4096, NN = 16384, KSH = 512, NSH = 2048;
constexpr int NWG = 256, NTHR = 512;
#define NLOAD 18
constexpr int NL = NLOAD;
constexpr u32 NFU = 32;
constexpr int NGRP = 16 + 96 + 256;
#define TAILT 16
#define TAILC 2
#define NPUSHG 4
constexpr u32 NP = NPUSHG;
constexpr u32 NTI = 4 * (96 - NP);
constexpr int NTK = NTI + 2 * (256 - TAILT) + (8 / TAILC) * TAILT;
constexpr u32 NTASK = 312 - 7 * NP;
#define SDELAY 24
constexpr int NREG = 7, NDEC = 6, NQ = 32, NJ = 128;
#define WINDOW 3
#define THRESH 5
#define OAUX 16
#define PSLOW 127
#define PFAST 16
#define DSLEEP 3
#define PWIN 1
#define FSLEEP 8
#define FIRSTSLEEP 80
constexpr u64 DEADLINE = 30000000ull;

constexpr u32 OFF_CALL = 0;
constexpr u32 OFF_TMO = 16384;
constexpr u32 OFF_TK = 17408;
constexpr u32 OFF_CNT = 20480;
constexpr u32 OFF_FLAG = 32768;
constexpr u32 OFF_DBG = 512u << 10;
constexpr u32 OFF_EX = 1u << 20;
constexpr u32 OFF_EW = 5u << 20;
constexpr u32 OFF_EO = 21u << 20;
constexpr u32 OFF_AT = 37u << 20;
constexpr u32 OFF_BT = 69u << 20;
constexpr u32 WS_NEED = 85u << 20;

constexpr int L_STAGE = 49152;
constexpr int L_HS = NREG * 16384 + 512;
constexpr int L_HS_STRIDE = 4160;
constexpr int L_MISC = 3 * L_STAGE;
constexpr int L_LUT = L_MISC;
constexpr int L_Z0 = L_MISC + 128;
constexpr int L_REQ = L_Z0;
constexpr int L_JOB = L_REQ + 8 * NQ;
constexpr int L_FREE = L_JOB + 8 * NJ;
constexpr int L_LANDED = L_FREE + 32;
constexpr int L_REND = L_LANDED + 4;
constexpr int L_JEND = L_LANDED + 8;
constexpr int L_DEAD = L_LANDED + 12;
constexpr int L_CALL = L_LANDED + 16;
constexpr int L_SZ = L_LANDED + 32;
constexpr int L_ARR = L_SZ + 192;
constexpr int L_Z1 = L_ARR + 32;
constexpr int L_EXC = L_MISC + 2048;
constexpr int L_TOTAL = L_EXC + 8 * 256;
static_assert(L_HS + NDEC * L_HS_STRIDE <= L_MISC, "lds");
static_assert(L_Z1 <= L_EXC && (L_Z1 - L_Z0) / 4 <= NTHR, "lds");
static_assert(L_TOTAL <= 163840, "lds");

constexpr u32 LO_OFF = 512;
constexpr u32 UB_IN = LO_OFF + 4096;

struct Params {
  const void* x;
  const void* w;
  void* out;
  void* ws[NR];
  int rank;
  int pad;
};

#define CFENCE() asm volatile("" ::: "memory")
#define WAIT_VM(n) asm volatile("s_waitcnt vmcnt(" #n ")" ::: "memory")
#define WAIT_LGKM0() asm volatile("s_waitcnt lgkmcnt(0)" ::: "memory")
#define SB() __builtin_amdgcn_sched_barrier(0)
#define STR2(x) #x
#define STR(x) STR2(x)

__device__ __forceinline__ u32 rfl(u32 v) { return __builtin_amdgcn_readfirstlane(v); }
__device__ __forceinline__ u32 vperm(u32 hi, u32 lo, u32 sel) { return __builtin_amdgcn_perm(hi, lo, sel); }
__device__ __forceinline__ u32 bsum(u32 v, u32 acc) { return __builtin_amdgcn_sad_u8(v, 0u, acc); }
__device__ __forceinline__ u64 now() { return __builtin_amdgcn_s_memrealtime(); }
__device__ __forceinline__ u32 mbcnt(u64 m) {
  return __builtin_amdgcn_mbcnt_hi((u32)(m >> 32), __builtin_amdgcn_mbcnt_lo((u32)m, 0u));
}
__device__ __forceinline__ u32 scan_incl(u32 x) {
  x += __builtin_amdgcn_update_dpp(0u, x, 0x111, 0xf, 0xf, true);
  x += __builtin_amdgcn_update_dpp(0u, x, 0x112, 0xf, 0xf, true);
  x += __builtin_amdgcn_update_dpp(0u, x, 0x114, 0xf, 0xf, true);
  x += __builtin_amdgcn_update_dpp(0u, x, 0x118, 0xf, 0xf, true);
  x += __builtin_amdgcn_update_dpp(0u, x, 0x142, 0xa, 0xf, false);
  x += __builtin_amdgcn_update_dpp(0u, x, 0x143, 0xc, 0xf, false);
  return x;
}
__device__ __forceinline__ rsrc_t make_rsrc(const void* p, u32 bytes) {
  return __builtin_amdgcn_make_buffer_rsrc((void*)p, 0, bytes, 0x00020000);
}
#define GSC " sc0 sc1"
template <int IMM>
__device__ __forceinline__ void dma16(rsrc_t r, u32 lds, u32 voff, u32 soff) {
  asm volatile("s_mov_b32 m0, %0\n\ts_nop 0\n\tbuffer_load_dwordx4 %1, %2, %3 offen offset:%4" GSC " lds"
               :: "s"(lds), "v"(voff), "s"(r), "s"(soff), "i"(IMM) : "memory");
}
__device__ __forceinline__ void dma16_sys(rsrc_t r, u32 lds, u32 voff, u32 soff) {
  asm volatile("s_mov_b32 m0, %0\n\ts_nop 3\n\tbuffer_load_dwordx4 %1, %2, %3 offen sc0 sc1 lds"
               :: "s"(lds), "v"(voff), "s"(r), "s"(soff) : "memory");
}
template <int AUX>
__device__ __forceinline__ void st16(u32x4 v, rsrc_t r, u32 voff, u32 soff) {
  __builtin_amdgcn_raw_buffer_store_b128(v, r, voff, soff, AUX);
  SB();
  asm volatile("s_nop 1");
  SB();
}
__device__ __forceinline__ u32 lds_ld(LAS u8* lds, int off) {
  return rfl(*(volatile LAS u32*)(lds + off));
}
__device__ __forceinline__ void lds_st(LAS u8* lds, int off, u32 v) { *(volatile LAS u32*)(lds + off) = v; }

#define STAMP(ws, slot) do {} while (0)
#define STAMPV(ws, slot, v) do {} while (0)
#define DBG(x)

__device__ __forceinline__ u64 rank_table(u32 k) {
  const u64 below = (1ull << (8 * k)) - 1ull;
  return (0x0001020304050607ull & ~below & ~(0xffull << (8 * k))) | (7ull << (8 * k)) | (0x0000010203040506ull & below);
}
template <int U>
__device__ __forceinline__ u32 unit_offset(LAS u8* szw, u32 wid) {
  u32 off = 0;
  for (u32 j = wid & ~(U - 1u); j < wid; ++j) {
    u32 v = lds_ld(szw, j * 4u);
    for (int spin = 0; v == 0u && spin < (1 << 22); ++spin) v = lds_ld(szw, j * 4u);
    off += v;
  }
  return off;
}
template <bool OUT>
__device__ __forceinline__ void encode_chunk(const u32 (&P)[32], LAS u8* bld, LAS u32* exs, const LAS u32* lutc, rsrc_t ws,
                                             u32 gbase, LAS u8* szw, u32 wid, u32 lane, bool force_raw) {
  constexpr int U = OUT ? 4 : 2;
  const u32 ub = gbase + (wid & ~(U - 1u)) * 8192u;
  u32 lo[16], len[16], nzm[16];
  u32 pl[2] = {0u, 0u};
  u32 size16 = 512u, nexc = 0, nw = 0, ubase = 0, pbase = 0, qbase = 0, trailer = 0;
  if (!force_raw) {
    u32 hi[16];
    u32 sum = 0, sgn = 0;
#pragma unroll
    for (int g = 0; g < 16; ++g) {
      lo[g] = vperm(P[2 * g + 1], P[2 * g], 0x06040200u);
      hi[g] = vperm(P[2 * g + 1], P[2 * g], 0x07050301u);
      if (OUT) {
        const s16x2 z = {0, 0};
        const u32 n0 = __builtin_bit_cast(u32, z - __builtin_bit_cast(s16x2, P[2 * g]));
        const u32 n1 = __builtin_bit_cast(u32, z - __builtin_bit_cast(s16x2, P[2 * g + 1]));
        nzm[g] = vperm(n1, n0, 0x0b0a0908u);
        pl[g >> 3] |= bsum(nzm[g] & 0x08040201u, 0u) << (4 * (g & 7));
        sgn |= hi[g];
      } else {
        pl[g >> 3] |= (hi[g] >> (7 - (g & 7))) & (0x01010101u << (g & 7));
        hi[g] &= 0x7f7f7f7fu;
        nzm[g] = 0xffffffffu;
      }
      if (g & 1) sum = bsum(hi[g] + hi[g - 1], sum);
    }
    const bool signed_out = OUT && __builtin_amdgcn_ballot_w64((sgn & 0x80808080u) != 0u) != 0ull;
    const u32 cnt = OUT ? (u32)(__builtin_popcount(pl[0]) + __builtin_popcount(pl[1])) : 64u;
    const u32 sc0 = scan_incl((sum & 0x1fffu) | (cnt << 19));
    const u32 tot0 = __builtin_amdgcn_readlane(sc0, 63);
    const u32 nnz = tot0 >> 19;
    pbase = (sc0 >> 19) - cnt;
    const float mean = (float)(tot0 & 0x7ffffu) * __builtin_amdgcn_rcpf((float)(nnz ? nnz : 1u));
    int b = (int)(mean + 0.78f);
    const float fr = mean - (float)(b - 1);
    const u32 k = rfl(1u + (fr < 0.32f) + (fr < 0.41f) + (fr < 0.60f) + (fr < 0.87f));
    b = b < 6 ? 6 : b > 126 ? 126 : b;
    const u32 base = rfl((u32)(b - 6));
    const u32 cneg = 0x80808080u - base * 0x01010101u;
    const u64 inv = (0x0001020304050607ull + (0x0101010101010101ull >> (8 * (k + 1)))) | ((u64)(k + 1) << 56);
    u32 inv_lo = (u32)inv;
    const u32 inv_hi = (u32)(inv >> 32);
    asm volatile("" : "+v"(inv_lo));
    u32 anybad = 0;
#pragma unroll
    for (int g = 0; g < 16; ++g) {
      const u32 e4 = (hi[g] + cneg) ^ 0x80808080u;
      anybad |= e4 & nzm[g];
      len[g] = vperm(inv_hi, inv_lo, e4) & nzm[g];
      if (OUT) lo[g] = vperm(0u, lo[g], lutc[(pl[g >> 3] >> (4 * (g & 7))) & 15u]);
    }
    if (__builtin_amdgcn_ballot_w64((anybad & 0xf8f8f8f8u) != 0u) != 0ull) {
      u32 bp[2] = {0u, 0u};
#pragma unroll
      for (int g = 0; g < 16; ++g) {
        const u32 bad = ((hi[g] + cneg) ^ 0x80808080u) & 0xf8f8f8f8u & nzm[g];
        const u32 t = ((bad >> 1) + 0x7c7c7c7cu) & 0x80808080u;
        const u32 fm = (t << 1) - (t >> 7);
        len[g] = (len[g] & ~fm) | (t >> 7);
        bp[g >> 3] |= bsum(fm & 0x08040201u, 0u) << (4 * (g & 7));
      }
#pragma unroll
      for (int j = 0; j < 4; ++j) {
        const u32x4 t = {hi[4 * j], hi[4 * j + 1], hi[4 * j + 2], hi[4 * j + 3]};
        *(LAS u32x4*)(bld + lane * 64u + j * 16) = t;
      }
      CFENCE();
      u64 bm = ((u64)bp[1] << 32) | bp[0];
      const u64 plm = ((u64)pl[1] << 32) | pl[0];
      for (;;) {
        const u64 m = __builtin_amdgcn_ballot_w64(bm != 0ull);
        if (m == 0ull || nexc > 64u) break;
        if (bm != 0ull) {
          const u32 j = __builtin_ctzll(bm);
          bm &= bm - 1ull;
          const u32 slot = nexc + mbcnt(m);
          const u32 idx = pbase + (OUT ? (u32)__builtin_popcountll(plm & ((1ull << j) - 1ull)) : j);
          if (slot < 64u) exs[slot] = (idx << 8) | bld[lane * 64u + j];
        }
        nexc += __builtin_popcountll(m);
      }
      CFENCE();
    }
    u32 acc = 0;
#pragma unroll
    for (int g = 0; g < 16; ++g) acc += len[g];
    const u32 bits = bsum(acc, 0u);
    const u32 sc1 = scan_incl(bits);
    qbase = sc1 - bits;
    nw = (__builtin_amdgcn_readlane(sc1, 63) + 31u) >> 5;
    ubase = OUT ? LO_OFF + ((nnz + 3u) & ~3u) : UB_IN;
    size16 = (ubase + 4u * nw + 4u * nexc + 4u + 15u) >> 4;
    if (size16 >= 512u || nexc > 64u || signed_out) size16 = 512u;
    trailer = base | (k << 7) | (nw << 10) | (nexc << 21);
  }
  lds_st(szw, wid * 4u, size16);
  if (size16 == 512u) {
    const u32 go = ub + unit_offset<U>(szw, wid) * 16u;
#pragma unroll
    for (int v = 0; v < 8; ++v) {
      const u32x4 q = {P[4 * v], P[4 * v + 1], P[4 * v + 2], P[4 * v + 3]};
      st16<17>(q, ws, lane * 16u + v * 1024u, go);
    }
    return;
  }
  {
    const u32x4 z = {0u, 0u, 0u, 0u};
    for (u32 o = (OUT ? LO_OFF : UB_IN); o < size16 * 16u; o += 1024u) *(LAS u32x4*)(bld + o + lane * 16u) = z;
  }
  CFENCE();
  u32 q = qbase + ((u32)(size_t)bld + ubase) * 8u;
  u32 p = pbase + (u32)(size_t)bld + LO_OFF;
  u32 ones = 0x01010101u;
  asm volatile("" : "+v"(ones));
#pragma unroll
  for (int g0 = 0; g0 < 16; g0 += 4) {
    u64 v[4], u[4];
    u32 da[4], ea[4];
#pragma unroll
    for (int i = 0; i < 4; ++i) {
      const int g = g0 + i;
      u32 t = len[g] + (len[g] << 8);
      asm volatile("" : "+v"(t));
      t += t << 16;
      const u32 tb = t + 0x1f1f1f1fu;
      const u32 one = OUT ? (nzm[g] & 0x01010101u) : ones;
      u32 b4, b5, b6;
      asm("v_lshlrev_b32_sdwa %0, %3, %4 dst_sel:DWORD dst_unused:UNUSED_PAD src0_sel:BYTE_0 src1_sel:BYTE_0\n\t"
          "v_lshlrev_b32_sdwa %1, %3, %4 dst_sel:DWORD dst_unused:UNUSED_PAD src0_sel:BYTE_1 src1_sel:BYTE_1\n\t"
          "v_lshlrev_b32_sdwa %2, %3, %4 dst_sel:DWORD dst_unused:UNUSED_PAD src0_sel:BYTE_2 src1_sel:BYTE_2\n\t"
          "v_or3_b32 %0, %0, %1, %2\n\t"
          "v_lshlrev_b32_sdwa %1, %3, %4 dst_sel:DWORD dst_unused:UNUSED_PAD src0_sel:BYTE_3 src1_sel:BYTE_3\n\t"
          "v_or_b32 %0, %0, %1"
          : "=&v"(b4), "=&v"(b5), "=&v"(b6) : "v"(tb), "v"(one));
      v[i] = (u64)b4 << (q & 31u);
      da[i] = (q >> 3) & ~3u;
      q += t >> 24;
      if (OUT) {
        u[i] = (u64)lo[g] << ((p & 3u) * 8u);
        ea[i] = p & ~3u;
        p += __builtin_popcount(one);
      }
    }
    SB();
#pragma unroll
    for (int i = 0; i < 4; ++i) {
      LAS u32* d = (LAS u32*)(size_t)da[i];
      __hip_atomic_fetch_or(d, (u32)v[i], __ATOMIC_RELAXED, __HIP_MEMORY_SCOPE_WAVEFRONT);
      __hip_atomic_fetch_or(d + 1, (u32)(v[i] >> 32), __ATOMIC_RELAXED, __HIP_MEMORY_SCOPE_WAVEFRONT);
      if (OUT) {
        LAS u32* e = (LAS u32*)(size_t)ea[i];
        __hip_atomic_fetch_or(e, (u32)u[i], __ATOMIC_RELAXED, __HIP_MEMORY_SCOPE_WAVEFRONT);
        __hip_atomic_fetch_or(e + 1, (u32)(u[i] >> 32), __ATOMIC_RELAXED, __HIP_MEMORY_SCOPE_WAVEFRONT);
      }
    }
    SB();
  }
  if (!OUT) {
#pragma unroll
    for (int j = 0; j < 4; ++j) {
      const u32x4 t = {lo[4 * j], lo[4 * j + 1], lo[4 * j + 2], lo[4 * j + 3]};
      *(LAS u32x4*)(bld + LO_OFF + j * 1024 + lane * 16u) = t;
    }
  }
  {
    const u32x2 t = {pl[0], pl[1]};
    *(LAS u32x2*)(bld + lane * 8u) = t;
  }
  CFENCE();
  *(LAS u32*)(bld + size16 * 16u - 4u) = trailer;
  if (lane < nexc) *(LAS u32*)(bld + ubase + 4u * nw + 4u * lane) = exs[lane];
  CFENCE();
  const u32 go = ub + unit_offset<U>(szw, wid) * 16u;
  constexpr int NB = OUT ? 4 : 6;
  for (u32 o = 0; o < size16; o += 64u * NB) {
    u32x4 t[NB];
#pragma unroll
    for (int j = 0; j < NB; ++j) t[j] = *(const LAS u32x4*)(bld + ((o * 16u + j * 1024u) & 8191u) + lane * 16u);
#pragma unroll
    for (int j = 0; j < NB; ++j) {
      const u32 o16 = o + j * 64u + lane;
      st16<17>(t[j], ws, o16 < size16 ? o16 * 16u : 0x80000000u, go);
    }
  }
  CFENCE();
}

__device__ __forceinline__ u32 parse4(u32& w, u32& q, u32 tab_hi, u32 tab_lo) {
  u32 r, p, t;
  asm("v_ffbl_b32 %3, %1\n\tv_add_u32 %4, -1, %1\n\tv_sub_u32 %0, %3, %2\n\tv_and_b32 %1, %4, %1\n\t"
      "v_ffbl_b32 %2, %1\n\tv_add_u32 %4, -1, %1\n\t"
      "v_sub_u32_sdwa %0, %2, %3 dst_sel:BYTE_1 dst_unused:UNUSED_PRESERVE src0_sel:DWORD src1_sel:DWORD\n\tv_and_b32 %1, %4, %1\n\t"
      "v_ffbl_b32 %3, %1\n\tv_add_u32 %4, -1, %1\n\t"
      "v_sub_u32_sdwa %0, %3, %2 dst_sel:BYTE_2 dst_unused:UNUSED_PRESERVE src0_sel:DWORD src1_sel:DWORD\n\tv_and_b32 %1, %4, %1\n\t"
      "v_ffbl_b32 %2, %1\n\tv_add_u32 %4, -1, %1\n\t"
      "v_sub_u32_sdwa %0, %2, %3 dst_sel:BYTE_3 dst_unused:UNUSED_PRESERVE src0_sel:DWORD src1_sel:DWORD\n\tv_and_b32 %1, %4, %1"
      : "=&v"(r), "+v"(w), "+v"(q), "=&v"(p), "=&v"(t));
  return vperm(tab_hi, tab_lo, r - 0x01010101u);
}

template <bool OUT>
__device__ __forceinline__ void decode_chunk(const LAS u8* ch, LAS u8* hs, const LAS u32x2* lutx, u32 size16, u32 lane,
                                             u32 (&P)[32]) {
  if (size16 >= 512u) {
#pragma unroll
    for (int v = 0; v < 8; ++v) {
      const u32x4 t = *(const LAS u32x4*)(ch + v * 1024 + lane * 16u);
      P[4 * v] = t.x; P[4 * v + 1] = t.y; P[4 * v + 2] = t.z; P[4 * v + 3] = t.w;
    }
    return;
  }
  const u32 tr = rfl(*(const LAS u32*)(ch + size16 * 16u - 4u));
  const u32x2 pl = *(const LAS u32x2*)(ch + lane * 8u);
  const u32 nw = (tr >> 10) & 0x7ffu, nexc = (tr >> 21) & 0x7fu;
  const u64 dec = rank_table((tr >> 7) & 7u);
  const u32 base4 = (tr & 127u) * 0x01010101u;
  u32 tab_lo = (u32)dec + base4;
  const u32 tab_hi = (u32)(dec >> 32) + base4;
  asm volatile("" : "+v"(tab_lo));
  u32 p = 0, ubase = UB_IN;
  if (OUT) {
    const u32 c = __builtin_popcount(pl.x) + __builtin_popcount(pl.y);
    const u32 incl = scan_incl(c);
    p = incl - c;
    ubase = LO_OFF + ((__builtin_amdgcn_readlane(incl, 63) + 3u) & ~3u);
  }
  const LAS u32* U = (const LAS u32*)(ch + ubase);
  u32 run = 0;
  for (u32 w0 = 0; w0 < nw; w0 += 64u) {
    const u32 i = w0 + lane;
    u32 w = U[i], prev = U[(int)i - 1];
    w = i < nw ? w : 0u;
    prev = i > 0u ? prev : 0x80000000u;
    const u32 c = __builtin_popcount(w);
    const u32 incl = scan_incl(c);
    const u32 dst = (u32)(size_t)hs + ((run + incl - c) & 4095u);
    run += __builtin_amdgcn_readlane(incl, 63);
    u32 q = ~(u32)__builtin_clz(prev | 1u);
    u32 R[8];
#pragma unroll
    for (int j = 0; j < 6; ++j) R[j] = parse4(w, q, tab_hi, tab_lo);
    R[6] = R[7] = 0u;
    if (__builtin_amdgcn_ballot_w64(w != 0u) != 0ull) {
      R[6] = parse4(w, q, tab_hi, tab_lo);
      if (__builtin_amdgcn_ballot_w64(w != 0u) != 0ull) R[7] = parse4(w, q, tab_hi, tab_lo);
    }
    const u32 wb = (dst - 1u) & ~3u, ca = c + dst - wb, sh = 0u - dst;
    u32 W[8];
    u64 m[8];
#pragma unroll
    for (int j = 0; j < 8; ++j) {
      W[j] = __builtin_amdgcn_alignbyte(j < 7 ? R[j + 1] : 0u, R[j], sh);
      m[j] = __builtin_amdgcn_ballot_w64(4u * (j + 1) < ca);
    }
    const u64 m0 = __builtin_amdgcn_ballot_w64(c != 0u);
    const u32 r1 = R[0] >> 8;
    asm volatile("s_mov_b64 exec, %9\n\tds_write_b32 %0, %1 offset:4\n\ts_mov_b64 exec, %10\n\tds_write_b32 %0, %2 offset:8\n\t"
                 "s_mov_b64 exec, %11\n\tds_write_b32 %0, %3 offset:12\n\ts_mov_b64 exec, %12\n\tds_write_b32 %0, %4 offset:16\n\t"
                 "s_mov_b64 exec, %13\n\tds_write_b32 %0, %5 offset:20\n\ts_mov_b64 exec, %14\n\tds_write_b32 %0, %6 offset:24\n\t"
                 "s_mov_b64 exec, %15\n\tds_write_b32 %0, %7 offset:28\n\ts_mov_b64 exec, %16\n\tds_write_b32 %0, %8 offset:32\n\t"
                 "s_mov_b64 exec, %17\n\tds_write_b8 %18, %19\n\tds_write_b8 %18, %20 offset:1\n\tds_write_b8_d16_hi %18, %19 offset:2\n\t"
                 "s_mov_b64 exec, -1"
                 :: "v"(wb), "v"(W[0]), "v"(W[1]), "v"(W[2]), "v"(W[3]), "v"(W[4]), "v"(W[5]), "v"(W[6]), "v"(W[7]),
                    "s"(m[0]), "s"(m[1]), "s"(m[2]), "s"(m[3]), "s"(m[4]), "s"(m[5]), "s"(m[6]), "s"(m[7]), "s"(m0),
                    "v"(dst), "v"(R[0]), "v"(r1) : "memory");
  }
  CFENCE();
  if (lane < nexc) {
    const u32 r = U[nw + lane];
    hs[(r >> 8) & 4095u] = (u8)r;
  }
  CFENCE();
  if (OUT) {
#pragma unroll
    for (int g0 = 0; g0 < 16; g0 += 8) {
      u32 pp[8];
      u32x2 sel[8], L[8], H[8];
      const LAS u8 *sa[8], *la[8], *ha[8];
#pragma unroll
      for (int i = 0; i < 8; ++i) {
        const u32 wd = g0 ? pl.y : pl.x;
        const u32 m8 = (i == 0 ? wd << 3 : wd >> (4 * i - 3)) & 0x78u;
        pp[i] = p;
        sa[i] = (const LAS u8*)lutx + m8;
        la[i] = ch + LO_OFF + (p & ~3u);
        ha[i] = hs + (p & ~3u);
        p += __builtin_popcount(m8);
      }
      SB();
#pragma unroll
      for (int i = 0; i < 8; ++i) {
        sel[i] = *(const LAS u32x2*)sa[i];
        L[i].x = ((const LAS u32*)la[i])[0]; L[i].y = ((const LAS u32*)la[i])[1];
        H[i].x = ((const LAS u32*)ha[i])[0]; H[i].y = ((const LAS u32*)ha[i])[1];
      }
      SB();
#pragma unroll
      for (int i = 0; i < 8; ++i) {
        const u32 l4 = __builtin_amdgcn_alignbyte(L[i].y, L[i].x, pp[i]);
        const u32 h4 = __builtin_amdgcn_alignbyte(H[i].y, H[i].x, pp[i]);
        P[2 * (g0 + i)] = vperm(h4, l4, sel[i].x);
        P[2 * (g0 + i) + 1] = vperm(h4, l4, sel[i].y);
      }
    }
  } else {
#pragma unroll
    for (int j = 0; j < 4; ++j) {
      const u32x4 L = *(const LAS u32x4*)(ch + LO_OFF + j * 1024 + lane * 16u);
      const u32x4 H = *(const LAS u32x4*)(hs + lane * 64u + j * 16);
#pragma unroll
      for (int i = 0; i < 4; ++i) {
        const int g = 4 * j + i;
        const u32 hh = H[i] | (((g < 8 ? pl.x : pl.y) << (7 - (g & 7))) & 0x80808080u);
        P[2 * g] = vperm(hh, L[i], 0x05010400u);
        P[2 * g + 1] = vperm(hh, L[i], 0x07030602u);
      }
    }
  }
}

__device__ __forceinline__ u32 pos_x(u32 g) { return g + (g < 8u ? 0u : g < 16u ? 8u - NP : g < 32u ? 16u - NP : 32u - NP); }
__device__ __forceinline__ u32 pos_w(u32 g) { return g - NP + (g < 8u ? 8u : g < 16u ? 16u : 32u); }
__device__ __forceinline__ u32 pos_code(u32 p) {
  if (p >= 96u) return 768u + (p - 96u) * 8u;
  if (p < 8u) return 256u + p * 8u;
  if (p < 16u - NP) return (p + NP - 8u) * 8u;
  if (p < 24u - NP) return 256u + (p + NP - 8u) * 8u;
  if (p < 32u - NP) return (p + NP - 16u) * 8u;
  if (p < 48u - NP) return 256u + (p + NP - 16u) * 8u;
  if (p < 64u - NP) return (p + NP - 32u) * 8u;
  return 256u + (p + NP - 32u) * 8u;
}
__device__ __forceinline__ void code_inv(u32 e, u32& kind, u32& id) {
  kind = e < 256u ? 0u : e < 768u ? 1u : 2u;
  id = e < 256u ? e : e < 768u ? e - 256u : e - 768u;
}
__device__ __forceinline__ void w_stage(rsrc_t rw, LAS u8* stg, u32 dest, u32 id, u32 lane) {
  const u32 cb = id >> 4, kb = id & 15u;
  const u32 voff = (lane >> 4) * (NN * 2) + (lane & 15u) * 16u;
  const u32 soff = kb * 32u * (NN * 2) + (dest * NSH + cb * 128u) * 2u;
  asm volatile("s_nop 4");
#pragma unroll
  for (int u = 0; u < 8; ++u) dma16<0>(rw, (u32)(size_t)stg + u * 1024u, voff, soff + u * 4u * (NN * 2));
}
__device__ __forceinline__ void w_fragments(const LAS u8* stg, u32 lane, u32 (&P)[32]) {
  const LAS u8* tb = stg + ((lane >> 4) * 8u + ((lane & 15u) >> 2)) * 256u + (lane & 3u) * 16u;
#pragma unroll
  for (int nf = 0; nf < 8; ++nf)
#pragma unroll
    for (int h = 0; h < 2; ++h) {
      const s16x4 r = __builtin_amdgcn_ds_read_tr16_b64_v4i16((LAS s16x4*)(tb + h * 1024 + (nf >> 1) * 64 + (nf & 1) * 8));
      const u32x2 r2 = __builtin_bit_cast(u32x2, r);
      P[4 * nf + 2 * h] = r2.x;
      P[4 * nf + 2 * h + 1] = r2.y;
    }
}
__device__ __forceinline__ void st_coord(u32 seq, u32& i, u32& j) {
  if (seq >= 8u) { i = seq >> 2; j = seq & 3u; }
  else if (seq < 4u) { i = seq >> 1; j = seq & 1u; }
  else { i = seq & 1u; j = seq >> 1; }
}
__device__ __forceinline__ u32 flag_off(u32 src, u32 g) { return OFF_FLAG + (src * NGRP + g) * 16u; }
__device__ __forceinline__ void publish(LAS u8* smem, u32 rnd, char* pws, u32 rank, u32 g, u32 lane, bool sel) {
  WAIT_VM(0);
  u32 a = 0;
  if (lane == 0) a = __hip_atomic_fetch_add((LAS u32*)(smem + L_ARR + rnd * 4u), 1u, __ATOMIC_RELAXED, __HIP_MEMORY_SCOPE_WORKGROUP);
  if (rfl(a) == (rnd ? 7u : 1u)) {
    CFENCE();
    const u32x4 s0 = *(volatile LAS u32x4*)(smem + L_SZ + rnd * 32u), s1 = *(volatile LAS u32x4*)(smem + L_SZ + rnd * 32u + 16u);
    const u32 c8 = lds_ld(smem, L_CALL) << 8;
    const u32x4 rec = {s0.x | (s0.y << 10) | (s0.z << 20) | (s0.w << 30), c8 | (s0.w >> 2),
                       s1.x | (s1.y << 10) | (s1.z << 20) | (s1.w << 30), c8 | (s1.w >> 2)};
    if (sel) {
      if (rnd) {
        asm volatile("global_store_dwordx4 %0, %1, off sc0 sc1\n\ts_nop 1" :: "v"(pws + flag_off(rank, 16u + g)), "v"(rec) : "memory");
      } else {
        const u32x2 h = {rec.x | rec.z, rec.y | rec.w};
        asm volatile("global_store_dwordx2 %0, %1, off sc0 sc1\n\ts_nop 1" :: "v"(pws + flag_off(rank, 0u) + g * 8u), "v"(h) : "memory");
      }
    }
  }
}

__device__ __forceinline__ bool timed_out(const GAS u32* tmo, u32 call) {
  return rfl(__hip_atomic_load(tmo, __ATOMIC_RELAXED, __HIP_MEMORY_SCOPE_AGENT)) == call;
}

}

__global__ __launch_bounds__(NTHR, 2) void gemm_ar_kernel(Params prm) {
  __shared__ __attribute__((aligned(16))) u8 smem_raw[L_TOTAL];
  LAS u8* smem = (LAS u8*)smem_raw;
  const u32 tid = threadIdx.x, lane = tid & 63u, wid = rfl(tid >> 6);
  const u32 rank = (u32)prm.rank;
  char* wsl = (char*)prm.ws[rank & 7u];
  const rsrc_t ws = make_rsrc(wsl, WS_NEED);
  GAS u32* tmo = (GAS u32*)(wsl + OFF_TMO);
  const u32 xcd = blockIdx.x & 7u, pos = blockIdx.x >> 3;
  STAMP(wsl, 0);

  if (tid < (L_Z1 - L_Z0) / 4u) *(LAS u32*)(smem + L_Z0 + tid * 4u) = 0u;
  if (tid >= 496u) {
    const u32 m = tid - 496u;
    u32 cs = 0x0c0c0c0cu, r = 0;
    u64 xs = 0x0c0c0c0c0c0c0c0cull;
    for (u32 i = 0; i < 4; ++i)
      if ((m >> i) & 1u) {
        cs = (cs & ~(0xffu << (8 * r))) | (i << (8 * r));
        xs = (xs & ~(0xffffull << (16 * i))) | ((u64)(r | ((4u + r) << 8)) << (16 * i));
        ++r;
      }
    if (pos < 16u) *(LAS u32*)(smem + L_LUT + 4 * m) = cs;
    else *(LAS u64*)(smem + L_LUT + 8 * m) = xs;
  }
  GAS u32* callp = (GAS u32*)(wsl + OFF_CALL + blockIdx.x * 64u);

  if (pos < 16u) {
    LAS u8* bld = smem + wid * 18432u;
    LAS u8* stg = bld + 10240;
    LAS u32* exs = (LAS u32*)(smem + L_EXC + wid * 256u);
    const LAS u32* lutc = (const LAS u32*)(smem + L_LUT);
    const rsrc_t rx = make_rsrc(prm.x, MM * KSH * 2), rw = make_rsrc(prm.w, KSH * NN * 2);
    char* pws = (char*)prm.ws[lane & 7u];
    u32 cv = 0;
    if (wid == 0u) cv = __hip_atomic_load(callp, __ATOMIC_RELAXED, __HIP_MEMORY_SCOPE_AGENT) + 1u;
    WAIT_LGKM0();
    __builtin_amdgcn_s_barrier();
    CFENCE();
    u32 call = 0;
    const u32 gi = pos * 8u + xcd;
    const bool fast = gi >= 128u - NFU && wid < 2u;
    if (!fast) __builtin_amdgcn_s_sleep(SDELAY);
    for (u32 rnd = fast ? 0u : 1u; rnd < 4u; ++rnd) {
      const u32 t = rnd < 3u ? (rnd - 1u) * 128u + gi : 256u + pos * 4u + (xcd - 4u);
      if (rnd == 3u && (xcd < 4u || t >= NTASK)) break;
      u32 isw = 1, gid, dest = rank, cw = wid;
      if (rnd == 0u) { isw = 0; gid = (gi - (128u - NFU)) >> 2; cw = (gi & 3u) * 2u + wid; }
      else if (t < NP) { gid = t; }
      else {
        const u32 u = t + 7u * NP + 8u;
        if (u < 72u) gid = u - 8u;
        else if (u < 80u) { isw = 0; gid = u - 64u; }
        else if (u < 144u) gid = u - 16u;
        else if (u < 160u) { isw = 0; gid = u - 128u; }
        else if (u < 288u) gid = u - 32u;
        else { isw = 0; gid = u - 256u; }
        if (isw) { dest = gid & 7u; gid >>= 3; }
      }
      const u32 id = gid * 8u + cw;
      u32 P[32];
      u32x4 xr[8];
      if (isw) {
        w_stage(rw, stg, dest, id, lane);
      } else {
        const u32 rb = id >> 5, ksl = (id >> 2) & 7u, mq = id & 3u;
        const u32 voff = (lane & 15u) * (KSH * 2) + (lane >> 4) * 16u;
        const u32 soff = (rb * 256u + mq * 64u) * (KSH * 2) + ksl * 128u;
#pragma unroll
        for (int v = 0; v < 8; ++v) xr[v] = __builtin_amdgcn_raw_buffer_load_b128(rx, voff + (v & 3) * 16 * (KSH * 2) + (v >> 2) * 64, soff, 0);
      }
      DBG(const u64 ti = now();)
      if (wid == 0u && call == 0u) {
        call = rfl(cv);
        lds_st(smem, L_CALL, call);
        __hip_atomic_store(callp, call, __ATOMIC_RELAXED, __HIP_MEMORY_SCOPE_AGENT);
      }
      if (isw) {
        WAIT_VM(0);
        w_fragments(stg, lane, P);
      } else {
#pragma unroll
        for (int v = 0; v < 8; ++v) { P[4 * v] = xr[v].x; P[4 * v + 1] = xr[v].y; P[4 * v + 2] = xr[v].z; P[4 * v + 3] = xr[v].w; }
      }
      DBG(for (int i = 0; i < 32; ++i) asm volatile("" : "+v"(P[i])); const u64 tl = now();)
      const bool send = !isw || dest != rank, keep = !isw || dest == rank;
      if (send)
        encode_chunk<false>(P, bld, exs, lutc, ws, isw ? OFF_EW + (dest * 32u + gid) * 65536u : OFF_EX + gid * 65536u,
                            smem + L_SZ + rnd * 32u, cw, lane, false);
      DBG(const u64 te = now();)
      if (send)
        publish(smem, rnd, pws, rank, rnd == 0u ? id >> 1 : isw ? pos_w(gid) : pos_x(gid), lane,
                      lane < (u32)NR && lane != rank && (!isw || lane == dest));
      DBG(if (rnd == (fast ? 0u : 1u)) { STAMP(wsl, 14); STAMPV(wsl, 12, tl); STAMPV(wsl, 13, te); STAMPV(wsl, 15, ti); })
      if (keep) {
        u32 o;
        if (isw) o = OFF_BT + (id >> 4) * (1u << 20) + (rank * 16u + (id & 15u)) * 8192u;
        else o = OFF_AT + (id >> 5) * (2u << 20) + (rank * 8u + ((id >> 2) & 7u)) * 32768u + (id & 3u) * 4096u;
        const u32 hstride = isw ? 4096u : 16384u;
#pragma unroll
        for (int v = 0; v < 8; ++v) {
          const u32x4 q = {P[4 * v], P[4 * v + 1], P[4 * v + 2], P[4 * v + 3]};
          st16<16>(q, ws, lane * 16u + (v & 3) * 1024u, o + (v >> 2) * hstride);
        }
        WAIT_VM(0);
        if (lane == 0)
          __hip_atomic_fetch_add((GAS u32*)(wsl + OFF_CNT + (isw ? 16u + (id >> 4) : (id >> 5)) * 128u), 1u, __ATOMIC_RELAXED, __HIP_MEMORY_SCOPE_AGENT);
      }
    }
    STAMP(wsl, 1);
    const rsrc_t ro = make_rsrc(prm.out, (u32)MM * NN * 2u);
    const u32 wm = wid & 3u, wn = wid >> 2;
    for (u32 round = 0; round < 2u; ++round) {
      const u32 seq = round * 8u + xcd;
      u32 si, sj;
      st_coord(seq, si, sj);
      const u32 rb = si * 4u + (pos >> 2), cb = sj * 4u + (pos & 3u);
      f32x4 acc[4][4];
#pragma unroll
      for (int a = 0; a < 4; ++a)
#pragma unroll
        for (int c = 0; c < 4; ++c) acc[a][c] = (f32x4){0.f, 0.f, 0.f, 0.f};
      u32 dead = 0;
      DBG(u64 ck0 = 0;)
      for (u32 seg = 0; seg < 2u; ++seg) {
        if (wid == 0) {
          const GAS u32* ca = (const GAS u32*)(wsl + OFF_CNT + (seg * 32u + rb) * 128u);
          const GAS u32* cbp = (const GAS u32*)(wsl + OFF_CNT + (seg * 32u + 16u + cb) * 128u);
          const u32 per = seg ? 7u : 1u, na = call * per * 32u, nb = call * per * 16u;
          const u64 t0 = now();
          u32 dd = 0;
          for (u32 it = 0;; ++it) {
            const u32 va = rfl(__hip_atomic_load(ca, __ATOMIC_RELAXED, __HIP_MEMORY_SCOPE_SYSTEM));
            const u32 vb = rfl(__hip_atomic_load(cbp, __ATOMIC_RELAXED, __HIP_MEMORY_SCOPE_SYSTEM));
            if (va >= na && vb >= nb) break;
            if ((it & 7u) == 7u && (timed_out(tmo, call) || now() - t0 > DEADLINE)) {
              __hip_atomic_store(tmo, call, __ATOMIC_RELAXED, __HIP_MEMORY_SCOPE_AGENT);
              break;
            }
            if (va + per * 16u < na || vb + per * 8u < nb) __builtin_amdgcn_s_sleep(PSLOW);
            else __builtin_amdgcn_s_sleep(PFAST);
          }
          if (timed_out(tmo, call)) dd = 1;
          if (dd) lds_st(smem, L_DEAD, 1u);
        }
        asm volatile("s_waitcnt vmcnt(0) lgkmcnt(0)" ::: "memory");
        __builtin_amdgcn_s_barrier();
        CFENCE();
        dead = lds_ld(smem, L_DEAD);
        if (seg) STAMP(wsl, 2 + round * 3);
        DBG(if (seg) ck0 = __builtin_amdgcn_s_memtime();)

        const u32 nks = seg ? 56u : 8u;
        u32 kn = rank * 8u + seg * 8u;
        const u32 ga = OFF_AT + rb * (2u << 20) + wid * 4096u, gb = OFF_BT + cb * (1u << 20) + wid * 2048u;
        const u32 la = (u32)(size_t)smem + wid * 4096u, lb = (u32)(size_t)smem + 32768u + wid * 2048u;
        const u32 vo = lane * 16u;
#define STG_A(I) dma16<(I) * 1024>(ws, la + sbuf, vo, ga + (kn & 63u) * 32768u)
#define STG_B(I) dma16<(I) * 1024>(ws, lb + sbuf, vo, gb + (kn & 63u) * 16384u)
        asm volatile("s_nop 4");
#pragma unroll
        for (int i = 0; i < 3; ++i) {
          const u32 sbuf = i * L_STAGE;
          STG_B(0); STG_B(1); STG_A(0); STG_A(1); STG_A(2); STG_A(3);
          ++kn;
        }
        const LAS u8* ra = smem + wm * 4096u + lane * 16u;
        const LAS u8* rbp = smem + 32768u + wn * 4096u + lane * 16u;
        bf16x8 a0[4], b0[4], a1[4], b1[4];
        WAIT_VM(12);
        __builtin_amdgcn_s_barrier();
        CFENCE();
#pragma unroll
        for (int i = 0; i < 4; ++i) {
          b0[i] = *(const LAS bf16x8*)(rbp + i * 1024);
          a0[i] = *(const LAS bf16x8*)(ra + i * 1024);
        }
        u32 sbuf = 0;
#define MM_(FA, FB, A, C) acc[A][C] = __builtin_amdgcn_mfma_f32_16x16x32_bf16(FB[C], FA[A], acc[A][C], 0, 0, 0)
#define RD(DST, PTR) DST = *(const LAS bf16x8*)(PTR)
#define KSTEP(MODE) do { \
          const LAS u8* pa = ra + sbuf; const LAS u8* pb = rbp + sbuf; \
          SB(); MM_(a0, b0, 0, 0); SB(); RD(b1[0], pb + 8192); SB(); MM_(a0, b0, 0, 1); SB(); RD(a1[0], pa + 16384); \
          SB(); MM_(a0, b0, 0, 2); SB(); RD(b1[1], pb + 9216); SB(); MM_(a0, b0, 0, 3); SB(); RD(b1[2], pb + 10240); \
          SB(); MM_(a0, b0, 1, 0); SB(); RD(b1[3], pb + 11264); SB(); MM_(a0, b0, 1, 1); SB(); RD(a1[1], pa + 17408); \
          SB(); MM_(a0, b0, 1, 2); SB(); RD(a1[2], pa + 18432); SB(); MM_(a0, b0, 1, 3); SB(); RD(a1[3], pa + 19456); \
          SB(); MM_(a0, b0, 2, 0); MM_(a0, b0, 2, 1); MM_(a0, b0, 2, 2); MM_(a0, b0, 2, 3); \
          MM_(a0, b0, 3, 0); MM_(a0, b0, 3, 1); MM_(a0, b0, 3, 2); MM_(a0, b0, 3, 3); SB(); \
          const u32 nbuf = sbuf == 2u * L_STAGE ? 0u : sbuf + L_STAGE; \
          if (MODE < 3) { \
            if (MODE < 2) asm volatile("s_waitcnt vmcnt(6) lgkmcnt(0)" ::: "memory"); \
            else asm volatile("s_waitcnt vmcnt(0) lgkmcnt(0)" ::: "memory"); \
            __builtin_amdgcn_s_barrier(); CFENCE(); \
            pa = ra + nbuf; pb = rbp + nbuf; \
          } \
          SB(); MM_(a1, b1, 0, 0); SB(); if (MODE < 3) RD(b0[0], pb); if (MODE == 0) STG_B(0); SB(); MM_(a1, b1, 0, 1); SB(); if (MODE < 3) RD(a0[0], pa); \
          SB(); MM_(a1, b1, 0, 2); SB(); if (MODE < 3) RD(b0[1], pb + 1024); SB(); MM_(a1, b1, 0, 3); SB(); if (MODE < 3) RD(b0[2], pb + 2048); if (MODE == 0) STG_B(1); \
          SB(); MM_(a1, b1, 1, 0); SB(); if (MODE < 3) RD(b0[3], pb + 3072); SB(); MM_(a1, b1, 1, 1); SB(); if (MODE < 3) RD(a0[1], pa + 1024); \
          SB(); MM_(a1, b1, 1, 2); SB(); if (MODE < 3) RD(a0[2], pa + 2048); if (MODE == 0) STG_A(0); SB(); MM_(a1, b1, 1, 3); SB(); if (MODE < 3) RD(a0[3], pa + 3072); \
          SB(); MM_(a1, b1, 2, 0); SB(); MM_(a1, b1, 2, 1); SB(); if (MODE == 0) STG_A(1); \
          SB(); MM_(a1, b1, 2, 2); SB(); MM_(a1, b1, 2, 3); SB(); MM_(a1, b1, 3, 0); SB(); if (MODE == 0) STG_A(2); \
          SB(); MM_(a1, b1, 3, 1); MM_(a1, b1, 3, 2); MM_(a1, b1, 3, 3); SB(); if (MODE == 0) STG_A(3); SB(); \
          ++kn; sbuf = nbuf; \
        } while (0)
        for (u32 ks = 3; ks < nks; ++ks) KSTEP(0);
        KSTEP(1);
        KSTEP(2);
        KSTEP(3);
      }
      STAMP(wsl, 3 + round * 3);
      DBG(STAMPV(wsl, 8 + round, __builtin_amdgcn_s_memtime() - ck0);)
      u32 P[32];
#pragma unroll
      for (int a = 0; a < 4; ++a)
#pragma unroll
        for (int c = 0; c < 4; ++c)
#pragma unroll
          for (int j = 0; j < 2; ++j) {
            const f32x2 v = {__builtin_elementwise_maximum(acc[a][c][2 * j], 0.f), __builtin_elementwise_maximum(acc[a][c][2 * j + 1], 0.f)};
            const u32 pk = __builtin_bit_cast(u32, __builtin_convertvector(v, bf16x2));
            P[a * 8 + c * 2 + j] = dead ? 0x7fc07fc0u : pk;
          }
      WAIT_LGKM0();
      __builtin_amdgcn_s_barrier();
      CFENCE();
      const u32 ts = seq * 16u + pos;
      DBG(const u64 te0 = now();)
      encode_chunk<true>(P, bld, exs, lutc, ws, OFF_EO + ts * 65536u, smem + L_SZ + (4u + round) * 32u, wid, lane, false);
      DBG(const u64 te1 = now();)
      publish(smem, 4u + round, pws, rank, 96u + ts, lane, lane < (u32)NR && lane != rank);
      DBG(STAMPV(wsl, 10, te1 - te0); STAMPV(wsl, 11, now() - te1);)
      STAMP(wsl, 4 + round * 3);
      {
        const u32 voff = (lane & 15u) * (NN * 2u) + (lane >> 4) * 16u;
        const u32 soff = (rb * 256u + wm * 64u) * (NN * 2u) + (rank * NSH + cb * 128u + wn * 64u) * 2u;
#pragma unroll
        for (int v = 0; v < 8; ++v) {
          const u32x4 q = {P[4 * v], P[4 * v + 1], P[4 * v + 2], P[4 * v + 3]};
          st16<OAUX>(q, ro, voff + (v & 1) * 64u, soff + (v >> 1) * 16u * (NN * 2u));
        }
      }
    }
  } else {
    const u32 cid = (pos - 16u) * 8u + xcd;
    if (cid >= 7u * NL) return;
    const u32 src = (rank + 1u + cid % 7u) & 7u;
    GAS u32* tk = (GAS u32*)(wsl + OFF_TK + src * 128u);
    u32 tnext = 0, cv = 0;
    if (wid == 1u) {
      cv = __hip_atomic_load(callp, __ATOMIC_RELAXED, __HIP_MEMORY_SCOPE_AGENT) + 1u;
      if (lane == 0) tnext = __hip_atomic_fetch_add(tk, 1u, __ATOMIC_RELAXED, __HIP_MEMORY_SCOPE_AGENT);
    }
    constexpr u32 PW = (NP * 8 + NL - 1) / NL;
    static_assert(PW <= 4 && NP % 2 == 0, "push");
    const u32 pq = cid / 7u + (wid - 2u) * NL;
    const bool pusher = wid >= 2u && wid - 2u < PW && pq < NP * 8u;
    LAS u8* pstg = smem + NREG * 16384 - (wid - 1u) * 8192u;
    if (pusher) w_stage(make_rsrc(prm.w, KSH * NN * 2), pstg, src, pq, lane);
    WAIT_LGKM0();
    __builtin_amdgcn_s_barrier();
    CFENCE();
    if (pusher) {
      u32 P[32];
      WAIT_VM(0);
      DBG(STAMP(wsl, 8);)
      w_fragments(pstg, lane, P);
      const rsrc_t rp = make_rsrc(prm.ws[src], WS_NEED);
      const u32 o = OFF_BT + (pq >> 4) * (1u << 20) + (rank * 16u + (pq & 15u)) * 8192u;
#pragma unroll
      for (int v = 0; v < 8; ++v) {
        const u32x4 q = {P[4 * v], P[4 * v + 1], P[4 * v + 2], P[4 * v + 3]};
        st16<17>(q, rp, lane * 16u + v * 1024u, o);
        asm volatile("s_waitcnt vmcnt(" STR(PWIN) ")" ::: "memory");
      }
      WAIT_VM(0);
      if (lane == 0)
        __hip_atomic_fetch_add((GAS u32*)((char*)prm.ws[src] + OFF_CNT + (48u + (pq >> 4)) * 128u), 1u, __ATOMIC_RELAXED, __HIP_MEMORY_SCOPE_SYSTEM);
      DBG(STAMP(wsl, 9);)
    }
    if (wid >= 2u && wid < 6u && lane == 0)
      __hip_atomic_fetch_add((LAS u32*)(smem + L_FREE + (NREG - 1u - ((wid - 2u) >> 1)) * 4u), 1u, __ATOMIC_RELAXED, __HIP_MEMORY_SCOPE_WORKGROUP);
    if (wid == 1u) {
      const u32 call = rfl(cv);
      lds_st(smem, L_CALL, call);
      __hip_atomic_store(callp, call, __ATOMIC_RELAXED, __HIP_MEMORY_SCOPE_AGENT);
      const u32 tbase = (call - 1u) * (u32)(NTK + NL);
      u32 nreq = 0, njob = 0, pieces = 0, region = 0, dead = 0;
      u32 cum = (lane == NREG - 1u || lane == NREG - 2u) ? 2u : 0u;
      u32 wbase = 0;
      u64 win = 0;
      __builtin_amdgcn_s_sleep(FIRSTSLEEP);
      DBG(u64 tw[4] = {};)
      for (;;) {
        const u32 t = rfl(tnext) - tbase;
        if (t >= (u32)NTK) break;
        u32 g, c0, cnt, ust;
        if (t < NTI) { g = t >> 2; c0 = (t & 3u) * 2u; cnt = 2; ust = c0; }
        else if (t < NTI + 2u * (256 - TAILT)) { const u32 tt = t - NTI; g = 96u + (tt >> 1); c0 = (tt & 1u) * 4u; cnt = 4; ust = c0; }
        else { const u32 tt = (t - (NTI + 2u * (256 - TAILT))) * TAILC; g = 96u + (256 - TAILT) + (tt >> 3); c0 = tt & 7u; cnt = TAILC; ust = c0 & 4u; }
        const u32 hx = t < NFU ? t : 32u + 2u * g + (c0 >> 2);
        u64 rec = 0;
        if (hx - wbase < 64u)
          rec = ((u64)__builtin_amdgcn_readlane((u32)(win >> 32), hx - wbase) << 32) | (u32)__builtin_amdgcn_readlane((u32)win, hx - wbase);
        if ((u32)(rec >> 40) != (call & 0xffffffu)) {
          const u64 t0 = now();
          const GAS u64* fp = (const GAS u64*)(wsl + flag_off(src, 0)) + hx;
          wbase = hx;
          DBG(u32 polls = 0;)
          if (!dead) win = __hip_atomic_load(fp + lane, __ATOMIC_RELAXED, __HIP_MEMORY_SCOPE_SYSTEM);
          rec = ((u64)rfl((u32)(win >> 32)) << 32) | rfl((u32)win);
          for (u32 it = 0; !dead && (u32)(rec >> 40) != (call & 0xffffffu); ++it) {
            DBG(++polls;)
            if ((it & 7u) == 7u && (timed_out(tmo, call) || now() - t0 > DEADLINE)) {
              __hip_atomic_store(tmo, call, __ATOMIC_RELAXED, __HIP_MEMORY_SCOPE_AGENT);
              dead = 1;
            }
            __builtin_amdgcn_s_sleep(FSLEEP);
            const u64 f = __hip_atomic_load(fp, __ATOMIC_RELAXED, __HIP_MEMORY_SCOPE_SYSTEM);
            rec = ((u64)rfl((u32)(f >> 32)) << 32) | rfl((u32)f);
          }
          if (dead) rec = 0;
          DBG(if (polls && nreq > 0) tw[t < NTI ? 0 : 2] += now() - t0;)
        }
        u32 kind, id;
        const u32 code = pos_code(g);
        code_inv(code, kind, id);
        const u32 ub = (kind == 0 ? OFF_EW + rank * (32u * 65536u) : kind == 1 ? OFF_EX : OFF_EO) + (id + ust) * 8192u;
        auto sz = [&](u32 i) { const u32 v = (u32)(rec >> (10u * (i & 3u))) & 1023u; return v > 512u ? 512u : v; };
        u32 pre = 0, tot = 0;
        for (u32 i = ust; i < c0; ++i) pre += sz(i);
        for (u32 i = c0; i < c0 + cnt; ++i) tot += sz(i);
        const u32 take = tot + (pre & 7u) <= 1024u ? cnt : 1u;
        if (pieces - lds_ld(smem, L_LANDED) > (u32)THRESH) {
          const u64 t0 = now();
          while (pieces - lds_ld(smem, L_LANDED) > (u32)THRESH && now() - t0 < 4 * DEADLINE) __builtin_amdgcn_s_sleep(DSLEEP);
        }
        for (u32 i = c0; i < c0 + cnt; i += take) {
          const u32 lead = pre & 7u;
          const u32 need = __builtin_amdgcn_readlane(cum, region);
          lds_st(smem, L_REQ + (nreq % NQ) * 8u + 4u, ub + (pre - lead) * 16u);
          lds_st(smem, L_REQ + (nreq % NQ) * 8u, ((nreq + 1u) & 255u) | ((lead + (take == cnt ? tot : sz(i))) << 8) | (need << 19));
          u32 off = lead;
          for (u32 j = 0; j < take; ++j) {
            const u32 v = sz(i + j), e1 = (off + v + 63u) >> 6;
            lds_st(smem, L_JOB + (njob % NJ) * 8u + 4u, (code + i + j) | (v << 12) | (off << 22));
            lds_st(smem, L_JOB + (njob % NJ) * 8u, ((njob + 1u) & 255u) | (region << 8) | ((pieces + (e1 ? e1 : 1u)) << 16));
            off += v;
            ++njob;
          }
          cum += lane == region ? take : 0u;
          const u32 np = (off + 63u) >> 6;
          pieces += np ? np : 1u;
          pre += off - lead;
          ++nreq;
          region = region == NREG - 1u ? 0u : region + 1u;
        }
        if (lane == 0) tnext = __hip_atomic_fetch_add(tk, 1u, __ATOMIC_RELAXED, __HIP_MEMORY_SCOPE_AGENT);
      }
      lds_st(smem, L_JEND, njob + 1u);
      lds_st(smem, L_REND, nreq + 1u);
      DBG(for (int i = 0; i < 4; ++i) STAMPV(wsl, 10 + i, tw[i]);)
    } else if (wid == 0) {
      const rsrc_t rp = make_rsrc(prm.ws[src], WS_NEED);
      const u32 lbase = (u32)(size_t)smem;
      u32 issued = 0, pub = 0, region = 0;
      DBG(u64 tw[4] = {}; u32 seen = 0;)
      for (u32 n = 0;; ++n) {
        u32 rq = lds_ld(smem, L_REQ + (n % NQ) * 8u);
        bool stop = false;
        if ((rq & 255u) != ((n + 1u) & 255u) || lds_ld(smem, L_FREE + region * 4u) < (rq >> 19)) {
          const u64 t0 = now();
          DBG(const bool wf = (rq & 255u) != ((n + 1u) & 255u);)
          for (;;) {
            if (issued > pub) {
              switch (issued - pub) {
                case 2: WAIT_VM(1); break;
                case 3: WAIT_VM(2); break;
                case 4: WAIT_VM(3); break;
                case 5: WAIT_VM(4); break;
                case 6: WAIT_VM(5); break;
                case 7: WAIT_VM(6); break;
                default: WAIT_VM(0); break;
              }
              ++pub;
              lds_st(smem, L_LANDED, pub);
            } else {
              const u32 ne = lds_ld(smem, L_REND);
              if ((ne != 0u && n + 1u >= ne) || now() - t0 > 4 * DEADLINE) { stop = true; break; }
              __builtin_amdgcn_s_sleep(1);
            }
            rq = lds_ld(smem, L_REQ + (n % NQ) * 8u);
            if ((rq & 255u) == ((n + 1u) & 255u) && lds_ld(smem, L_FREE + region * 4u) >= (rq >> 19)) break;
          }
          DBG(if (n > 0 && !stop) { u64 dt = now() - t0; if (!(seen & 16)) { if (wf) tw[0] += dt; else tw[1] += dt; } else { if (wf) tw[2] += dt; else tw[3] += dt; } })
        }
        if (stop) break;
        const u32 go = lds_ld(smem, L_REQ + (n % NQ) * 8u + 4u), tot = (rq >> 8) & 2047u;
        u32 np = (tot + 63u) >> 6;
        np = np ? (np > 16u ? 16u : np) : 1u;
        for (u32 pc = 0; pc < np; ++pc) {
          const u32 o16 = pc * 64u + lane;
          dma16_sys(rp, lbase + region * 16384u + pc * 1024u, o16 < tot ? o16 * 16u : 0x80000000u, go);
          ++issued;
          if (issued >= pub + WINDOW) {
            asm volatile("s_waitcnt vmcnt(" STR(WINDOW) " - 1)" ::: "memory");
            pub = issued - (WINDOW - 1);
            lds_st(smem, L_LANDED, pub);
          }
        }
        region = region == NREG - 1u ? 0u : region + 1u;
      }
      STAMP(wsl, 3);
      DBG(for (int i = 0; i < 4; ++i) STAMPV(wsl, 10 + i, tw[i]);)
    } else {
      const rsrc_t ro = make_rsrc(prm.out, (u32)MM * NN * 2u);
      LAS u8* hs = smem + L_HS + (wid - 2u) * L_HS_STRIDE;
      const LAS u32x2* lutx = (const LAS u32x2*)(smem + L_LUT);
      DBG(u64 td[4] = {};)
      for (u32 n = wid - 2u;; n += NDEC) {
        bool stop = false;
        u32 j0;
        {
          const u64 t0 = now();
          for (;;) {
            j0 = lds_ld(smem, L_JOB + (n % NJ) * 8u);
            if ((j0 & 255u) == ((n + 1u) & 255u)) {
              if (lds_ld(smem, L_LANDED) >= (j0 >> 16)) break;
            } else {
              const u32 ne = lds_ld(smem, L_JEND);
              if (ne != 0u && n + 1u >= ne) { stop = true; break; }
            }
            if (now() - t0 > 5 * DEADLINE) {
              if (lane == 0) {
                __hip_atomic_store(tmo, lds_ld(smem, L_CALL), __ATOMIC_RELAXED, __HIP_MEMORY_SCOPE_AGENT);
                ((GAS unsigned short*)prm.out)[src * NSH] = 0x7fc0;
              }
              stop = true;
              break;
            }
            __builtin_amdgcn_s_sleep(DSLEEP);
          }
        }
        if (stop) break;
        CFENCE();
        DBG(const u64 td0 = now(); if (n == wid - 2u) STAMP(wsl, 1);)
        const u32 j1 = lds_ld(smem, L_JOB + (n % NJ) * 8u + 4u);
        const u32 size16 = (j1 >> 12) & 1023u, region = (j0 >> 8) & 7u;
        const LAS u8* ch = smem + region * 16384u + (j1 >> 22) * 16u;
        const bool dead = size16 == 0u;
        u32 kind, id;
        code_inv(j1 & 4095u, kind, id);
        u32 P[32];
        if (dead) {
#pragma unroll
          for (int i = 0; i < 32; ++i) P[i] = 0x7fc07fc0u;
        } else if (kind == 2) {
          decode_chunk<true>(ch, hs, lutx, size16, lane, P);
        } else {
          decode_chunk<false>(ch, hs, lutx, size16, lane, P);
        }
#pragma unroll
        for (int i = 0; i < 32; ++i) asm volatile("" : "+v"(P[i]));
        WAIT_LGKM0();
        if (lane == 0) __hip_atomic_fetch_add((LAS u32*)(smem + L_FREE + region * 4u), 1u, __ATOMIC_RELAXED, __HIP_MEMORY_SCOPE_WORKGROUP);
        DBG(td[kind == 2 ? 2 : 0] += now() - td0;)
        if (kind == 2) {
          u32 si, sj;
          st_coord(id >> 7, si, sj);
          const u32 t = (id >> 3) & 15u, w = id & 7u;
          const u32 voff = (lane & 15u) * (NN * 2u) + (lane >> 4) * 16u;
          const u32 soff = ((si * 4u + (t >> 2)) * 256u + (w & 3u) * 64u) * (NN * 2u) + (src * NSH + (sj * 4u + (t & 3u)) * 128u + (w >> 2) * 64u) * 2u;
#pragma unroll
          for (int v = 0; v < 8; ++v) {
            const u32x4 q = {P[4 * v], P[4 * v + 1], P[4 * v + 2], P[4 * v + 3]};
            st16<OAUX>(q, ro, voff + (v & 1) * 64u, soff + (v >> 1) * 16u * (NN * 2u));
          }
        } else {
          u32 o;
          if (kind == 0) o = OFF_BT + (id >> 4) * (1u << 20) + (src * 16u + (id & 15u)) * 8192u;
          else o = OFF_AT + (id >> 5) * (2u << 20) + (src * 8u + ((id >> 2) & 7u)) * 32768u + (id & 3u) * 4096u;
          const u32 hstride = kind == 0 ? 4096u : 16384u;
#pragma unroll
          for (int v = 0; v < 8; ++v) {
            const u32x4 q = {P[4 * v], P[4 * v + 1], P[4 * v + 2], P[4 * v + 3]};
            st16<16>(q, ws, lane * 16u + (v & 3) * 1024u, o + (v >> 2) * hstride);
          }
          WAIT_VM(0);
          if (lane == 0)
            __hip_atomic_fetch_add((GAS u32*)(wsl + OFF_CNT + (32u + (kind == 0 ? 16u + (id >> 4) : (id >> 5))) * 128u), 1u, __ATOMIC_RELAXED, __HIP_MEMORY_SCOPE_AGENT);
        }
        DBG(td[kind == 2 ? 3 : 1] += now() - td0; if (n == wid - 2u) STAMP(wsl, 2);)
      }
      STAMP(wsl, 3);
      DBG(for (int i = 0; i < 4; ++i) STAMPV(wsl, 10 + i, td[i]);)
    }
  }
}

extern "C" void kernel_launch(int rank, int nranks, ncclComm_t comm, hipStream_t stream, void* const* d_in, const int* in_sizes,
                              int n_in, void* d_out, int out_size, const int* in_dtypes, int out_dtype, void* const* d_out_all,
                              void* const* d_ws_all, size_t ws_bytes) {
  (void)comm; (void)in_sizes; (void)n_in; (void)out_size; (void)in_dtypes; (void)out_dtype; (void)d_out_all;
  if (nranks != NR || ws_bytes < WS_NEED || (unsigned)rank >= (unsigned)NR) {
    fprintf(stderr, "unsupported configuration\n");
    return;
  }
  Params p;
  p.x = d_in[0];
  p.w = d_in[1];
  p.out = d_out;
  for (int i = 0; i < NR; ++i) p.ws[i] = d_ws_all[i];
  p.rank = rank;
  p.pad = 0;
  gemm_ar_kernel<<<NWG, NTHR, 0, stream>>>(p);
}
```

```cpp
#include <hip/hip_runtime.h>
#include <rccl/rccl.h>
#include <cstdio>

typedef unsigned u32;
typedef unsigned long long u64;
typedef unsigned char u8;
typedef u32 u32x2 __attribute__((ext_vector_type(2)));
typedef u32 u32x4 __attribute__((ext_vector_type(4)));
typedef float f32x2 __attribute__((ext_vector_type(2)));
typedef float f32x4 __attribute__((ext_vector_type(4)));
typedef short s16x2 __attribute__((ext_vector_type(2)));
typedef short s16x4 __attribute__((ext_vector_type(4)));
typedef __bf16 bf16x2 __attribute__((ext_vector_type(2)));
typedef __bf16 bf16x8 __attribute__((ext_vector_type(8)));
#define LAS __attribute__((address_space(3)))
#define GAS __attribute__((address_space(1)))
typedef __amdgpu_buffer_rsrc_t rsrc_t;

namespace {

constexpr int NR = 8;
constexpr int MM = 4096, NN = 16384, KSH = 512, NSH = 2048;
constexpr int NWG = 256, NTHR = 512;
#define NLOAD 18
constexpr int NL = NLOAD;
#define NFASTG 16
constexpr u32 NF = NFASTG;
constexpr u32 NFU = 4 * NF;
constexpr int NGRP = 32 + 96 + 256;
#define TAILT 16
#define TAILC 2
#define NPUSHG 4
constexpr u32 NP = NPUSHG;
constexpr u32 NTI = 4 * (96 - NP);
constexpr int NTK = NTI + 2 * (256 - TAILT) + (8 / TAILC) * TAILT;
constexpr u32 NTASK = 320 - NF - 7 * NP;
#define SDELAY 8
#define SSTAG 8
constexpr int NREG = 7, NDEC = 6, NQ = 32, NJ = 128;
#define WINDOW 3
#define THRESH 6
#define OAUX 16
#define PSLOW 127
#define PFAST 16
#define DSLEEP 3
#define PWIN 1
#define FSLEEP 8
#define FIRSTSLEEP 48
constexpr u64 DEADLINE = 30000000ull;

constexpr u32 OFF_CALL = 0;
constexpr u32 OFF_TMO = 16384;
constexpr u32 OFF_TK = 17408;
constexpr u32 OFF_CNT = 20480;
constexpr u32 OFF_FLAG = 32768;
constexpr u32 OFF_DBG = 512u << 10;
constexpr u32 OFF_EX = 1u << 20;
constexpr u32 OFF_EW = 5u << 20;
constexpr u32 OFF_EO = 21u << 20;
constexpr u32 OFF_AT = 37u << 20;
constexpr u32 OFF_BT = 69u << 20;
constexpr u32 WS_NEED = 85u << 20;

constexpr int L_STAGE = 49152;
constexpr int L_HS = NREG * 16384 + 512;
constexpr int L_HS_STRIDE = 4160;
constexpr int L_MISC = 3 * L_STAGE;
constexpr int L_LUT = L_MISC;
constexpr int L_Z0 = L_MISC + 128;
constexpr int L_REQ = L_Z0;
constexpr int L_JOB = L_REQ + 8 * NQ;
constexpr int L_FREE = L_JOB + 8 * NJ;
constexpr int L_LANDED = L_FREE + 32;
constexpr int L_REND = L_LANDED + 4;
constexpr int L_JEND = L_LANDED + 8;
constexpr int L_DEAD = L_LANDED + 12;
constexpr int L_CALL = L_LANDED + 16;
constexpr int L_SZ = L_LANDED + 32;
constexpr int L_ARR = L_SZ + 192;
constexpr int L_Z1 = L_ARR + 32;
constexpr int L_EXC = L_MISC + 2048;
constexpr int L_TOTAL = L_EXC + 8 * 256;
static_assert(L_HS + NDEC * L_HS_STRIDE <= L_MISC, "lds");
static_assert(L_Z1 <= L_EXC && (L_Z1 - L_Z0) / 4 <= NTHR, "lds");
static_assert(L_TOTAL <= 163840, "lds");

constexpr u32 HD_OFF = 512, UN_OFF = 516;

struct Params {
  const void* x;
  const void* w;
  void* out;
  void* ws;
  int rank;
  int pad;
  void* peer[NR];
};

#define CFENCE() asm volatile("" ::: "memory")
#define WAIT_VM(n) asm volatile("s_waitcnt vmcnt(" #n ")" ::: "memory")
#define WAIT_LGKM0() asm volatile("s_waitcnt lgkmcnt(0)" ::: "memory")
#define SB() __builtin_amdgcn_sched_barrier(0)
#define STR2(x) #x
#define STR(x) STR2(x)

__device__ __forceinline__ u32 rfl(u32 v) { return __builtin_amdgcn_readfirstlane(v); }
__device__ __forceinline__ u32 vperm(u32 hi, u32 lo, u32 sel) { return __builtin_amdgcn_perm(hi, lo, sel); }
__device__ __forceinline__ u32 bsum(u32 v, u32 acc) { return __builtin_amdgcn_sad_u8(v, 0u, acc); }
__device__ __forceinline__ u64 now() { return __builtin_amdgcn_s_memrealtime(); }
__device__ __forceinline__ u32 mbcnt(u64 m) {
  return __builtin_amdgcn_mbcnt_hi((u32)(m >> 32), __builtin_amdgcn_mbcnt_lo((u32)m, 0u));
}
__device__ __forceinline__ u32 scan_incl(u32 x) {
  x += __builtin_amdgcn_update_dpp(0u, x, 0x111, 0xf, 0xf, true);
  x += __builtin_amdgcn_update_dpp(0u, x, 0x112, 0xf, 0xf, true);
  x += __builtin_amdgcn_update_dpp(0u, x, 0x114, 0xf, 0xf, true);
  x += __builtin_amdgcn_update_dpp(0u, x, 0x118, 0xf, 0xf, true);
  x += __builtin_amdgcn_update_dpp(0u, x, 0x142, 0xa, 0xf, false);
  x += __builtin_amdgcn_update_dpp(0u, x, 0x143, 0xc, 0xf, false);
  return x;
}
__device__ __forceinline__ rsrc_t make_rsrc(const void* p, u32 bytes) {
  return __builtin_amdgcn_make_buffer_rsrc((void*)p, 0, bytes, 0x00020000);
}
#define GSC " sc0 sc1"
template <int IMM>
__device__ __forceinline__ void dma16(rsrc_t r, u32 lds, u32 voff, u32 soff) {
  asm volatile("s_mov_b32 m0, %0\n\ts_nop 0\n\tbuffer_load_dwordx4 %1, %2, %3 offen offset:%4" GSC " lds"
               :: "s"(lds), "v"(voff), "s"(r), "s"(soff), "i"(IMM) : "memory");
}
template <int AUX>
__device__ __forceinline__ void st16(u32x4 v, rsrc_t r, u32 voff, u32 soff) {
  __builtin_amdgcn_raw_buffer_store_b128(v, r, voff, soff, AUX);
  SB();
  asm volatile("s_nop 1");
  SB();
}
__device__ __forceinline__ u32 lds_ld(LAS u8* lds, int off) {
  return rfl(*(volatile LAS u32*)(lds + off));
}
__device__ __forceinline__ void lds_st(LAS u8* lds, int off, u32 v) { *(volatile LAS u32*)(lds + off) = v; }

#define STAMP(ws, slot) do {} while (0)
#define STAMPV(ws, slot, v) do {} while (0)
#define DBG(x)

__device__ __forceinline__ u64 rank_table(u32 k) {
  const u64 below = (1ull << (8 * k)) - 1ull;
  return (0x0001020304050607ull & ~below & ~(0xffull << (8 * k))) | (7ull << (8 * k)) | (0x0000010203040506ull & below);
}
template <int U>
__device__ __forceinline__ u32 unit_offset(LAS u8* szw, u32 wid) {
  u32 off = 0;
  for (u32 j = wid & ~(U - 1u); j < wid; ++j) {
    u32 v = lds_ld(szw, j * 4u);
    for (int spin = 0; v == 0u && spin < (1 << 22); ++spin) v = lds_ld(szw, j * 4u);
    off += v;
  }
  return off;
}
template <bool OUT>
__device__ __forceinline__ void encode_chunk(const u32 (&P)[32], LAS u8* bld, LAS u32* exs, const LAS u32* lutc, rsrc_t ws,
                                             u32 gbase, LAS u8* szw, u32 wid, u32 lane, bool force_raw) {
  constexpr int U = OUT ? 4 : 2;
  const u32 ub = gbase + (wid & ~(U - 1u)) * 8192u;
  u32 lo[16], len[16], nzm[16];
  u32 pl[2] = {0u, 0u};
  u32 size16 = 512u, nexc = 0, nw = 0, lbase = 0, pbase = 0, qbase = 0, header = 0;
  if (!force_raw) {
    u32 hi[16];
    u32 sum = 0, sgn = 0;
#pragma unroll
    for (int g = 0; g < 16; ++g) {
      lo[g] = vperm(P[2 * g + 1], P[2 * g], 0x06040200u);
      hi[g] = vperm(P[2 * g + 1], P[2 * g], 0x07050301u);
      if (OUT) {
        const s16x2 z = {0, 0};
        const u32 n0 = __builtin_bit_cast(u32, z - __builtin_bit_cast(s16x2, P[2 * g]));
        const u32 n1 = __builtin_bit_cast(u32, z - __builtin_bit_cast(s16x2, P[2 * g + 1]));
        nzm[g] = vperm(n1, n0, 0x0b0a0908u);
        pl[g >> 3] |= bsum(nzm[g] & 0x08040201u, 0u) << (4 * (g & 7));
        sgn |= hi[g];
      } else {
        pl[g >> 3] |= (hi[g] >> (7 - (g & 7))) & (0x01010101u << (g & 7));
        hi[g] &= 0x7f7f7f7fu;
        nzm[g] = 0xffffffffu;
      }
      if (g & 1) sum = bsum(hi[g] + hi[g - 1], sum);
    }
    const bool signed_out = OUT && __builtin_amdgcn_ballot_w64((sgn & 0x80808080u) != 0u) != 0ull;
    const u32 cnt = OUT ? (u32)(__builtin_popcount(pl[0]) + __builtin_popcount(pl[1])) : 64u;
    const u32 sc0 = scan_incl((sum & 0x1fffu) | (cnt << 19));
    const u32 tot0 = __builtin_amdgcn_readlane(sc0, 63);
    const u32 nnz = tot0 >> 19;
    pbase = (sc0 >> 19) - cnt;
    const float mean = (float)(tot0 & 0x7ffffu) * __builtin_amdgcn_rcpf((float)(nnz ? nnz : 1u));
    int b = (int)(mean + 0.78f);
    const float fr = mean - (float)(b - 1);
    const u32 k = rfl(1u + (fr < 0.32f) + (fr < 0.41f) + (fr < 0.60f) + (fr < 0.87f));
    b = b < 6 ? 6 : b > 126 ? 126 : b;
    const u32 base = rfl((u32)(b - 6));
    const u32 cneg = 0x80808080u - base * 0x01010101u;
    const u64 inv = (0x0001020304050607ull + (0x0101010101010101ull >> (8 * (k + 1)))) | ((u64)(k + 1) << 56);
    u32 inv_lo = (u32)inv;
    const u32 inv_hi = (u32)(inv >> 32);
    asm volatile("" : "+v"(inv_lo));
    u32 anybad = 0;
#pragma unroll
    for (int g = 0; g < 16; ++g) {
      const u32 e4 = (hi[g] + cneg) ^ 0x80808080u;
      anybad |= e4 & nzm[g];
      len[g] = vperm(inv_hi, inv_lo, e4) & nzm[g];
      if (OUT) lo[g] = vperm(0u, lo[g], lutc[(pl[g >> 3] >> (4 * (g & 7))) & 15u]);
    }
    if (__builtin_amdgcn_ballot_w64((anybad & 0xf8f8f8f8u) != 0u) != 0ull) {
      u32 bp[2] = {0u, 0u};
#pragma unroll
      for (int g = 0; g < 16; ++g) {
        const u32 bad = ((hi[g] + cneg) ^ 0x80808080u) & 0xf8f8f8f8u & nzm[g];
        const u32 t = ((bad >> 1) + 0x7c7c7c7cu) & 0x80808080u;
        const u32 fm = (t << 1) - (t >> 7);
        len[g] = (len[g] & ~fm) | (t >> 7);
        bp[g >> 3] |= bsum(fm & 0x08040201u, 0u) << (4 * (g & 7));
      }
#pragma unroll
      for (int j = 0; j < 4; ++j) {
        const u32x4 t = {hi[4 * j], hi[4 * j + 1], hi[4 * j + 2], hi[4 * j + 3]};
        *(LAS u32x4*)(bld + lane * 64u + j * 16) = t;
      }
      CFENCE();
      u64 bm = ((u64)bp[1] << 32) | bp[0];
      const u64 plm = ((u64)pl[1] << 32) | pl[0];
      for (;;) {
        const u64 m = __builtin_amdgcn_ballot_w64(bm != 0ull);
        if (m == 0ull || nexc > 64u) break;
        if (bm != 0ull) {
          const u32 j = __builtin_ctzll(bm);
          bm &= bm - 1ull;
          const u32 slot = nexc + mbcnt(m);
          const u32 idx = pbase + (OUT ? (u32)__builtin_popcountll(plm & ((1ull << j) - 1ull)) : j);
          if (slot < 64u) exs[slot] = (idx << 8) | bld[lane * 64u + j];
        }
        nexc += __builtin_popcountll(m);
      }
      CFENCE();
    }
    u32 acc = 0;
#pragma unroll
    for (int g = 0; g < 16; ++g) acc += len[g];
    const u32 bits = bsum(acc, 0u);
    const u32 sc1 = scan_incl(bits);
    qbase = sc1 - bits;
    nw = (__builtin_amdgcn_readlane(sc1, 63) + 31u) >> 5;
    lbase = UN_OFF + 4u * nw + 4u * nexc;
    if (!OUT) lbase = (lbase + 15u) & ~15u;
    size16 = (lbase + nnz + 15u) >> 4;
    if (size16 >= 512u || nexc > 64u || signed_out) size16 = 512u;
    header = base | (k << 7) | (nw << 10) | (nexc << 21);
  }
  lds_st(szw, wid * 4u, size16);
  if (size16 == 512u) {
    const u32 go = ub + unit_offset<U>(szw, wid) * 16u;
#pragma unroll
    for (int v = 0; v < 8; ++v) {
      const u32x4 q = {P[4 * v], P[4 * v + 1], P[4 * v + 2], P[4 * v + 3]};
      st16<17>(q, ws, lane * 16u + v * 1024u, go);
    }
    return;
  }
  {
    const u32x4 z = {0u, 0u, 0u, 0u};
    for (u32 o = HD_OFF; o < (OUT ? size16 * 16u : lbase); o += 1024u) *(LAS u32x4*)(bld + o + lane * 16u) = z;
  }
  CFENCE();
  u32 q = qbase + ((u32)(size_t)bld + UN_OFF) * 8u;
  u32 p = pbase + (u32)(size_t)bld + lbase;
  u32 ones = 0x01010101u;
  asm volatile("" : "+v"(ones));
#pragma unroll
  for (int g0 = 0; g0 < 16; g0 += 4) {
    u64 v[4], u[4];
    u32 da[4], ea[4];
#pragma unroll
    for (int i = 0; i < 4; ++i) {
      const int g = g0 + i;
      u32 t = len[g] + (len[g] << 8);
      asm volatile("" : "+v"(t));
      t += t << 16;
      const u32 tb = t + 0x1f1f1f1fu;
      const u32 one = OUT ? (nzm[g] & 0x01010101u) : ones;
      u32 b4, b5, b6;
      asm("v_lshlrev_b32_sdwa %0, %3, %4 dst_sel:DWORD dst_unused:UNUSED_PAD src0_sel:BYTE_0 src1_sel:BYTE_0\n\t"
          "v_lshlrev_b32_sdwa %1, %3, %4 dst_sel:DWORD dst_unused:UNUSED_PAD src0_sel:BYTE_1 src1_sel:BYTE_1\n\t"
          "v_lshlrev_b32_sdwa %2, %3, %4 dst_sel:DWORD dst_unused:UNUSED_PAD src0_sel:BYTE_2 src1_sel:BYTE_2\n\t"
          "v_or3_b32 %0, %0, %1, %2\n\t"
          "v_lshlrev_b32_sdwa %1, %3, %4 dst_sel:DWORD dst_unused:UNUSED_PAD src0_sel:BYTE_3 src1_sel:BYTE_3\n\t"
          "v_or_b32 %0, %0, %1"
          : "=&v"(b4), "=&v"(b5), "=&v"(b6) : "v"(tb), "v"(one));
      v[i] = (u64)b4 << (q & 31u);
      da[i] = (q >> 3) & ~3u;
      q += t >> 24;
      if (OUT) {
        u[i] = (u64)lo[g] << ((p & 3u) * 8u);
        ea[i] = p & ~3u;
        p += __builtin_popcount(one);
      }
    }
    SB();
#pragma unroll
    for (int i = 0; i < 4; ++i) {
      LAS u32* d = (LAS u32*)(size_t)da[i];
      __hip_atomic_fetch_or(d, (u32)v[i], __ATOMIC_RELAXED, __HIP_MEMORY_SCOPE_WAVEFRONT);
      __hip_atomic_fetch_or(d + 1, (u32)(v[i] >> 32), __ATOMIC_RELAXED, __HIP_MEMORY_SCOPE_WAVEFRONT);
      if (OUT) {
        LAS u32* e = (LAS u32*)(size_t)ea[i];
        __hip_atomic_fetch_or(e, (u32)u[i], __ATOMIC_RELAXED, __HIP_MEMORY_SCOPE_WAVEFRONT);
        __hip_atomic_fetch_or(e + 1, (u32)(u[i] >> 32), __ATOMIC_RELAXED, __HIP_MEMORY_SCOPE_WAVEFRONT);
      }
    }
    SB();
  }
  if (!OUT) {
#pragma unroll
    for (int j = 0; j < 4; ++j) {
      const u32x4 t = {lo[4 * j], lo[4 * j + 1], lo[4 * j + 2], lo[4 * j + 3]};
      *(LAS u32x4*)(bld + lbase + j * 1024 + lane * 16u) = t;
    }
  }
  {
    const u32x2 t = {pl[0], pl[1]};
    *(LAS u32x2*)(bld + lane * 8u) = t;
  }
  CFENCE();
  *(LAS u32*)(bld + HD_OFF) = header;
  if (lane < nexc) *(LAS u32*)(bld + UN_OFF + 4u * nw + 4u * lane) = exs[lane];
  CFENCE();
  const u32 go = ub + unit_offset<U>(szw, wid) * 16u;
  constexpr int NB = OUT ? 4 : 6;
  for (u32 o = 0; o < size16; o += 64u * NB) {
    u32x4 t[NB];
#pragma unroll
    for (int j = 0; j < NB; ++j) t[j] = *(const LAS u32x4*)(bld + ((o * 16u + j * 1024u) & 8191u) + lane * 16u);
#pragma unroll
    for (int j = 0; j < NB; ++j) {
      const u32 o16 = o + j * 64u + lane;
      st16<17>(t[j], ws, o16 < size16 ? o16 * 16u : 0x80000000u, go);
    }
  }
  CFENCE();
}

__device__ __forceinline__ u32 parse4(u32& w, u32& q, u32 tab_hi, u32 tab_lo) {
  u32 r, p, t;
  asm("v_ffbl_b32 %3, %1\n\tv_add_u32 %4, -1, %1\n\tv_sub_u32 %0, %3, %2\n\tv_and_b32 %1, %4, %1\n\t"
      "v_ffbl_b32 %2, %1\n\tv_add_u32 %4, -1, %1\n\t"
      "v_sub_u32_sdwa %0, %2, %3 dst_sel:BYTE_1 dst_unused:UNUSED_PRESERVE src0_sel:DWORD src1_sel:DWORD\n\tv_and_b32 %1, %4, %1\n\t"
      "v_ffbl_b32 %3, %1\n\tv_add_u32 %4, -1, %1\n\t"
      "v_sub_u32_sdwa %0, %3, %2 dst_sel:BYTE_2 dst_unused:UNUSED_PRESERVE src0_sel:DWORD src1_sel:DWORD\n\tv_and_b32 %1, %4, %1\n\t"
      "v_ffbl_b32 %2, %1\n\tv_add_u32 %4, -1, %1\n\t"
      "v_sub_u32_sdwa %0, %2, %3 dst_sel:BYTE_3 dst_unused:UNUSED_PRESERVE src0_sel:DWORD src1_sel:DWORD\n\tv_and_b32 %1, %4, %1"
      : "=&v"(r), "+v"(w), "+v"(q), "=&v"(p), "=&v"(t));
  return vperm(tab_hi, tab_lo, r - 0x01010101u);
}

template <bool OUT, class LF>
__device__ __forceinline__ void decode_chunk(const LAS u8* ch, LAS u8* hs, const LAS u32x2* lutx, u32 size16, u32 lane,
                                             u32 (&P)[32], LF landed) {
  if (size16 >= 512u) {
    landed(8192u);
#pragma unroll
    for (int v = 0; v < 8; ++v) {
      const u32x4 t = *(const LAS u32x4*)(ch + v * 1024 + lane * 16u);
      P[4 * v] = t.x; P[4 * v + 1] = t.y; P[4 * v + 2] = t.z; P[4 * v + 3] = t.w;
    }
    return;
  }
  landed(UN_OFF);
  const u32 tr = rfl(*(const volatile LAS u32*)(ch + HD_OFF));
  const u32x2 pl = *(const LAS u32x2*)(ch + lane * 8u);
  const u32 nw = (tr >> 10) & 0x7ffu, nexc = (tr >> 21) & 0x7fu;
  const u64 dec = rank_table((tr >> 7) & 7u);
  const u32 base4 = (tr & 127u) * 0x01010101u;
  u32 tab_lo = (u32)dec + base4;
  const u32 tab_hi = (u32)(dec >> 32) + base4;
  asm volatile("" : "+v"(tab_lo));
  u32 p = 0;
  if (OUT) {
    const u32 c = __builtin_popcount(pl.x) + __builtin_popcount(pl.y);
    p = scan_incl(c) - c;
  }
  u32 lbase = UN_OFF + 4u * nw + 4u * nexc;
  landed(lbase);
  if (!OUT) lbase = (lbase + 15u) & ~15u;
  lbase &= 8191u;
  const LAS u32* U = (const LAS u32*)(ch + UN_OFF);
  u32 run = 0;
  for (u32 w0 = 0; w0 < nw; w0 += 64u) {
    const u32 i = w0 + lane;
    u32 w = U[i], prev = U[(int)i - 1];
    w = i < nw ? w : 0u;
    prev = i > 0u ? prev : 0x80000000u;
    const u32 c = __builtin_popcount(w);
    const u32 incl = scan_incl(c);
    const u32 dst = (u32)(size_t)hs + ((run + incl - c) & 4095u);
    run += __builtin_amdgcn_readlane(incl, 63);
    u32 q = ~(u32)__builtin_clz(prev | 1u);
    u32 R[8];
#pragma unroll
    for (int j = 0; j < 6; ++j) R[j] = parse4(w, q, tab_hi, tab_lo);
    R[6] = R[7] = 0u;
    if (__builtin_amdgcn_ballot_w64(w != 0u) != 0ull) {
      R[6] = parse4(w, q, tab_hi, tab_lo);
      if (__builtin_amdgcn_ballot_w64(w != 0u) != 0ull) R[7] = parse4(w, q, tab_hi, tab_lo);
    }
    const u32 wb = (dst - 1u) & ~3u, ca = c + dst - wb, sh = 0u - dst;
    u32 W[8];
    u64 m[8];
#pragma unroll
    for (int j = 0; j < 8; ++j) {
      W[j] = __builtin_amdgcn_alignbyte(j < 7 ? R[j + 1] : 0u, R[j], sh);
      m[j] = __builtin_amdgcn_ballot_w64(4u * (j + 1) < ca);
    }
    const u64 m0 = __builtin_amdgcn_ballot_w64(c != 0u);
    const u32 r1 = R[0] >> 8;
    asm volatile("s_mov_b64 exec, %9\n\tds_write_b32 %0, %1 offset:4\n\ts_mov_b64 exec, %10\n\tds_write_b32 %0, %2 offset:8\n\t"
                 "s_mov_b64 exec, %11\n\tds_write_b32 %0, %3 offset:12\n\ts_mov_b64 exec, %12\n\tds_write_b32 %0, %4 offset:16\n\t"
                 "s_mov_b64 exec, %13\n\tds_write_b32 %0, %5 offset:20\n\ts_mov_b64 exec, %14\n\tds_write_b32 %0, %6 offset:24\n\t"
                 "s_mov_b64 exec, %15\n\tds_write_b32 %0, %7 offset:28\n\ts_mov_b64 exec, %16\n\tds_write_b32 %0, %8 offset:32\n\t"
                 "s_mov_b64 exec, %17\n\tds_write_b8 %18, %19\n\tds_write_b8 %18, %20 offset:1\n\tds_write_b8_d16_hi %18, %19 offset:2\n\t"
                 "s_mov_b64 exec, -1"
                 :: "v"(wb), "v"(W[0]), "v"(W[1]), "v"(W[2]), "v"(W[3]), "v"(W[4]), "v"(W[5]), "v"(W[6]), "v"(W[7]),
                    "s"(m[0]), "s"(m[1]), "s"(m[2]), "s"(m[3]), "s"(m[4]), "s"(m[5]), "s"(m[6]), "s"(m[7]), "s"(m0),
                    "v"(dst), "v"(R[0]), "v"(r1) : "memory");
  }
  CFENCE();
  if (lane < nexc) {
    const u32 r = U[nw + lane];
    hs[(r >> 8) & 4095u] = (u8)r;
  }
  landed(size16 * 16u);
  if (OUT) {
#pragma unroll
    for (int g0 = 0; g0 < 16; g0 += 8) {
      u32 pp[8];
      u32x2 sel[8], L[8], H[8];
      const LAS u8 *sa[8], *la[8], *ha[8];
#pragma unroll
      for (int i = 0; i < 8; ++i) {
        const u32 wd = g0 ? pl.y : pl.x;
        const u32 m8 = (i == 0 ? wd << 3 : wd >> (4 * i - 3)) & 0x78u;
        pp[i] = p;
        sa[i] = (const LAS u8*)lutx + m8;
        la[i] = ch + lbase + (p & ~3u);
        ha[i] = hs + (p & ~3u);
        p += __builtin_popcount(m8);
      }
      SB();
#pragma unroll
      for (int i = 0; i < 8; ++i) {
        sel[i] = *(const LAS u32x2*)sa[i];
        L[i].x = ((const LAS u32*)la[i])[0]; L[i].y = ((const LAS u32*)la[i])[1];
        H[i].x = ((const LAS u32*)ha[i])[0]; H[i].y = ((const LAS u32*)ha[i])[1];
      }
      SB();
#pragma unroll
      for (int i = 0; i < 8; ++i) {
        const u32 l4 = __builtin_amdgcn_alignbyte(L[i].y, L[i].x, pp[i]);
        const u32 h4 = __builtin_amdgcn_alignbyte(H[i].y, H[i].x, pp[i]);
        P[2 * (g0 + i)] = vperm(h4, l4, sel[i].x);
        P[2 * (g0 + i) + 1] = vperm(h4, l4, sel[i].y);
      }
    }
  } else {
#pragma unroll
    for (int j = 0; j < 4; ++j) {
      const u32x4 L = *(const LAS u32x4*)(ch + lbase + j * 1024 + lane * 16u);
      const u32x4 H = *(const LAS u32x4*)(hs + lane * 64u + j * 16);
#pragma unroll
      for (int i = 0; i < 4; ++i) {
        const int g = 4 * j + i;
        const u32 hh = H[i] | (((g < 8 ? pl.x : pl.y) << (7 - (g & 7))) & 0x80808080u);
        P[2 * g] = vperm(hh, L[i], 0x05010400u);
        P[2 * g + 1] = vperm(hh, L[i], 0x07030602u);
      }
    }
  }
}

__device__ __forceinline__ u32 pos_x(u32 g) { return g + (g < NF ? 0u : g < 16u ? 8u - NP : g < 32u ? 16u - NP : 32u - NP); }
__device__ __forceinline__ u32 pos_w(u32 g) { return g - NP + (g < 8u ? NF : g < 16u ? 16u : 32u); }
__device__ __forceinline__ u32 pos_code(u32 p) {
  if (p >= 96u) return 768u + (p - 96u) * 8u;
  if (p < NF) return 256u + p * 8u;
  if (p < NF + 8u - NP) return (p + NP - NF) * 8u;
  if (p < 24u - NP) return 256u + (p + NP - 8u) * 8u;
  if (p < 32u - NP) return (p + NP - 16u) * 8u;
  if (p < 48u - NP) return 256u + (p + NP - 16u) * 8u;
  if (p < 64u - NP) return (p + NP - 32u) * 8u;
  return 256u + (p + NP - 32u) * 8u;
}
__device__ __forceinline__ void code_inv(u32 e, u32& kind, u32& id) {
  kind = e < 256u ? 0u : e < 768u ? 1u : 2u;
  id = e < 256u ? e : e < 768u ? e - 256u : e - 768u;
}
__device__ __forceinline__ void w_stage(rsrc_t rw, LAS u8* stg, u32 dest, u32 id, u32 lane) {
  const u32 cb = id >> 4, kb = id & 15u;
  const u32 voff = (lane >> 4) * (NN * 2) + (lane & 15u) * 16u;
  u32 soff = rfl(kb * 32u * (NN * 2) + (dest * NSH + cb * 128u) * 2u), lds = rfl((u32)(size_t)stg);
  asm volatile("s_nop 4" : "+s"(soff), "+s"(lds));
#pragma unroll
  for (int u = 0; u < 8; ++u) dma16<0>(rw, lds + u * 1024u, voff, soff + u * 4u * (NN * 2));
}
__device__ __forceinline__ void w_fragments(const LAS u8* stg, u32 lane, u32 (&P)[32]) {
  const LAS u8* tb = stg + ((lane >> 4) * 8u + ((lane & 15u) >> 2)) * 256u + (lane & 3u) * 16u;
#pragma unroll
  for (int nf = 0; nf < 8; ++nf)
#pragma unroll
    for (int h = 0; h < 2; ++h) {
      const s16x4 r = __builtin_amdgcn_ds_read_tr16_b64_v4i16((LAS s16x4*)(tb + h * 1024 + (nf >> 1) * 64 + (nf & 1) * 8));
      const u32x2 r2 = __builtin_bit_cast(u32x2, r);
      P[4 * nf + 2 * h] = r2.x;
      P[4 * nf + 2 * h + 1] = r2.y;
    }
}
__device__ __forceinline__ void x_stage(rsrc_t rx, LAS u8* stg, u32 id, u32 lane) {
  const u32 rb = id >> 5, ksl = (id >> 2) & 7u, mq = id & 3u;
  const u32 voff = (lane >> 3) * (KSH * 2) + (lane & 7u) * 16u;
  u32 soff = rfl((rb * 256u + mq * 64u) * (KSH * 2) + ksl * 128u), lds = rfl((u32)(size_t)stg);
  asm volatile("s_nop 4" : "+s"(soff), "+s"(lds));
#pragma unroll
  for (int u = 0; u < 8; ++u) dma16<0>(rx, lds + u * 1024u, voff, soff + u * 8u * (KSH * 2));
}
__device__ __forceinline__ void x_fragments(const LAS u8* stg, u32 lane, u32 (&P)[32]) {
  const LAS u8* tb = stg + (lane & 15u) * 128u + (lane >> 4) * 16u;
#pragma unroll
  for (int v = 0; v < 8; ++v) {
    const u32x4 r = *(const LAS u32x4*)(tb + (v & 3) * 2048 + (v >> 2) * 64);
    P[4 * v] = r.x; P[4 * v + 1] = r.y; P[4 * v + 2] = r.z; P[4 * v + 3] = r.w;
  }
}
__device__ __forceinline__ bool task_of(u32 rnd, u32 gi, u32 wid, u32 rank, u32& isw, u32& gid, u32& dest, u32& cw) {
  const u32 t = rnd < 3u ? (rnd - 1u) * 128u + gi : 256u + (gi >> 3) * 4u + ((gi & 7u) - 4u);
  isw = 1; gid = 0; dest = rank; cw = wid;
  if (rnd > 3u || (rnd == 3u && ((gi & 7u) < 4u || t >= NTASK))) return false;
  if (rnd == 0u) { isw = 0; gid = (gi - (128u - NFU)) >> 2; cw = (gi & 3u) * 2u + wid; }
  else if (t < NP) { gid = t; }
  else {
    const u32 u = t + 7u * NP + NF;
    if (u < 64u + NF) gid = u - NF;
    else if (u < 80u) { isw = 0; gid = u - 64u; }
    else if (u < 144u) gid = u - 16u;
    else if (u < 160u) { isw = 0; gid = u - 128u; }
    else if (u < 288u) gid = u - 32u;
    else { isw = 0; gid = u - 256u; }
    if (isw) { dest = gid & 7u; gid >>= 3; }
  }
  return true;
}
__device__ __forceinline__ void st_coord(u32 seq, u32& i, u32& j) {
  if (seq >= 8u) { i = seq >> 2; j = seq & 3u; }
  else if (seq < 4u) { i = seq >> 1; j = seq & 1u; }
  else { i = seq & 1u; j = seq >> 1; }
}
__device__ __forceinline__ u32 flag_off(u32 src, u32 g) { return OFF_FLAG + (src * NGRP + g) * 16u; }
__device__ __forceinline__ void publish(LAS u8* smem, u32 rnd, char* pws, u32 rank, u32 g, u32 lane, bool sel) {
  WAIT_VM(0);
  asm volatile("" : "+v"(pws));
  u32 a = 0;
  if (lane == 0) a = __hip_atomic_fetch_add((LAS u32*)(smem + L_ARR + rnd * 4u), 1u, __ATOMIC_RELAXED, __HIP_MEMORY_SCOPE_WORKGROUP);
  if (rfl(a) == (rnd ? 7u : 1u)) {
    CFENCE();
    const u32x4 s0 = *(volatile LAS u32x4*)(smem + L_SZ + rnd * 32u), s1 = *(volatile LAS u32x4*)(smem + L_SZ + rnd * 32u + 16u);
    const u32 c8 = lds_ld(smem, L_CALL) << 8;
    const u32x4 rec = {s0.x | (s0.y << 10) | (s0.z << 20) | (s0.w << 30), c8 | (s0.w >> 2),
                       s1.x | (s1.y << 10) | (s1.z << 20) | (s1.w << 30), c8 | (s1.w >> 2)};
    if (sel) {
      if (rnd) {
        asm volatile("global_store_dwordx4 %0, %1, off sc0 sc1\n\ts_nop 1" :: "v"(pws + flag_off(rank, 32u + g)), "v"(rec) : "memory");
      } else {
        const u32x2 h = {rec.x | rec.z, rec.y | rec.w};
        asm volatile("global_store_dwordx2 %0, %1, off sc0 sc1\n\ts_nop 1" :: "v"(pws + flag_off(rank, 0u) + g * 8u), "v"(h) : "memory");
      }
    }
  }
}

__device__ __forceinline__ bool timed_out(const GAS u32* tmo, u32 call) {
  return rfl(__hip_atomic_load(tmo, __ATOMIC_RELAXED, __HIP_MEMORY_SCOPE_AGENT)) == call;
}

}

__global__ __launch_bounds__(NTHR, 2) void gemm_ar_kernel(Params prm) {
  __shared__ __attribute__((aligned(16))) u8 smem_raw[L_TOTAL];
  LAS u8* smem = (LAS u8*)smem_raw;
  const u32 tid = threadIdx.x, lane = tid & 63u, wid = rfl(tid >> 6);
  const u32 rank = (u32)prm.rank;
  char* wsl = (char*)prm.ws;
  const rsrc_t ws = make_rsrc(wsl, WS_NEED);
  GAS u32* tmo = (GAS u32*)(wsl + OFF_TMO);
  const u32 xcd = blockIdx.x & 7u, pos = blockIdx.x >> 3;
  DBG(const u64 tstart = now();)
  const rsrc_t rx = make_rsrc(prm.x, MM * KSH * 2), rw = make_rsrc(prm.w, KSH * NN * 2);
  const bool fast = blockIdx.x >= 128u - NFU && blockIdx.x < 128u && wid < 2u;
  if (fast) x_stage(rx, smem + wid * 18432u + 10240u, (blockIdx.x - (128u - NFU)) * 2u + wid, lane);
  const u32 cid = blockIdx.x - 128u;
  constexpr u32 PW = (NP * 8 + NL - 1) / NL;
  static_assert(PW <= 4, "push");
  const u32 pq = cid / 7u + (wid - 2u) * NL;
  const bool pusher = cid < 7u * NL && wid >= 2u && wid - 2u < PW && pq < NP * 8u;
  const u32 src = (rank + 1u + cid % 7u) & 7u;
  LAS u8* pstg = smem + NREG * 16384 - (wid - 1u) * 8192u;
  if (pusher) w_stage(rw, pstg, src, pq, lane);
  DBG(const u64 tiss = now(); STAMPV(wsl, 0, tstart);)

  if (tid < (L_Z1 - L_Z0) / 4u) *(LAS u32*)(smem + L_Z0 + tid * 4u) = 0u;
  if (tid >= 496u) {
    const u32 m = tid - 496u;
    u32 cs = 0x0c0c0c0cu, r = 0;
    u64 xs = 0x0c0c0c0c0c0c0c0cull;
    for (u32 i = 0; i < 4; ++i)
      if ((m >> i) & 1u) {
        cs = (cs & ~(0xffu << (8 * r))) | (i << (8 * r));
        xs = (xs & ~(0xffffull << (16 * i))) | ((u64)(r | ((4u + r) << 8)) << (16 * i));
        ++r;
      }
    if (pos < 16u) *(LAS u32*)(smem + L_LUT + 4 * m) = cs;
    else *(LAS u64*)(smem + L_LUT + 8 * m) = xs;
  }
  GAS u32* callp = (GAS u32*)(wsl + OFF_CALL + blockIdx.x * 64u);

  if (pos < 16u) {
    LAS u8* bld = smem + wid * 18432u;
    LAS u8* stg = bld + 10240;
    LAS u32* exs = (LAS u32*)(smem + L_EXC + wid * 256u);
    const LAS u32* lutc = (const LAS u32*)(smem + L_LUT);
    char* pws = (char*)prm.peer[lane & 7u];
    const u32 prank = (rank + 1u + lane) & 7u;
    const u32 cv = __hip_atomic_load(callp, __ATOMIC_RELAXED, __HIP_MEMORY_SCOPE_AGENT);
    WAIT_LGKM0();
    __builtin_amdgcn_s_barrier();
    CFENCE();
    u32 call = 0;
    const u32 gi = blockIdx.x;
    u32 isw, gid, dest, cw;
    task_of(fast ? 0u : 1u, gi, wid, rank, isw, gid, dest, cw);
    if (!fast) {
      for (u32 i = SDELAY + (pos * SSTAG) / 4u; i > 0u; --i) __builtin_amdgcn_s_sleep(8);
      if (isw) w_stage(rw, stg, dest, gid * 8u + cw, lane);
      else x_stage(rx, stg, gid * 8u + cw, lane);
    }
    bool more = true;
    for (u32 rnd = fast ? 0u : 1u; more; ++rnd) {
      task_of(rnd, gi, wid, rank, isw, gid, dest, cw);
      const u32 id = gid * 8u + cw;
      u32 P[32];
      DBG(const u64 ti = tiss;)
      WAIT_VM(0);
      if (isw) w_fragments(stg, lane, P);
      else x_fragments(stg, lane, P);
      if (wid == 0u && call == 0u) {
        call = rfl(cv) + 1u;
        lds_st(smem, L_CALL, call);
        __hip_atomic_store(callp, call, __ATOMIC_RELAXED, __HIP_MEMORY_SCOPE_AGENT);
      }
      {
        u32 isw2, gid2, dest2, cw2;
        more = task_of(rnd + 1u, gi, wid, rank, isw2, gid2, dest2, cw2);
#pragma unroll
        for (int i = 0; i < 32; ++i) asm volatile("" : "+v"(P[i]));
        WAIT_LGKM0();
        if (more && rnd != 0u) {
          if (isw2) w_stage(rw, stg, dest2, gid2 * 8u + cw2, lane);
          else x_stage(rx, stg, gid2 * 8u + cw2, lane);
        }
      }
      DBG(for (int i = 0; i < 32; ++i) asm volatile("" : "+v"(P[i])); const u64 tl = now();)
      const bool send = !isw || dest != rank, keep = !isw || dest == rank;
      if (send)
        encode_chunk<false>(P, bld, exs, lutc, ws, isw ? OFF_EW + (dest * 32u + gid) * 65536u : OFF_EX + gid * 65536u,
                            smem + L_SZ + rnd * 32u, cw, lane, false);
      DBG(const u64 te = now();)
      if (send)
        publish(smem, rnd, pws, rank, rnd == 0u ? id >> 1 : isw ? pos_w(gid) : pos_x(gid), lane,
                      lane < 7u && (!isw || prank == dest));
      DBG(if (rnd == (fast ? 0u : 1u)) { STAMP(wsl, 14); STAMPV(wsl, 12, tl); STAMPV(wsl, 13, te); STAMPV(wsl, 15, ti); })
      if (rnd == 0u) {
        u32 isw2, gid2, dest2, cw2;
        task_of(1u, gi, wid, rank, isw2, gid2, dest2, cw2);
        if (isw2) w_stage(rw, stg, dest2, gid2 * 8u + cw2, lane);
        else x_stage(rx, stg, gid2 * 8u + cw2, lane);
      }
      if (keep) {
        u32 o;
        if (isw) o = OFF_BT + (id >> 4) * (1u << 20) + (rank * 16u + (id & 15u)) * 8192u;
        else o = OFF_AT + (id >> 5) * (2u << 20) + (rank * 8u + ((id >> 2) & 7u)) * 32768u + (id & 3u) * 4096u;
        const u32 hstride = isw ? 4096u : 16384u;
#pragma unroll
        for (int v = 0; v < 8; ++v) {
          const u32x4 q = {P[4 * v], P[4 * v + 1], P[4 * v + 2], P[4 * v + 3]};
          st16<16>(q, ws, lane * 16u + (v & 3) * 1024u, o + (v >> 2) * hstride);
        }
        WAIT_VM(0);
        if (lane == 0)
          __hip_atomic_fetch_add((GAS u32*)(wsl + OFF_CNT + (isw ? 16u + (id >> 4) : (id >> 5)) * 128u), 1u, __ATOMIC_RELAXED, __HIP_MEMORY_SCOPE_AGENT);
      }
    }
    STAMP(wsl, 1);
    const rsrc_t ro = make_rsrc(prm.out, (u32)MM * NN * 2u);
    const u32 wm = wid & 3u, wn = wid >> 2;
    for (u32 round = 0; round < 2u; ++round) {
      const u32 seq = round * 8u + xcd;
      u32 si, sj;
      st_coord(seq, si, sj);
      const u32 rb = si * 4u + (pos >> 2), cb = sj * 4u + (pos & 3u);
      f32x4 acc[4][4];
#pragma unroll
      for (int a = 0; a < 4; ++a)
#pragma unroll
        for (int c = 0; c < 4; ++c) acc[a][c] = (f32x4){0.f, 0.f, 0.f, 0.f};
      u32 dead = 0;
      DBG(u64 ck0 = 0;)
      for (u32 seg = 0; seg < 2u; ++seg) {
        if (wid == 0) {
          const GAS u32* ca = (const GAS u32*)(wsl + OFF_CNT + (seg * 32u + rb) * 128u);
          const GAS u32* cbp = (const GAS u32*)(wsl + OFF_CNT + (seg * 32u + 16u + cb) * 128u);
          const u32 per = seg ? 7u : 1u, na = call * per * 32u, nb = call * per * 16u;
          const u64 t0 = now();
          u32 dd = 0;
          for (u32 it = 0;; ++it) {
            const u32 va = rfl(__hip_atomic_load(ca, __ATOMIC_RELAXED, __HIP_MEMORY_SCOPE_SYSTEM));
            const u32 vb = rfl(__hip_atomic_load(cbp, __ATOMIC_RELAXED, __HIP_MEMORY_SCOPE_SYSTEM));
            if (va >= na && vb >= nb) break;
            if ((it & 7u) == 7u && (timed_out(tmo, call) || now() - t0 > DEADLINE)) {
              __hip_atomic_store(tmo, call, __ATOMIC_RELAXED, __HIP_MEMORY_SCOPE_AGENT);
              break;
            }
            if (va + per * 16u < na || vb + per * 8u < nb) __builtin_amdgcn_s_sleep(PSLOW);
            else __builtin_amdgcn_s_sleep(PFAST);
          }
          if (timed_out(tmo, call)) dd = 1;
          if (dd) lds_st(smem, L_DEAD, 1u);
        }
        asm volatile("s_waitcnt vmcnt(0) lgkmcnt(0)" ::: "memory");
        __builtin_amdgcn_s_barrier();
        CFENCE();
        dead = lds_ld(smem, L_DEAD);
        if (seg) STAMP(wsl, 2 + round * 3);
        DBG(if (seg) ck0 = __builtin_amdgcn_s_memtime();)

        const u32 nks = seg ? 56u : 8u;
        u32 kn = rank * 8u + seg * 8u;
        const u32 ga = OFF_AT + rb * (2u << 20) + wid * 4096u, gb = OFF_BT + cb * (1u << 20) + wid * 2048u;
        const u32 la = (u32)(size_t)smem + wid * 4096u, lb = (u32)(size_t)smem + 32768u + wid * 2048u;
        const u32 vo = lane * 16u;
#define STG_A(I) dma16<(I) * 1024>(ws, la + sbuf, vo, ga + (kn & 63u) * 32768u)
#define STG_B(I) dma16<(I) * 1024>(ws, lb + sbuf, vo, gb + (kn & 63u) * 16384u)
        asm volatile("s_nop 4");
#pragma unroll
        for (int i = 0; i < 3; ++i) {
          const u32 sbuf = i * L_STAGE;
          STG_B(0); STG_B(1); STG_A(0); STG_A(1); STG_A(2); STG_A(3);
          ++kn;
        }
        const LAS u8* ra = smem + wm * 4096u + lane * 16u;
        const LAS u8* rbp = smem + 32768u + wn * 4096u + lane * 16u;
        bf16x8 a0[4], b0[4], a1[4], b1[4];
        WAIT_VM(12);
        __builtin_amdgcn_s_barrier();
        CFENCE();
#pragma unroll
        for (int i = 0; i < 4; ++i) {
          b0[i] = *(const LAS bf16x8*)(rbp + i * 1024);
          a0[i] = *(const LAS bf16x8*)(ra + i * 1024);
        }
        u32 sbuf = 0;
#define MM_(FA, FB, A, C) acc[A][C] = __builtin_amdgcn_mfma_f32_16x16x32_bf16(FB[C], FA[A], acc[A][C], 0, 0, 0)
#define RD(DST, PTR) DST = *(const LAS bf16x8*)(PTR)
#define KSTEP(MODE) do { \
          const LAS u8* pa = ra + sbuf; const LAS u8* pb = rbp + sbuf; \
          SB(); MM_(a0, b0, 0, 0); SB(); RD(b1[0], pb + 8192); SB(); MM_(a0, b0, 0, 1); SB(); RD(a1[0], pa + 16384); \
          SB(); MM_(a0, b0, 0, 2); SB(); RD(b1[1], pb + 9216); SB(); MM_(a0, b0, 0, 3); SB(); RD(b1[2], pb + 10240); \
          SB(); MM_(a0, b0, 1, 0); SB(); RD(b1[3], pb + 11264); SB(); MM_(a0, b0, 1, 1); SB(); RD(a1[1], pa + 17408); \
          SB(); MM_(a0, b0, 1, 2); SB(); RD(a1[2], pa + 18432); SB(); MM_(a0, b0, 1, 3); SB(); RD(a1[3], pa + 19456); \
          SB(); MM_(a0, b0, 2, 0); MM_(a0, b0, 2, 1); MM_(a0, b0, 2, 2); MM_(a0, b0, 2, 3); \
          MM_(a0, b0, 3, 0); MM_(a0, b0, 3, 1); MM_(a0, b0, 3, 2); MM_(a0, b0, 3, 3); SB(); \
          const u32 nbuf = sbuf == 2u * L_STAGE ? 0u : sbuf + L_STAGE; \
          if (MODE < 3) { \
            if (MODE < 2) asm volatile("s_waitcnt vmcnt(6) lgkmcnt(0)" ::: "memory"); \
            else asm volatile("s_waitcnt vmcnt(0) lgkmcnt(0)" ::: "memory"); \
            __builtin_amdgcn_s_barrier(); CFENCE(); \
            pa = ra + nbuf; pb = rbp + nbuf; \
          } \
          SB(); MM_(a1, b1, 0, 0); SB(); if (MODE < 3) RD(b0[0], pb); if (MODE == 0) STG_B(0); SB(); MM_(a1, b1, 0, 1); SB(); if (MODE < 3) RD(a0[0], pa); \
          SB(); MM_(a1, b1, 0, 2); SB(); if (MODE < 3) RD(b0[1], pb + 1024); SB(); MM_(a1, b1, 0, 3); SB(); if (MODE < 3) RD(b0[2], pb + 2048); if (MODE == 0) STG_B(1); \
          SB(); MM_(a1, b1, 1, 0); SB(); if (MODE < 3) RD(b0[3], pb + 3072); SB(); MM_(a1, b1, 1, 1); SB(); if (MODE < 3) RD(a0[1], pa + 1024); \
          SB(); MM_(a1, b1, 1, 2); SB(); if (MODE < 3) RD(a0[2], pa + 2048); if (MODE == 0) STG_A(0); SB(); MM_(a1, b1, 1, 3); SB(); if (MODE < 3) RD(a0[3], pa + 3072); \
          SB(); MM_(a1, b1, 2, 0); SB(); MM_(a1, b1, 2, 1); SB(); if (MODE == 0) STG_A(1); \
          SB(); MM_(a1, b1, 2, 2); SB(); MM_(a1, b1, 2, 3); SB(); MM_(a1, b1, 3, 0); SB(); if (MODE == 0) STG_A(2); \
          SB(); MM_(a1, b1, 3, 1); MM_(a1, b1, 3, 2); MM_(a1, b1, 3, 3); SB(); if (MODE == 0) STG_A(3); SB(); \
          ++kn; sbuf = nbuf; \
        } while (0)
        for (u32 ks = 3; ks < nks; ++ks) KSTEP(0);
        KSTEP(1);
        KSTEP(2);
        KSTEP(3);
      }
      STAMP(wsl, 3 + round * 3);
      DBG(STAMPV(wsl, 8 + round, __builtin_amdgcn_s_memtime() - ck0);)
      u32 P[32];
#pragma unroll
      for (int a = 0; a < 4; ++a)
#pragma unroll
        for (int c = 0; c < 4; ++c)
#pragma unroll
          for (int j = 0; j < 2; ++j) {
            const f32x2 v = {__builtin_elementwise_maximum(acc[a][c][2 * j], 0.f), __builtin_elementwise_maximum(acc[a][c][2 * j + 1], 0.f)};
            const u32 pk = __builtin_bit_cast(u32, __builtin_convertvector(v, bf16x2));
            P[a * 8 + c * 2 + j] = dead ? 0x7fc07fc0u : pk;
          }
      WAIT_LGKM0();
      __builtin_amdgcn_s_barrier();
      CFENCE();
      const u32 ts = seq * 16u + pos;
      DBG(const u64 te0 = now();)
      encode_chunk<true>(P, bld, exs, lutc, ws, OFF_EO + ts * 65536u, smem + L_SZ + (4u + round) * 32u, wid, lane, false);
      DBG(const u64 te1 = now();)
      publish(smem, 4u + round, pws, rank, 96u + ts, lane, lane < 7u);
      DBG(STAMPV(wsl, 10, te1 - te0); STAMPV(wsl, 11, now() - te1);)
      STAMP(wsl, 4 + round * 3);
      {
        const u32 voff = (lane & 15u) * (NN * 2u) + (lane >> 4) * 16u;
        const u32 soff = (rb * 256u + wm * 64u) * (NN * 2u) + (rank * NSH + cb * 128u + wn * 64u) * 2u;
#pragma unroll
        for (int v = 0; v < 8; ++v) {
          const u32x4 q = {P[4 * v], P[4 * v + 1], P[4 * v + 2], P[4 * v + 3]};
          st16<OAUX>(q, ro, voff + (v & 1) * 64u, soff + (v >> 1) * 16u * (NN * 2u));
        }
      }
    }
  } else {
    if (cid >= 7u * NL) return;
    char* wsp = (char*)prm.peer[cid % 7u];
    WAIT_LGKM0();
    __builtin_amdgcn_s_barrier();
    CFENCE();
    if (pusher) {
      u32 P[32];
      WAIT_VM(0);
      DBG(STAMP(wsl, 8);)
      w_fragments(pstg, lane, P);
      const rsrc_t rp = make_rsrc(wsp, WS_NEED);
      const u32 o = OFF_BT + (pq >> 4) * (1u << 20) + (rank * 16u + (pq & 15u)) * 8192u;
#pragma unroll
      for (int v = 0; v < 8; ++v) {
        const u32x4 q = {P[4 * v], P[4 * v + 1], P[4 * v + 2], P[4 * v + 3]};
        st16<17>(q, rp, lane * 16u + v * 1024u, o);
        asm volatile("s_waitcnt vmcnt(" STR(PWIN) ")" ::: "memory");
      }
      WAIT_VM(0);
      if (lane == 0)
        __hip_atomic_fetch_add((GAS u32*)(wsp + OFF_CNT + (48u + (pq >> 4)) * 128u), 1u, __ATOMIC_RELAXED, __HIP_MEMORY_SCOPE_SYSTEM);
      DBG(STAMP(wsl, 9);)
    }
    if (wid >= 2u && wid < 6u && lane == 0)
      __hip_atomic_fetch_add((LAS u32*)(smem + L_FREE + (NREG - 1u - ((wid - 2u) >> 1)) * 4u), 1u, __ATOMIC_RELAXED, __HIP_MEMORY_SCOPE_WORKGROUP);
    if (wid == 1u) {
      GAS u32* tk = (GAS u32*)(wsl + OFF_TK + src * 128u);
      const u32 cv = __hip_atomic_load(callp, __ATOMIC_RELAXED, __HIP_MEMORY_SCOPE_AGENT);
      u32 tnext = 0;
      if (lane == 0) tnext = __hip_atomic_fetch_add(tk, 1u, __ATOMIC_RELAXED, __HIP_MEMORY_SCOPE_AGENT);
      const u32 call = rfl(cv) + 1u;
      lds_st(smem, L_CALL, call);
      __hip_atomic_store(callp, call, __ATOMIC_RELAXED, __HIP_MEMORY_SCOPE_AGENT);
      const u32 tbase = (call - 1u) * (u32)(NTK + NL);
      u32 nreq = 0, njob = 0, pieces = 0, region = 0, dead = 0;
      u32 cum = (lane == NREG - 1u || lane == NREG - 2u) ? 2u : 0u;
      u32 wbase = 0;
      u64 win = 0;
      __builtin_amdgcn_s_sleep(FIRSTSLEEP);
      DBG(u64 tw[4] = {};)
      for (;;) {
        const u32 t = rfl(tnext) - tbase;
        if (t >= (u32)NTK) break;
        u32 g, c0, cnt, ust;
        if (t < NTI) { g = t >> 2; c0 = (t & 3u) * 2u; cnt = 2; ust = c0; }
        else if (t < NTI + 2u * (256 - TAILT)) { const u32 tt = t - NTI; g = 96u + (tt >> 1); c0 = (tt & 1u) * 4u; cnt = 4; ust = c0; }
        else { const u32 tt = (t - (NTI + 2u * (256 - TAILT))) * TAILC; g = 96u + (256 - TAILT) + (tt >> 3); c0 = tt & 7u; cnt = TAILC; ust = c0 & 4u; }
        u32 kind, id;
        const u32 code = pos_code(g);
        code_inv(code, kind, id);
        const u32 ub = (kind == 0 ? OFF_EW + rank * (32u * 65536u) : kind == 1 ? OFF_EX : OFF_EO) + (id + ust) * 8192u;
        const u32 hx = t < NFU ? t : 64u + 2u * g + (c0 >> 2);
        u64 rec = 0;
        if (hx - wbase < 64u)
          rec = ((u64)__builtin_amdgcn_readlane((u32)(win >> 32), hx - wbase) << 32) | (u32)__builtin_amdgcn_readlane((u32)win, hx - wbase);
        if ((u32)(rec >> 40) != (call & 0xffffffu)) {
          const u64 t0 = now();
          const GAS u64* fp = (const GAS u64*)(wsl + flag_off(src, 0)) + hx;
          wbase = hx;
          DBG(u32 polls = 0;)
          if (!dead) win = __hip_atomic_load(fp + lane, __ATOMIC_RELAXED, __HIP_MEMORY_SCOPE_SYSTEM);
          rec = ((u64)rfl((u32)(win >> 32)) << 32) | rfl((u32)win);
          for (u32 it = 0; !dead && (u32)(rec >> 40) != (call & 0xffffffu); ++it) {
            DBG(++polls;)
            if ((it & 7u) == 7u && (timed_out(tmo, call) || now() - t0 > DEADLINE)) {
              __hip_atomic_store(tmo, call, __ATOMIC_RELAXED, __HIP_MEMORY_SCOPE_AGENT);
              dead = 1;
            }
            if (nreq != 0u) __builtin_amdgcn_s_sleep(FSLEEP);
            const u64 f = __hip_atomic_load(fp, __ATOMIC_RELAXED, __HIP_MEMORY_SCOPE_SYSTEM);
            rec = ((u64)rfl((u32)(f >> 32)) << 32) | rfl((u32)f);
          }
          if (dead) rec = 0;
          DBG(if (polls && nreq > 0) tw[t < NTI ? 0 : 2] += now() - t0;)
        }
        auto sz = [&](u32 i) { const u32 v = (u32)(rec >> (10u * (i & 3u))) & 1023u; return v > 512u ? 512u : v; };
        u32 pre = 0, tot = 0;
        for (u32 i = ust; i < c0; ++i) pre += sz(i);
        for (u32 i = c0; i < c0 + cnt; ++i) tot += sz(i);
        const u32 take = tot + (pre & 7u) <= 1024u ? cnt : 1u;
        if (pieces - lds_ld(smem, L_LANDED) > (u32)THRESH) {
          const u64 t0 = now();
          while (pieces - lds_ld(smem, L_LANDED) > (u32)THRESH && now() - t0 < 4 * DEADLINE) __builtin_amdgcn_s_sleep(DSLEEP);
        }
        for (u32 i = c0; i < c0 + cnt; i += take) {
          const u32 lead = pre & 7u;
          const u32 need = __builtin_amdgcn_readlane(cum, region);
          lds_st(smem, L_REQ + (nreq % NQ) * 8u + 4u, ub + (pre - lead) * 16u);
          lds_st(smem, L_REQ + (nreq % NQ) * 8u, ((nreq + 1u) & 255u) | ((lead + (take == cnt ? tot : sz(i))) << 8) | (need << 19));
          u32 off = lead;
          for (u32 j = 0; j < take; ++j) {
            const u32 v = sz(i + j);
            lds_st(smem, L_JOB + (njob % NJ) * 8u + 4u, (code + i + j) | (v << 12) | (off << 22));
            lds_st(smem, L_JOB + (njob % NJ) * 8u, ((njob + 1u) & 255u) | (region << 8) | (pieces << 16));
            off += v;
            ++njob;
          }
          cum += lane == region ? take : 0u;
          const u32 np = (off + 63u) >> 6;
          pieces += np ? np : 1u;
          pre += off - lead;
          ++nreq;
          region = region == NREG - 1u ? 0u : region + 1u;
        }
        if (lane == 0) tnext = __hip_atomic_fetch_add(tk, 1u, __ATOMIC_RELAXED, __HIP_MEMORY_SCOPE_AGENT);
      }
      lds_st(smem, L_JEND, njob + 1u);
      lds_st(smem, L_REND, nreq + 1u);
      DBG(for (int i = 0; i < 4; ++i) STAMPV(wsl, 10 + i, tw[i]);)
    } else if (wid == 0) {
      const rsrc_t rp = make_rsrc(wsp, WS_NEED);
      const u32 lbase = (u32)(size_t)smem;
      u32 issued = 0, pub = 0, region = 0;
      DBG(u64 tw[4] = {}; u32 seen = 0;)
      for (u32 n = 0;; ++n) {
        u32 rq = lds_ld(smem, L_REQ + (n % NQ) * 8u);
        bool stop = false;
        if ((rq & 255u) != ((n + 1u) & 255u) || lds_ld(smem, L_FREE + region * 4u) < (rq >> 19)) {
          const u64 t0 = now();
          DBG(const bool wf = (rq & 255u) != ((n + 1u) & 255u);)
          for (;;) {
            if (issued > pub) {
              switch (issued - pub) {
                case 2: WAIT_VM(1); break;
                case 3: WAIT_VM(2); break;
                case 4: WAIT_VM(3); break;
                case 5: WAIT_VM(4); break;
                case 6: WAIT_VM(5); break;
                case 7: WAIT_VM(6); break;
                case 8: WAIT_VM(7); break;
                default: WAIT_VM(0); break;
              }
              ++pub;
              lds_st(smem, L_LANDED, pub);
            } else {
              const u32 ne = lds_ld(smem, L_REND);
              if ((ne != 0u && n + 1u >= ne) || now() - t0 > 4 * DEADLINE) { stop = true; break; }
              __builtin_amdgcn_s_sleep(1);
            }
            rq = lds_ld(smem, L_REQ + (n % NQ) * 8u);
            if ((rq & 255u) == ((n + 1u) & 255u) && lds_ld(smem, L_FREE + region * 4u) >= (rq >> 19)) break;
          }
          DBG(if (n > 0 && !stop) { u64 dt = now() - t0; if (!(seen & 16)) { if (wf) tw[0] += dt; else tw[1] += dt; } else { if (wf) tw[2] += dt; else tw[3] += dt; } })
        }
        if (stop) break;
        const u32 go = lds_ld(smem, L_REQ + (n % NQ) * 8u + 4u), tot = (rq >> 8) & 2047u;
        u32 np = (tot + 63u) >> 6;
        np = np ? (np > 16u ? 16u : np) : 1u;
        for (u32 pc = 0; pc < np; ++pc) {
          const u32 o16 = pc * 64u + lane;
          const u32 vo = o16 < tot ? o16 * 16u : 0x80000000u, ld = lbase + region * 16384u + pc * 1024u;
          asm volatile("s_mov_b32 m0, %0\n\ts_nop 3\n\ts_waitcnt vmcnt(" STR(WINDOW) " - 1)\n\tbuffer_load_dwordx4 %1, %2, %3 offen sc0 sc1 lds"
                       :: "s"(ld), "v"(vo), "s"(rp), "s"(go) : "memory");
          if (issued >= pub + WINDOW) { pub = issued - (WINDOW - 1u); lds_st(smem, L_LANDED, pub); }
          ++issued;
        }
        region = region == NREG - 1u ? 0u : region + 1u;
      }
      STAMP(wsl, 3);
      DBG(for (int i = 0; i < 4; ++i) STAMPV(wsl, 10 + i, tw[i]);)
    } else {
      const rsrc_t ro = make_rsrc(prm.out, (u32)MM * NN * 2u);
      LAS u8* hs = smem + L_HS + (wid - 2u) * L_HS_STRIDE;
      const LAS u32x2* lutx = (const LAS u32x2*)(smem + L_LUT);
      DBG(u64 td[4] = {};)
      for (u32 n = wid - 2u;; n += NDEC) {
        bool stop = false;
        u32 j0;
        {
          const u64 t0 = now();
          for (;;) {
            j0 = lds_ld(smem, L_JOB + (n % NJ) * 8u);
            if ((j0 & 255u) == ((n + 1u) & 255u)) break;
            const u32 ne = lds_ld(smem, L_JEND);
            if (ne != 0u && n + 1u >= ne) { stop = true; break; }
            if (now() - t0 > 5 * DEADLINE) {
              if (lane == 0) {
                __hip_atomic_store(tmo, lds_ld(smem, L_CALL), __ATOMIC_RELAXED, __HIP_MEMORY_SCOPE_AGENT);
                ((GAS unsigned short*)prm.out)[src * NSH] = 0x7fc0;
              }
              stop = true;
              break;
            }
            __builtin_amdgcn_s_sleep(DSLEEP);
          }
        }
        if (stop) break;
        CFENCE();
        DBG(u64 td0 = now(); if (n == wid - 2u) STAMP(wsl, 1);)
        const u32 j1 = lds_ld(smem, L_JOB + (n % NJ) * 8u + 4u);
        const u32 size16 = (j1 >> 12) & 1023u, region = (j0 >> 8) & 7u;
        const LAS u8* ch = smem + region * 16384u + (j1 >> 22) * 16u;
        const bool dead = size16 == 0u;
        u32 kind, id;
        code_inv(j1 & 4095u, kind, id);
        bool late = false;
        auto landed = [&](u32 bytes) {
          const u32 lim = size16 * 16u, need = (j0 >> 16) + (((j1 >> 22) * 16u + (bytes < lim ? bytes : lim) + 1023u) >> 10);
          if (lds_ld(smem, L_LANDED) < need) {
            const u64 t0 = now();
            while (lds_ld(smem, L_LANDED) < need) {
              if (now() - t0 > 5 * DEADLINE) { late = true; break; }
              __builtin_amdgcn_s_sleep(DSLEEP);
            }
          }
          CFENCE();
          DBG(if (bytes == lim) td0 = now();)
        };
        u32 P[32];
        if (dead) {
          const u64 t0 = now();
          while (lds_ld(smem, L_LANDED) <= (j0 >> 16) && now() - t0 < 5 * DEADLINE) __builtin_amdgcn_s_sleep(DSLEEP);
        } else if (kind == 2) {
          decode_chunk<true>(ch, hs, lutx, size16, lane, P, landed);
        } else {
          decode_chunk<false>(ch, hs, lutx, size16, lane, P, landed);
        }
        if (dead || late) {
#pragma unroll
          for (int i = 0; i < 32; ++i) P[i] = 0x7fc07fc0u;
        }
#pragma unroll
        for (int i = 0; i < 32; ++i) asm volatile("" : "+v"(P[i]));
        WAIT_LGKM0();
        if (lane == 0) __hip_atomic_fetch_add((LAS u32*)(smem + L_FREE + region * 4u), 1u, __ATOMIC_RELAXED, __HIP_MEMORY_SCOPE_WORKGROUP);
        DBG(td[kind == 2 ? 2 : 0] += now() - td0;)
        if (kind == 2) {
          u32 si, sj;
          st_coord(id >> 7, si, sj);
          const u32 t = (id >> 3) & 15u, w = id & 7u;
          const u32 voff = (lane & 15u) * (NN * 2u) + (lane >> 4) * 16u;
          const u32 soff = ((si * 4u + (t >> 2)) * 256u + (w & 3u) * 64u) * (NN * 2u) + (src * NSH + (sj * 4u + (t & 3u)) * 128u + (w >> 2) * 64u) * 2u;
#pragma unroll
          for (int v = 0; v < 8; ++v) {
            const u32x4 q = {P[4 * v], P[4 * v + 1], P[4 * v + 2], P[4 * v + 3]};
            st16<OAUX>(q, ro, voff + (v & 1) * 64u, soff + (v >> 1) * 16u * (NN * 2u));
          }
        } else {
          u32 o;
          if (kind == 0) o = OFF_BT + (id >> 4) * (1u << 20) + (src * 16u + (id & 15u)) * 8192u;
          else o = OFF_AT + (id >> 5) * (2u << 20) + (src * 8u + ((id >> 2) & 7u)) * 32768u + (id & 3u) * 4096u;
          const u32 hstride = kind == 0 ? 4096u : 16384u;
#pragma unroll
          for (int v = 0; v < 8; ++v) {
            const u32x4 q = {P[4 * v], P[4 * v + 1], P[4 * v + 2], P[4 * v + 3]};
            st16<16>(q, ws, lane * 16u + (v & 3) * 1024u, o + (v >> 2) * hstride);
          }
          WAIT_VM(0);
          if (lane == 0)
            __hip_atomic_fetch_add((GAS u32*)(wsl + OFF_CNT + (32u + (kind == 0 ? 16u + (id >> 4) : (id >> 5))) * 128u), 1u, __ATOMIC_RELAXED, __HIP_MEMORY_SCOPE_SYSTEM);
        }
        DBG(td[kind == 2 ? 3 : 1] += now() - td0; if (n == wid - 2u) STAMP(wsl, 2);)
      }
      STAMP(wsl, 3);
      DBG(for (int i = 0; i < 4; ++i) STAMPV(wsl, 10 + i, td[i]);)
    }
  }
}

extern "C" void kernel_launch(int rank, int nranks, ncclComm_t comm, hipStream_t stream, void* const* d_in, const int* in_sizes,
                              int n_in, void* d_out, int out_size, const int* in_dtypes, int out_dtype, void* const* d_out_all,
                              void* const* d_ws_all, size_t ws_bytes) {
  (void)comm; (void)in_sizes; (void)n_in; (void)out_size; (void)in_dtypes; (void)out_dtype; (void)d_out_all;
  if (nranks != NR || ws_bytes < WS_NEED || (unsigned)rank >= (unsigned)NR) {
    fprintf(stderr, "unsupported configuration\n");
    return;
  }
  Params p;
  p.x = d_in[0];
  p.w = d_in[1];
  p.out = d_out;
  p.ws = d_ws_all[rank];
  for (int i = 0; i < NR; ++i) p.peer[i] = d_ws_all[(rank + 1 + i) % NR];
  p.rank = rank;
  p.pad = 0;
  gemm_ar_kernel<<<NWG, NTHR, 0, stream>>>(p);
}
```

```cpp
#include <hip/hip_runtime.h>
#include <rccl/rccl.h>
#include <cstdio>

typedef unsigned u32;
typedef unsigned long long u64;
typedef unsigned char u8;
typedef u32 u32x2 __attribute__((ext_vector_type(2)));
typedef u32 u32x4 __attribute__((ext_vector_type(4)));
typedef float f32x2 __attribute__((ext_vector_type(2)));
typedef float f32x4 __attribute__((ext_vector_type(4)));
typedef short s16x2 __attribute__((ext_vector_type(2)));
typedef short s16x4 __attribute__((ext_vector_type(4)));
typedef __bf16 bf16x2 __attribute__((ext_vector_type(2)));
typedef __bf16 bf16x8 __attribute__((ext_vector_type(8)));
#define LAS __attribute__((address_space(3)))
#define GAS __attribute__((address_space(1)))
typedef __amdgpu_buffer_rsrc_t rsrc_t;

namespace {

constexpr int NR = 8;
constexpr int MM = 4096, NN = 16384, KSH = 512, NSH = 2048;
constexpr int NWG = 256, NTHR = 512;
#define NLOAD 18
constexpr int NL = NLOAD;
#define NFASTG 16
constexpr u32 NF = NFASTG;
constexpr u32 NFU = 4 * NF;
constexpr int NGRP = 32 + 96 + 256;
#define TAILT 8
#define TAILC 2
#define NPUSHG 4
constexpr u32 NP = NPUSHG;
#define PQUAD 16
#define OCT0 96
#define OCT1 220
constexpr u32 TK_B = 4 * PQUAD, NTI = TK_B + 2 * (96 - NP - PQUAD), TK_D = NTI + 2 * OCT0, TK_E = TK_D + (OCT1 - OCT0);
constexpr u32 TK_F = TK_E + 2 * (256 - TAILT - OCT1);
constexpr int NTK = TK_F + (8 / TAILC) * TAILT;
static_assert(PQUAD >= NF && PQUAD <= 96 - NP && OCT0 <= OCT1 && OCT1 <= 256 - TAILT, "units");
constexpr u32 NTASK = 320 - NF - 7 * NP;
#define SDELAY 8
#define SSTAG 8
constexpr int NREG = 4, REGK = 25, NDEC = 6, NQ = 32, NJ = 128;
#define WINDOW 3
#define THRESH 6
#define OAUX 16
#define PSLOW 127
#define PFAST 16
#define DSLEEP 3
#define PWIN 1
#define FSLEEP 8
#define FIRSTSLEEP 48
#define TWIN 6
#define NWT 0
#define GV 2
constexpr u64 DEADLINE = 30000000ull;

constexpr u32 OFF_CALL = 0;
constexpr u32 OFF_TMO = 16384;
constexpr u32 OFF_TK = 17408;
constexpr u32 OFF_CNT = 20480;
constexpr u32 OFF_FLAG = 32768;
constexpr u32 OFF_DBG = 512u << 10;
constexpr u32 OFF_EX = 1u << 20;
constexpr u32 OFF_EW = 5u << 20;
constexpr u32 OFF_EO = 21u << 20;
constexpr u32 OFF_AT = 37u << 20;
constexpr u32 OFF_BT = 69u << 20;
constexpr u32 WS_NEED = 85u << 20;

constexpr int L_STAGE = 49152;
constexpr int L_HS = NREG * REGK * 1024 + 512;
constexpr int L_HS_STRIDE = 4160;
constexpr int L_MISC = 3 * L_STAGE;
constexpr int L_LUT = L_MISC;
constexpr int L_Z0 = L_MISC + 128;
constexpr int L_REQ = L_Z0;
constexpr int L_JOB = L_REQ + 8 * NQ;
constexpr int L_FREE = L_JOB + 8 * NJ;
constexpr int L_LANDED = L_FREE + 32;
constexpr int L_REND = L_LANDED + 4;
constexpr int L_JEND = L_LANDED + 8;
constexpr int L_CALL = L_LANDED + 16;
constexpr int L_SZ = L_LANDED + 32;
constexpr int L_ARR = L_SZ + 192;
constexpr int L_GST = L_ARR + 32;
constexpr int L_Z1 = L_GST + 16;
constexpr int L_EXC = L_MISC + 2048;
constexpr int L_TOTAL = L_EXC + 8 * 256;
static_assert(L_HS + NDEC * L_HS_STRIDE <= L_MISC, "lds");
static_assert(L_Z1 <= L_EXC && (L_Z1 - L_Z0) / 4 <= NTHR, "lds");
static_assert(L_TOTAL <= 163840, "lds");

constexpr u32 HEAD_OUT = 1024, HEAD_IN = 1536;
constexpr u32 HD_OFF = 512, UN_OFF = 516;

struct Params {
  const void* x;
  const void* w;
  void* out;
  void* ws;
  int rank;
  int pad;
  void* peer[NR];
};

#define CFENCE() asm volatile("" ::: "memory")
#define WAIT_VM(n) asm volatile("s_waitcnt vmcnt(" #n ")" ::: "memory")
#define WAIT_LGKM0() asm volatile("s_waitcnt lgkmcnt(0)" ::: "memory")
#define SB() __builtin_amdgcn_sched_barrier(0)
#define STR2(x) #x
#define STR(x) STR2(x)

__device__ __forceinline__ u32 rfl(u32 v) { return __builtin_amdgcn_readfirstlane(v); }
__device__ __forceinline__ u32 vperm(u32 hi, u32 lo, u32 sel) { return __builtin_amdgcn_perm(hi, lo, sel); }
__device__ __forceinline__ u32 bsum(u32 v, u32 acc) { return __builtin_amdgcn_sad_u8(v, 0u, acc); }
__device__ __forceinline__ u64 now() { return __builtin_amdgcn_s_memrealtime(); }
__device__ __forceinline__ u32 mbcnt(u64 m) {
  return __builtin_amdgcn_mbcnt_hi((u32)(m >> 32), __builtin_amdgcn_mbcnt_lo((u32)m, 0u));
}
__device__ __forceinline__ u32 scan_incl(u32 x) {
  x += __builtin_amdgcn_update_dpp(0u, x, 0x111, 0xf, 0xf, true);
  x += __builtin_amdgcn_update_dpp(0u, x, 0x112, 0xf, 0xf, true);
  x += __builtin_amdgcn_update_dpp(0u, x, 0x114, 0xf, 0xf, true);
  x += __builtin_amdgcn_update_dpp(0u, x, 0x118, 0xf, 0xf, true);
  x += __builtin_amdgcn_update_dpp(0u, x, 0x142, 0xa, 0xf, false);
  x += __builtin_amdgcn_update_dpp(0u, x, 0x143, 0xc, 0xf, false);
  return x;
}
__device__ __forceinline__ rsrc_t make_rsrc(const void* p, u32 bytes) {
  return __builtin_amdgcn_make_buffer_rsrc((void*)p, 0, bytes, 0x00020000);
}
#define GSC " sc0 sc1"
template <int IMM>
__device__ __forceinline__ void dma16(rsrc_t r, u32 lds, u32 voff, u32 soff) {
  asm volatile("s_mov_b32 m0, %0\n\ts_nop 0\n\tbuffer_load_dwordx4 %1, %2, %3 offen offset:%4" GSC " lds"
               :: "s"(lds), "v"(voff), "s"(r), "s"(soff), "i"(IMM) : "memory");
}
template <int AUX>
__device__ __forceinline__ void st16(u32x4 v, rsrc_t r, u32 voff, u32 soff) {
  __builtin_amdgcn_raw_buffer_store_b128(v, r, voff, soff, AUX);
  SB();
  asm volatile("s_nop 1");
  SB();
}
__device__ __forceinline__ u32 lds_ld(LAS u8* lds, int off) {
  return rfl(*(volatile LAS u32*)(lds + off));
}
__device__ __forceinline__ void lds_st(LAS u8* lds, int off, u32 v) { *(volatile LAS u32*)(lds + off) = v; }

#define STAMP(ws, slot) do {} while (0)
#define STAMPV(ws, slot, v) do {} while (0)
#define DBG(x)

__device__ __forceinline__ u64 rank_table(u32 c) {
  const u32 k = c & 7u;
  const u64 below = (1ull << (8 * k)) - 1ull;
  const u64 t = (0x0001020304050607ull & ~below & ~(0xffull << (8 * k))) | (7ull << (8 * k)) | (0x0000010203040506ull & below);
  return (c & 8u) ? (t + 0x0707070707070707ull) & 0x0707070707070707ull : t;
}
__device__ __forceinline__ u64 len_table(u32 c) {
  const u32 k = c & 7u;
  const u64 t = (0x0001020304050607ull + ((0x0101010101010101ull >> (8 * k)) >> 8)) | ((u64)(k + 1) << 56);
  return (c & 8u) ? (t >> 8) | (t << 56) : t;
}
__device__ __forceinline__ u32 top_mask(u32 v) { return vperm(v << 8, v, 0x090b080au); }
__device__ __forceinline__ u32 bsel(u32 m, u32 a, u32 b) { return (m & a) | (~m & b); }
__device__ __forceinline__ u32 unit_offset(LAS u8* szw, u32 wid, u32 U) {
  u32 off = 0;
  for (u32 j = wid & ~(U - 1u); j < wid; ++j) {
    u32 v = lds_ld(szw, j * 4u);
    for (int spin = 0; v == 0u && spin < (1 << 22); ++spin) v = lds_ld(szw, j * 4u);
    off += v;
  }
  return off;
}
template <bool OUT>
__device__ __forceinline__ void encode_chunk(const u32 (&P)[32], LAS u8* bld, LAS u32* exs, const LAS u32* lutc, char* wsl,
                                             u32 gbase, LAS u8* szw, u32 wid, u32 lane, u32 U) {
  const u32 ub = gbase + (wid & ~(U - 1u)) * 8192u;
  u32 lo[16], len[16], nzm[16];
  u32 pl[2] = {0u, 0u};
  u32 size4 = 2048u, nexc = 0, nw = 0, lbase = 0, pbase = 0, qbase = 0, header = 0;
  {
    u32 hi[16];
    u32 sum = 0, sgn = 0;
#pragma unroll
    for (int g = 0; g < 16; ++g) {
      lo[g] = vperm(P[2 * g + 1], P[2 * g], 0x06040200u);
      hi[g] = vperm(P[2 * g + 1], P[2 * g], 0x07050301u);
      if (OUT) {
        const s16x2 z = {0, 0};
        const u32 n0 = __builtin_bit_cast(u32, z - __builtin_bit_cast(s16x2, P[2 * g]));
        const u32 n1 = __builtin_bit_cast(u32, z - __builtin_bit_cast(s16x2, P[2 * g + 1]));
        nzm[g] = vperm(n1, n0, 0x0b0a0908u);
        pl[g >> 3] |= bsum(nzm[g] & 0x08040201u, 0u) << (4 * (g & 7));
        sgn |= hi[g];
      } else {
        pl[g >> 3] |= (hi[g] >> (7 - (g & 7))) & (0x01010101u << (g & 7));
        hi[g] &= 0x7f7f7f7fu;
        nzm[g] = 0xffffffffu;
      }
      if (g & 1) sum = bsum(hi[g] + hi[g - 1], sum);
    }
    const bool signed_out = OUT && __builtin_amdgcn_ballot_w64((sgn & 0x80808080u) != 0u) != 0ull;
    const u32 cnt = OUT ? (u32)(__builtin_popcount(pl[0]) + __builtin_popcount(pl[1])) : 64u;
    const u32 sc0 = scan_incl((sum & 0x1fffu) | (cnt << 19));
    const u32 tot0 = __builtin_amdgcn_readlane(sc0, 63);
    const u32 nnz = tot0 >> 19;
    pbase = (sc0 >> 19) - cnt;
    const float mean = (float)(tot0 & 0x7ffffu) * __builtin_amdgcn_rcpf((float)(nnz ? nnz : 1u));
    int b = (int)(mean + 0.97f);
    const float fr = mean - (float)(b - 1);
    const u32 c0 = rfl((u32)(fr < 0.08f) + (fr < 0.18f) + (fr < 0.225f) + (fr < 0.32f) + (fr < 0.46f) + (fr < 0.655f) + (fr < 0.92f));
    const u32 c1 = rfl(2u + (fr < 0.975f) + (fr < 0.815f) + (fr < 0.71f) + (fr < 0.655f) + (fr < 0.555f) + 2u * (fr < 0.41f) + (fr < 0.18f));
    b = b < 6 ? 6 : b > 126 ? 126 : b;
    const u32 base = rfl((u32)(b - 6));
    const u32 cneg = 0x80808080u - base * 0x01010101u;
    const u64 inv0 = len_table(c0), inv1 = len_table(c1);
    u32 inv0_lo = (u32)inv0, inv1_lo = (u32)inv1;
    asm volatile("" : "+v"(inv0_lo), "+v"(inv1_lo));
    u32 anybad = 0;
#pragma unroll
    for (int g = 0; g < 16; ++g) {
      const u32 e4 = (hi[g] + cneg) ^ 0x80808080u;
      anybad |= e4 & nzm[g];
      len[g] = bsel(top_mask(lo[g]), vperm((u32)(inv1 >> 32), inv1_lo, e4), vperm((u32)(inv0 >> 32), inv0_lo, e4)) & nzm[g];
      if (OUT) lo[g] = vperm(0u, lo[g], lutc[(pl[g >> 3] >> (4 * (g & 7))) & 15u]);
    }
    if (__builtin_amdgcn_ballot_w64((anybad & 0xf8f8f8f8u) != 0u) != 0ull) {
      u32 bp[2] = {0u, 0u};
#pragma unroll
      for (int g = 0; g < 16; ++g) {
        const u32 bad = ((hi[g] + cneg) ^ 0x80808080u) & 0xf8f8f8f8u & nzm[g];
        const u32 t = ((bad >> 1) + 0x7c7c7c7cu) & 0x80808080u;
        const u32 fm = (t << 1) - (t >> 7);
        len[g] = (len[g] & ~fm) | (t >> 7);
        bp[g >> 3] |= bsum(fm & 0x08040201u, 0u) << (4 * (g & 7));
      }
#pragma unroll
      for (int j = 0; j < 4; ++j) {
        const u32x4 t = {hi[4 * j], hi[4 * j + 1], hi[4 * j + 2], hi[4 * j + 3]};
        *(LAS u32x4*)(bld + lane * 64u + j * 16) = t;
      }
      CFENCE();
      u64 bm = ((u64)bp[1] << 32) | bp[0];
      const u64 plm = ((u64)pl[1] << 32) | pl[0];
      for (;;) {
        const u64 m = __builtin_amdgcn_ballot_w64(bm != 0ull);
        if (m == 0ull || nexc > 63u) break;
        if (bm != 0ull) {
          const u32 j = __builtin_ctzll(bm);
          bm &= bm - 1ull;
          const u32 slot = nexc + mbcnt(m);
          const u32 idx = pbase + (OUT ? (u32)__builtin_popcountll(plm & ((1ull << j) - 1ull)) : j);
          if (slot < 64u) exs[slot] = (idx << 8) | bld[lane * 64u + j];
        }
        nexc += __builtin_popcountll(m);
      }
      CFENCE();
    }
    u32 acc = 0;
#pragma unroll
    for (int g = 0; g < 16; ++g) acc += len[g];
    const u32 bits = bsum(acc, 0u);
    const u32 sc1 = scan_incl(bits);
    qbase = sc1 - bits;
    nw = (__builtin_amdgcn_readlane(sc1, 63) + 31u) >> 5;
    lbase = UN_OFF + 4u * nw + 4u * nexc;
    size4 = (lbase + nnz + 3u) >> 2;
    if (size4 >= 2048u || nexc > 63u || signed_out) size4 = 2048u;
    header = base | (c0 << 7) | (c1 << 11) | (nw << 15) | (nexc << 26);
  }
  lds_st(szw, wid * 4u, size4);
  if (size4 == 2048u) {
    const u32 go = ub + unit_offset(szw, wid, U) * 4u;
    const rsrc_t ws = make_rsrc(wsl, WS_NEED);
#pragma unroll
    for (int v = 0; v < 8; ++v) {
      const u32x4 q = {P[4 * v], P[4 * v + 1], P[4 * v + 2], P[4 * v + 3]};
      st16<17>(q, ws, lane * 16u + v * 1024u, go);
    }
    return;
  }
  {
    const u32x4 z = {0u, 0u, 0u, 0u};
    for (u32 o = HD_OFF; o < (OUT ? size4 * 4u : lbase); o += 1024u) *(LAS u32x4*)(bld + o + lane * 16u) = z;
  }
  CFENCE();
  u32 q = qbase + ((u32)(size_t)bld + UN_OFF) * 8u;
  u32 p = pbase + (u32)(size_t)bld + lbase;
  u32 ones = 0x01010101u;
  asm volatile("" : "+v"(ones));
#pragma unroll
  for (int g0 = 0; g0 < 16; g0 += 4) {
    u64 v[4], u[4];
    u32 da[4], ea[4];
#pragma unroll
    for (int i = 0; i < 4; ++i) {
      const int g = g0 + i;
      u32 t = len[g] + (len[g] << 8);
      asm volatile("" : "+v"(t));
      t += t << 16;
      const u32 tb = t + 0x1f1f1f1fu;
      const u32 one = OUT ? (nzm[g] & 0x01010101u) : ones;
      u32 b4, b5, b6;
      asm("v_lshlrev_b32_sdwa %0, %3, %4 dst_sel:DWORD dst_unused:UNUSED_PAD src0_sel:BYTE_0 src1_sel:BYTE_0\n\t"
          "v_lshlrev_b32_sdwa %1, %3, %4 dst_sel:DWORD dst_unused:UNUSED_PAD src0_sel:BYTE_1 src1_sel:BYTE_1\n\t"
          "v_lshlrev_b32_sdwa %2, %3, %4 dst_sel:DWORD dst_unused:UNUSED_PAD src0_sel:BYTE_2 src1_sel:BYTE_2\n\t"
          "v_or3_b32 %0, %0, %1, %2\n\t"
          "v_lshlrev_b32_sdwa %1, %3, %4 dst_sel:DWORD dst_unused:UNUSED_PAD src0_sel:BYTE_3 src1_sel:BYTE_3\n\t"
          "v_or_b32 %0, %0, %1"
          : "=&v"(b4), "=&v"(b5), "=&v"(b6) : "v"(tb), "v"(one));
      v[i] = (u64)b4 << (q & 31u);
      da[i] = (q >> 3) & ~3u;
      q += t >> 24;
      if (OUT) {
        u[i] = (u64)lo[g] << ((p & 3u) * 8u);
        ea[i] = p & ~3u;
        p += __builtin_popcount(one);
      }
    }
    SB();
#pragma unroll
    for (int i = 0; i < 4; ++i) {
      LAS u32* d = (LAS u32*)(size_t)da[i];
      __hip_atomic_fetch_or(d, (u32)v[i], __ATOMIC_RELAXED, __HIP_MEMORY_SCOPE_WAVEFRONT);
      __hip_atomic_fetch_or(d + 1, (u32)(v[i] >> 32), __ATOMIC_RELAXED, __HIP_MEMORY_SCOPE_WAVEFRONT);
      if (OUT) {
        LAS u32* e = (LAS u32*)(size_t)ea[i];
        __hip_atomic_fetch_or(e, (u32)u[i], __ATOMIC_RELAXED, __HIP_MEMORY_SCOPE_WAVEFRONT);
        __hip_atomic_fetch_or(e + 1, (u32)(u[i] >> 32), __ATOMIC_RELAXED, __HIP_MEMORY_SCOPE_WAVEFRONT);
      }
    }
    SB();
  }
  if (!OUT) {
#pragma unroll
    for (int g = 0; g < 16; ++g) *(LAS u32*)(bld + lbase + g * 256 + lane * 4u) = lo[g];
  }
  *(LAS u32*)(bld + lane * 4u) = pl[0];
  *(LAS u32*)(bld + 256 + lane * 4u) = pl[1];
  CFENCE();
  *(LAS u32*)(bld + HD_OFF) = header;
  if (lane < nexc) *(LAS u32*)(bld + UN_OFF + 4u * nw + 4u * lane) = exs[lane];
  CFENCE();
  const u32 go = ub + unit_offset(szw, wid, U) * 4u;
  u32 lim = go + size4 * 4u;
  const rsrc_t wc = make_rsrc(wsl, lim < WS_NEED ? lim : WS_NEED);
  constexpr int NB = OUT ? 3 : 6;
  for (u32 o = 0; o < size4 * 4u; o += 1024u * NB) {
    u32x4 t[NB];
#pragma unroll
    for (int j = 0; j < NB; ++j) t[j] = *(const LAS u32x4*)(bld + ((o + j * 1024u) & 8191u) + lane * 16u);
#pragma unroll
    for (int j = 0; j < NB; ++j) st16<17>(t[j], wc, lane * 16u + j * 1024u, go + o);
  }
  CFENCE();
}

__device__ __forceinline__ u32 parse4(u32& w, u32& q) {
  u32 r, p, t;
  asm("v_ffbl_b32 %3, %1\n\tv_add_u32 %4, -1, %1\n\tv_sub_u32 %0, %3, %2\n\tv_and_b32 %1, %4, %1\n\t"
      "v_ffbl_b32 %2, %1\n\tv_add_u32 %4, -1, %1\n\t"
      "v_sub_u32_sdwa %0, %2, %3 dst_sel:BYTE_1 dst_unused:UNUSED_PRESERVE src0_sel:DWORD src1_sel:DWORD\n\tv_and_b32 %1, %4, %1\n\t"
      "v_ffbl_b32 %3, %1\n\tv_add_u32 %4, -1, %1\n\t"
      "v_sub_u32_sdwa %0, %3, %2 dst_sel:BYTE_2 dst_unused:UNUSED_PRESERVE src0_sel:DWORD src1_sel:DWORD\n\tv_and_b32 %1, %4, %1\n\t"
      "v_ffbl_b32 %2, %1\n\tv_add_u32 %4, -1, %1\n\t"
      "v_sub_u32_sdwa %0, %2, %3 dst_sel:BYTE_3 dst_unused:UNUSED_PRESERVE src0_sel:DWORD src1_sel:DWORD\n\tv_and_b32 %1, %4, %1"
      : "=&v"(r), "+v"(w), "+v"(q), "=&v"(p), "=&v"(t));
  return r - 0x01010101u;
}

template <bool OUT, class LF>
__device__ __forceinline__ void decode_chunk(const LAS u8* ch, LAS u8* hs, const LAS u32x2* lutx, bool raw, u32 lane,
                                             u32 (&P)[32], LF landed) {
  if (raw) {
    landed(8192u);
#pragma unroll
    for (int v = 0; v < 8; ++v) {
      const volatile LAS u32* t = (const volatile LAS u32*)(ch + v * 1024 + lane * 16u);
      P[4 * v] = t[0]; P[4 * v + 1] = t[1]; P[4 * v + 2] = t[2]; P[4 * v + 3] = t[3];
    }
    return;
  }
  landed(UN_OFF);
  const u32 tr = rfl(*(const volatile LAS u32*)(ch + HD_OFF));
  const u32x2 pl = {*(const LAS u32*)(ch + lane * 4u), *(const LAS u32*)(ch + 256 + lane * 4u)};
  const u32 nw = (tr >> 15) & 0x7ffu, nexc = tr >> 26;
  const u64 dec0 = rank_table((tr >> 7) & 15u), dec1 = rank_table((tr >> 11) & 15u);
  const u32 base4 = (tr & 127u) * 0x01010101u;
  u32 t0_lo = (u32)dec0 + base4, t1_lo = (u32)dec1 + base4;
  const u32 t0_hi = (u32)(dec0 >> 32) + base4, t1_hi = (u32)(dec1 >> 32) + base4;
  asm volatile("" : "+v"(t0_lo), "+v"(t1_lo));
  u32 p = 0;
  if (OUT) {
    const u32 c = __builtin_popcount(pl.x) + __builtin_popcount(pl.y);
    p = scan_incl(c) - c;
  }
  u32 lbase = UN_OFF + 4u * nw + 4u * nexc;
  landed(lbase);
  lbase &= 8191u;
  const LAS u32* U = (const LAS u32*)(ch + UN_OFF);
  u32 run = 0;
  for (u32 w0 = 0; w0 < nw; w0 += 64u) {
    const u32 i = w0 + lane;
    u32 w = U[i], prev = U[(int)i - 1];
    w = i < nw ? w : 0u;
    prev = i > 0u ? prev : 0x80000000u;
    const u32 c = __builtin_popcount(w);
    const u32 incl = scan_incl(c);
    const u32 dst = (u32)(size_t)hs + ((run + incl - c) & 4095u);
    run += __builtin_amdgcn_readlane(incl, 63);
    u32 q = ~(u32)__builtin_clz(prev | 1u);
    u32 R[8];
#pragma unroll
    for (int j = 0; j < 6; ++j) R[j] = parse4(w, q);
    R[6] = R[7] = 0u;
    if (__builtin_amdgcn_ballot_w64(w != 0u) != 0ull) {
      R[6] = parse4(w, q);
      if (__builtin_amdgcn_ballot_w64(w != 0u) != 0ull) R[7] = parse4(w, q);
    }
    const u32 wb = (dst - 1u) & ~3u, ca = c + dst - wb, sh = 0u - dst;
    u32 W[8];
    u64 m[8];
#pragma unroll
    for (int j = 0; j < 8; ++j) {
      W[j] = __builtin_amdgcn_alignbyte(j < 7 ? R[j + 1] : 0u, R[j], sh);
      m[j] = __builtin_amdgcn_ballot_w64(4u * (j + 1) < ca);
    }
    const u64 m0 = __builtin_amdgcn_ballot_w64(c != 0u);
    const u32 r1 = R[0] >> 8;
    asm volatile("s_mov_b64 exec, %9\n\tds_write_b32 %0, %1 offset:4\n\ts_mov_b64 exec, %10\n\tds_write_b32 %0, %2 offset:8\n\t"
                 "s_mov_b64 exec, %11\n\tds_write_b32 %0, %3 offset:12\n\ts_mov_b64 exec, %12\n\tds_write_b32 %0, %4 offset:16\n\t"
                 "s_mov_b64 exec, %13\n\tds_write_b32 %0, %5 offset:20\n\ts_mov_b64 exec, %14\n\tds_write_b32 %0, %6 offset:24\n\t"
                 "s_mov_b64 exec, %15\n\tds_write_b32 %0, %7 offset:28\n\ts_mov_b64 exec, %16\n\tds_write_b32 %0, %8 offset:32\n\t"
                 "s_mov_b64 exec, %17\n\tds_write_b8 %18, %19\n\tds_write_b8 %18, %20 offset:1\n\tds_write_b8_d16_hi %18, %19 offset:2\n\t"
                 "s_mov_b64 exec, -1"
                 :: "v"(wb), "v"(W[0]), "v"(W[1]), "v"(W[2]), "v"(W[3]), "v"(W[4]), "v"(W[5]), "v"(W[6]), "v"(W[7]),
                    "s"(m[0]), "s"(m[1]), "s"(m[2]), "s"(m[3]), "s"(m[4]), "s"(m[5]), "s"(m[6]), "s"(m[7]), "s"(m0),
                    "v"(dst), "v"(R[0]), "v"(r1) : "memory");
  }
  CFENCE();
  if (lane < nexc) {
    const u32 r = U[nw + lane];
    hs[(r >> 8) & 4095u] = (u8)(r | 0x80u);
  }
  CFENCE();
  u32 s4[16], pp[16];
  u32x2 sel[16];
  if (OUT) {
#pragma unroll
    for (int g0 = 0; g0 < 16; g0 += 8) {
      u32x2 H[8];
      const LAS u8 *sa[8], *ha[8];
#pragma unroll
      for (int i = 0; i < 8; ++i) {
        const u32 wd = g0 ? pl.y : pl.x;
        const u32 m8 = (i == 0 ? wd << 3 : wd >> (4 * i - 3)) & 0x78u;
        pp[g0 + i] = p;
        sa[i] = (const LAS u8*)lutx + m8;
        ha[i] = hs + (p & ~3u);
        p += __builtin_popcount(m8);
      }
      SB();
#pragma unroll
      for (int i = 0; i < 8; ++i) {
        sel[g0 + i] = *(const LAS u32x2*)sa[i];
        H[i].x = ((const LAS u32*)ha[i])[0]; H[i].y = ((const LAS u32*)ha[i])[1];
      }
      SB();
#pragma unroll
      for (int i = 0; i < 8; ++i) s4[g0 + i] = __builtin_amdgcn_alignbyte(H[i].y, H[i].x, pp[g0 + i]);
    }
  } else {
#pragma unroll
    for (int j = 0; j < 4; ++j) {
      const u32x4 H = *(const LAS u32x4*)(hs + lane * 64u + j * 16);
      s4[4 * j] = H.x; s4[4 * j + 1] = H.y; s4[4 * j + 2] = H.z; s4[4 * j + 3] = H.w;
    }
  }
  u32 h0[16], h1[16];
#pragma unroll
  for (int g = 0; g < 16; ++g) { h0[g] = vperm(t0_hi, t0_lo, s4[g]); h1[g] = vperm(t1_hi, t1_lo, s4[g]); }
  if (nexc) {
#pragma unroll
    for (int g = 0; g < 16; ++g) {
      const u32 xm = top_mask(s4[g]), xv = s4[g] & 0x7f7f7f7fu;
      h0[g] = bsel(xm, xv, h0[g]);
      h1[g] = bsel(xm, xv, h1[g]);
    }
  }
  if (OUT) {
    u32 la[16];
#pragma unroll
    for (int g = 0; g < 16; ++g) {
      la[g] = (u32)(size_t)ch + lbase + (pp[g] & ~3u);
      asm volatile("" : "+v"(la[g]), "+v"(h0[g]), "+v"(h1[g]), "+v"(sel[g]));
    }
    landed(8192u);
    u32x2 L[16];
#pragma unroll
    for (int g = 0; g < 16; ++g) { L[g].x = ((const LAS u32*)(size_t)la[g])[0]; L[g].y = ((const LAS u32*)(size_t)la[g])[1]; }
    SB();
#pragma unroll
    for (int g = 0; g < 16; ++g) {
      const u32 l4 = __builtin_amdgcn_alignbyte(L[g].y, L[g].x, pp[g]);
      const u32 h4 = bsel(top_mask(l4), h1[g], h0[g]);
      P[2 * g] = vperm(h4, l4, sel[g].x);
      P[2 * g + 1] = vperm(h4, l4, sel[g].y);
    }
  } else {
#pragma unroll
    for (int g = 0; g < 16; ++g) {
      const u32 sg = ((g < 8 ? pl.x : pl.y) << (7 - (g & 7))) & 0x80808080u;
      h0[g] |= sg;
      h1[g] |= sg;
      asm volatile("" : "+v"(h0[g]), "+v"(h1[g]));
    }
    landed(8192u);
    u32 L[16];
#pragma unroll
    for (int g = 0; g < 16; ++g) L[g] = *(const LAS u32*)(ch + lbase + g * 256 + lane * 4u);
    SB();
#pragma unroll
    for (int g = 0; g < 16; ++g) {
      const u32 h4 = bsel(top_mask(L[g]), h1[g], h0[g]);
      P[2 * g] = vperm(h4, L[g], 0x05010400u);
      P[2 * g + 1] = vperm(h4, L[g], 0x07030602u);
    }
  }
}

__device__ __forceinline__ u32 pos_x(u32 g) { return g + (g < NF ? 0u : g < 16u ? 8u - NP : g < 32u ? 16u - NP : 32u - NP); }
__device__ __forceinline__ u32 pos_w(u32 g) { return g - NP + (g < 8u ? NF : g < 16u ? 16u : 32u); }
__device__ __forceinline__ u32 pos_code(u32 p) {
  if (p >= 96u) return 768u + (p - 96u) * 8u;
  if (p < NF) return 256u + p * 8u;
  if (p < NF + 8u - NP) return (p + NP - NF) * 8u;
  if (p < 24u - NP) return 256u + (p + NP - 8u) * 8u;
  if (p < 32u - NP) return (p + NP - 16u) * 8u;
  if (p < 48u - NP) return 256u + (p + NP - 16u) * 8u;
  if (p < 64u - NP) return (p + NP - 32u) * 8u;
  return 256u + (p + NP - 32u) * 8u;
}
__device__ __forceinline__ void code_inv(u32 e, u32& kind, u32& id) {
  kind = e < 256u ? 0u : e < 768u ? 1u : 2u;
  id = e < 256u ? e : e < 768u ? e - 256u : e - 768u;
}
__device__ __forceinline__ void w_stage(rsrc_t rw, LAS u8* stg, u32 dest, u32 id, u32 lane) {
  const u32 cb = id >> 4, kb = id & 15u;
  const u32 voff = (lane >> 4) * (NN * 2) + (lane & 15u) * 16u;
  u32 soff = rfl(kb * 32u * (NN * 2) + (dest * NSH + cb * 128u) * 2u), lds = rfl((u32)(size_t)stg);
  asm volatile("s_nop 4" : "+s"(soff), "+s"(lds));
#pragma unroll
  for (int u = 0; u < 8; ++u) dma16<0>(rw, lds + u * 1024u, voff, soff + u * 4u * (NN * 2));
}
__device__ __forceinline__ void w_fragments(const LAS u8* stg, u32 lane, u32 (&P)[32]) {
  const LAS u8* tb = stg + ((lane >> 4) * 8u + ((lane & 15u) >> 2)) * 256u + (lane & 3u) * 16u;
#pragma unroll
  for (int nf = 0; nf < 8; ++nf)
#pragma unroll
    for (int h = 0; h < 2; ++h) {
      const s16x4 r = __builtin_amdgcn_ds_read_tr16_b64_v4i16((LAS s16x4*)(tb + h * 1024 + (nf >> 1) * 64 + (nf & 1) * 8));
      const u32x2 r2 = __builtin_bit_cast(u32x2, r);
      P[4 * nf + 2 * h] = r2.x;
      P[4 * nf + 2 * h + 1] = r2.y;
    }
}
__device__ __forceinline__ void x_stage(rsrc_t rx, LAS u8* stg, u32 id, u32 lane) {
  const u32 rb = id >> 5, ksl = (id >> 2) & 7u, mq = id & 3u;
  const u32 voff = (lane >> 3) * (KSH * 2) + (lane & 7u) * 16u;
  u32 soff = rfl((rb * 256u + mq * 64u) * (KSH * 2) + ksl * 128u), lds = rfl((u32)(size_t)stg);
  asm volatile("s_nop 4" : "+s"(soff), "+s"(lds));
#pragma unroll
  for (int u = 0; u < 8; ++u) dma16<0>(rx, lds + u * 1024u, voff, soff + u * 8u * (KSH * 2));
}
__device__ __forceinline__ void x_fragments(const LAS u8* stg, u32 lane, u32 (&P)[32]) {
  const LAS u8* tb = stg + (lane & 15u) * 128u + (lane >> 4) * 16u;
#pragma unroll
  for (int v = 0; v < 8; ++v) {
    const u32x4 r = *(const LAS u32x4*)(tb + (v & 3) * 2048 + (v >> 2) * 64);
    P[4 * v] = r.x; P[4 * v + 1] = r.y; P[4 * v + 2] = r.z; P[4 * v + 3] = r.w;
  }
}
__device__ __forceinline__ bool task_of(u32 rnd, u32 gi, u32 wid, u32 rank, u32& isw, u32& gid, u32& dest, u32& cw) {
  const u32 x = gi & 7u;
  const u32 t = rnd < 2u ? gi : rnd == 2u ? 128u + (gi >> 3) * 7u + (x - 1u) : 240u + (gi >> 3) * 4u + (x - 4u);
  isw = 1; gid = 0; dest = rank; cw = wid;
  static_assert(NTASK >= 240 && NTASK <= 304, "tasks");
  if (rnd > 3u || (rnd == 2u && x == 0u) || (rnd == 3u && (x < 4u || t >= NTASK))) return false;
  if (rnd == 0u) { isw = 0; gid = (gi - (128u - NFU)) >> 2; cw = (gi & 3u) * 2u + wid; }
  else if (t < NP) { gid = t; }
  else {
    const u32 u = t + 7u * NP + NF;
    if (u < 64u + NF) gid = u - NF;
    else if (u < 80u) { isw = 0; gid = u - 64u; }
    else if (u < 144u) gid = u - 16u;
    else if (u < 160u) { isw = 0; gid = u - 128u; }
    else if (u < 288u) gid = u - 32u;
    else { isw = 0; gid = u - 256u; }
    if (isw) { dest = gid & 7u; gid >>= 3; }
  }
  return true;
}
__device__ __forceinline__ void st_coord(u32 seq, u32& i, u32& j) {
  if (seq >= 8u) { i = seq >> 2; j = seq & 3u; }
  else if (seq < 4u) { i = seq >> 1; j = seq & 1u; }
  else { i = seq & 1u; j = seq >> 1; }
}
__device__ __forceinline__ u32 tile_seq(u32 seq, u32 pos) {
  const u32 r = pos >> 2, c = pos & 3u, x = seq & 7u, b = (seq >> 3) * 128u;
  if (x == 0u) return b + (c >> 1) * 8u + r * 2u + (c & 1u);
  if (x == 1u) return b + 16u + c * 4u + r;
  if (x < 4u) return b + 32u + r * 8u + (x & 1u) * 4u + c;
  return b + (x >> 1) * 32u + c * 8u + (x & 1u) * 4u + r;
}
__device__ __forceinline__ void tile_inv(u32 ts, u32& rb, u32& cb) {
  const u32 t = ts & 127u;
  u32 seq, r, c;
  if (t < 16u) { seq = 0u; r = (t >> 1) & 3u; c = (t >> 3) * 2u + (t & 1u); }
  else if (t < 32u) { seq = 1u; c = (t >> 2) & 3u; r = t & 3u; }
  else if (t < 64u) { seq = 2u + ((t >> 2) & 1u); r = (t >> 3) & 3u; c = t & 3u; }
  else { seq = (t >> 5) * 2u + ((t >> 2) & 1u); c = (t >> 3) & 3u; r = t & 3u; }
  seq += (ts >> 7) * 8u;
  u32 si, sj;
  st_coord(seq, si, sj);
  rb = si * 4u + r;
  cb = sj * 4u + c;
}
__device__ __forceinline__ u32 flag_off(u32 src, u32 g) { return OFF_FLAG + (src * NGRP + g) * 16u; }
__device__ __forceinline__ void publish(LAS u8* smem, u32 rnd, char* pws, u32 rank, u32 g, u32 lane, bool sel) {
  WAIT_VM(0);
  asm volatile("" : "+v"(pws));
  u32 a = 0;
  if (lane == 0) a = __hip_atomic_fetch_add((LAS u32*)(smem + L_ARR + rnd * 4u), 1u, __ATOMIC_RELAXED, __HIP_MEMORY_SCOPE_WORKGROUP);
  if (rfl(a) == (rnd ? 7u : 1u)) {
    CFENCE();
    const u32x4 s0 = *(volatile LAS u32x4*)(smem + L_SZ + rnd * 32u), s1 = *(volatile LAS u32x4*)(smem + L_SZ + rnd * 32u + 16u);
    const u32 c16 = (lds_ld(smem, L_CALL) << 16) | 0x80000000u;
    const u32x4 rec = {s0.x | (s0.y << 12) | (s0.z << 24), c16 | (s0.z >> 8) | (s0.w << 4),
                       s1.x | (s1.y << 12) | (s1.z << 24), c16 | (s1.z >> 8) | (s1.w << 4)};
    if (sel) {
      if (rnd) {
        asm volatile("global_store_dwordx4 %0, %1, off sc0 sc1\n\ts_nop 1" :: "v"(pws + flag_off(rank, 32u + g)), "v"(rec) : "memory");
      } else {
        const u32x2 h = {rec.x | rec.z, rec.y | rec.w};
        asm volatile("global_store_dwordx2 %0, %1, off sc0 sc1\n\ts_nop 1" :: "v"(pws + flag_off(rank, 0u) + g * 8u), "v"(h) : "memory");
      }
    }
  }
}

__device__ __forceinline__ bool timed_out(const GAS u32* tmo, u32 call) {
  return rfl(__hip_atomic_load(tmo, __ATOMIC_RELAXED, __HIP_MEMORY_SCOPE_AGENT)) == call;
}

}

__global__ __launch_bounds__(NTHR, 2) void gemm_ar_kernel(Params prm) {
  __shared__ __attribute__((aligned(16))) u8 smem_raw[L_TOTAL];
  LAS u8* smem = (LAS u8*)smem_raw;
  const u32 tid = threadIdx.x, lane = tid & 63u, wid = rfl(tid >> 6);
  const u32 rank = (u32)prm.rank;
  char* wsl = (char*)prm.ws;
  const rsrc_t ws = make_rsrc(wsl, WS_NEED);
  GAS u32* tmo = (GAS u32*)(wsl + OFF_TMO);
  const u32 xcd = blockIdx.x & 7u, pos = blockIdx.x >> 3;
  DBG(const u64 tstart = now();)
  const rsrc_t rx = make_rsrc(prm.x, MM * KSH * 2), rw = make_rsrc(prm.w, KSH * NN * 2);
  const bool fast = blockIdx.x >= 128u - NFU && blockIdx.x < 128u && wid < 2u;
  if (fast) x_stage(rx, smem + wid * 18432u + 10240u, (blockIdx.x - (128u - NFU)) * 2u + wid, lane);
  const u32 cid = blockIdx.x - 128u;
  constexpr u32 PW = (NP * 8 + NL - 1) / NL;
  static_assert(PW <= 4, "push");
  const u32 pq = cid / 7u + (wid - 2u) * NL;
  const bool pusher = cid < 7u * NL && wid >= 2u && wid - 2u < PW && pq < NP * 8u;
  const u32 src = (rank + 1u + cid % 7u) & 7u;
  LAS u8* pstg = smem + (NREG - 1u - ((wid - 2u) >> 1)) * (REGK * 1024u) + (wid & 1u) * 8192u;
  if (pusher) w_stage(rw, pstg, src, pq, lane);
  DBG(const u64 tiss = now(); STAMPV(wsl, 0, tstart);)

  if (tid < (L_Z1 - L_Z0) / 4u) *(LAS u32*)(smem + L_Z0 + tid * 4u) = 0u;
  if (tid >= 496u) {
    const u32 m = tid - 496u;
    u32 cs = 0x0c0c0c0cu, r = 0;
    u64 xs = 0x0c0c0c0c0c0c0c0cull;
    for (u32 i = 0; i < 4; ++i)
      if ((m >> i) & 1u) {
        cs = (cs & ~(0xffu << (8 * r))) | (i << (8 * r));
        xs = (xs & ~(0xffffull << (16 * i))) | ((u64)(r | ((4u + r) << 8)) << (16 * i));
        ++r;
      }
    if (pos < 16u) *(LAS u32*)(smem + L_LUT + 4 * m) = cs;
    else *(LAS u64*)(smem + L_LUT + 8 * m) = xs;
  }
  GAS u32* callp = (GAS u32*)(wsl + OFF_CALL + blockIdx.x * 64u);

  if (pos < 16u) {
    LAS u8* bld = smem + wid * 18432u;
    LAS u8* stg = bld + 10240;
    LAS u32* exs = (LAS u32*)(smem + L_EXC + wid * 256u);
    const LAS u32* lutc = (const LAS u32*)(smem + L_LUT);
    char* pws = (char*)prm.peer[lane & 7u];
    const u32 prank = (rank + 1u + lane) & 7u;
    const u32 cv = __hip_atomic_load(callp, __ATOMIC_RELAXED, __HIP_MEMORY_SCOPE_AGENT);
    WAIT_LGKM0();
    __builtin_amdgcn_s_barrier();
    CFENCE();
    u32 call = 0;
    const u32 gi = blockIdx.x;
    u32 isw, gid, dest, cw;
    task_of(fast ? 0u : 1u, gi, wid, rank, isw, gid, dest, cw);
    if (!fast) {
      for (u32 i = SDELAY + (pos * SSTAG) / 4u; i > 0u; --i) __builtin_amdgcn_s_sleep(8);
      if (isw) w_stage(rw, stg, dest, gid * 8u + cw, lane);
      else x_stage(rx, stg, gid * 8u + cw, lane);
    }
    bool more = true;
    for (u32 rnd = fast ? 0u : 1u; more; ++rnd) {
      task_of(rnd, gi, wid, rank, isw, gid, dest, cw);
      const u32 id = gid * 8u + cw;
      u32 P[32];
      DBG(const u64 ti = tiss;)
      WAIT_VM(0);
      if (isw) w_fragments(stg, lane, P);
      else x_fragments(stg, lane, P);
      if (wid == 0u && call == 0u) {
        call = rfl(cv) + 1u;
        lds_st(smem, L_CALL, call);
        __hip_atomic_store(callp, call, __ATOMIC_RELAXED, __HIP_MEMORY_SCOPE_AGENT);
      }
      {
        u32 isw2, gid2, dest2, cw2;
        more = task_of(rnd + 1u, gi, wid, rank, isw2, gid2, dest2, cw2);
#pragma unroll
        for (int i = 0; i < 32; ++i) asm volatile("" : "+v"(P[i]));
        WAIT_LGKM0();
        if (more && rnd != 0u) {
          if (isw2) w_stage(rw, stg, dest2, gid2 * 8u + cw2, lane);
          else x_stage(rx, stg, gid2 * 8u + cw2, lane);
        }
      }
      DBG(for (int i = 0; i < 32; ++i) asm volatile("" : "+v"(P[i])); const u64 tl = now();)
      const bool send = !isw || dest != rank, keep = !isw || dest == rank;
      if (send)
        encode_chunk<false>(P, bld, exs, lutc, wsl, isw ? OFF_EW + (dest * 32u + gid) * 65536u : OFF_EX + gid * 65536u,
                            smem + L_SZ + rnd * 32u, cw, lane, (isw ? pos_w(gid) : pos_x(gid)) < (u32)PQUAD ? 2u : 4u);
      DBG(const u64 te = now();)
      if (send)
        publish(smem, rnd, pws, rank, rnd == 0u ? id >> 1 : isw ? pos_w(gid) : pos_x(gid), lane,
                      lane < 7u && (!isw || prank == dest));
      DBG(if (rnd == (fast ? 0u : 1u)) { STAMP(wsl, 14); STAMPV(wsl, 12, tl); STAMPV(wsl, 13, te); STAMPV(wsl, 15, ti); })
      if (rnd == 0u) {
        u32 isw2, gid2, dest2, cw2;
        task_of(1u, gi, wid, rank, isw2, gid2, dest2, cw2);
        if (isw2) w_stage(rw, stg, dest2, gid2 * 8u + cw2, lane);
        else x_stage(rx, stg, gid2 * 8u + cw2, lane);
      }
      if (keep) {
        u32 o;
        if (isw) o = OFF_BT + (id >> 4) * (1u << 20) + (rank * 16u + (id & 15u)) * 8192u;
        else o = OFF_AT + (id >> 5) * (2u << 20) + (rank * 8u + ((id >> 2) & 7u)) * 32768u + (id & 3u) * 4096u;
        const u32 hstride = isw ? 4096u : 16384u;
#pragma unroll
        for (int v = 0; v < 8; ++v) {
          const u32x4 q = {P[4 * v], P[4 * v + 1], P[4 * v + 2], P[4 * v + 3]};
          st16<16>(q, ws, lane * 16u + (v & 3) * 1024u, o + (v >> 2) * hstride);
        }
        WAIT_VM(0);
        if (lane == 0)
          __hip_atomic_fetch_add((GAS u32*)(wsl + OFF_CNT + (isw ? 16u + (id >> 4) : (id >> 5)) * 128u), 1u, __ATOMIC_RELAXED, __HIP_MEMORY_SCOPE_AGENT);
      }
    }
    STAMP(wsl, 1);
    const rsrc_t ro = make_rsrc(prm.out, (u32)MM * NN * 2u);
    const u32 wm = wid & 3u, wn = wid >> 2;
    for (u32 round = 0; round < 2u; ++round) {
      const u32 seq = round * 8u + xcd;
      u32 si, sj;
      st_coord(seq, si, sj);
      const u32 rb = si * 4u + (pos >> 2), cb = sj * 4u + (pos & 3u);
      f32x4 acc[4][4];
#pragma unroll
      for (int a = 0; a < 4; ++a)
#pragma unroll
        for (int c = 0; c < 4; ++c) acc[a][c] = (f32x4){0.f, 0.f, 0.f, 0.f};
      u32 dead = 0;
      DBG(u64 ck0 = 0;)
      for (u32 seg = 0; seg < 2u; ++seg) {
        if (wid == 0) {
          const GAS u32* cp = lane < 4u ? (const GAS u32*)(wsl + OFF_CNT + ((lane >> 1) * 32u + ((lane & 1u) ? 16u + cb : rb)) * 128u) : tmo;
          const u32 per = seg ? 7u : 1u, na = call * per * 32u, nb = call * per * 16u;
          const u64 t0 = now();
          u32 st = 0;
          for (;;) {
            const u32 v = __hip_atomic_load(cp, __ATOMIC_RELAXED, __HIP_MEMORY_SCOPE_SYSTEM);
            const u32 va = __builtin_amdgcn_readlane(v, seg * 2u), vb = __builtin_amdgcn_readlane(v, seg * 2u + 1u);
            if ((u32)__builtin_amdgcn_readlane(v, 4) == call) { st = 1; break; }
            if (va >= na && vb >= nb) {
              if ((u32)__builtin_amdgcn_readlane(v, 2) >= call * 224u && (u32)__builtin_amdgcn_readlane(v, 3) >= call * 112u) st = 2;
              break;
            }
            if (now() - t0 > DEADLINE) {
              __hip_atomic_store(tmo, call, __ATOMIC_RELAXED, __HIP_MEMORY_SCOPE_AGENT);
              st = 1;
              break;
            }
            if (va + per * 16u < na || vb + per * 8u < nb) __builtin_amdgcn_s_sleep(PSLOW);
            else __builtin_amdgcn_s_sleep(PFAST);
          }
          lds_st(smem, L_GST + (round * 2u + seg) * 4u, st);
        }
        asm volatile("s_waitcnt vmcnt(0) lgkmcnt(0)" ::: "memory");
        __builtin_amdgcn_s_barrier();
        CFENCE();
        const u32 st = lds_ld(smem, L_GST + (round * 2u + seg) * 4u);
        dead |= st & 1u;
        const u32 nks = seg ? 56u : st == 2u ? 64u : 8u;
        if (nks > 8u) STAMP(wsl, 2 + round * 3);
        DBG(if (nks > 8u) ck0 = __builtin_amdgcn_s_memtime();)
        u32 kn = rank * 8u + seg * 8u;
        if (nks == 64u) seg = 1u;
        const u32 ga = OFF_AT + rb * (2u << 20) + wid * 4096u, gb = OFF_BT + cb * (1u << 20) + wid * 2048u;
        const u32 la = (u32)(size_t)smem + wid * 4096u, lb = (u32)(size_t)smem + 32768u + wid * 2048u;
        const u32 vo = lane * 16u;
#define STG_A(I, BUF, KN) dma16<(I) * 1024>(ws, la + (BUF), vo, ga + ((KN) & 63u) * 32768u)
#define STG_B(I, BUF, KN) dma16<(I) * 1024>(ws, lb + (BUF), vo, gb + ((KN) & 63u) * 16384u)
        asm volatile("s_nop 4");
#pragma unroll
        for (int i = 0; i < 3; ++i) {
          STG_B(0, i * L_STAGE, kn); STG_B(1, i * L_STAGE, kn); STG_A(0, i * L_STAGE, kn);
          if (GV == 1 || i < 2) { STG_A(1, i * L_STAGE, kn); STG_A(2, i * L_STAGE, kn); STG_A(3, i * L_STAGE, kn); }
          ++kn;
        }
        const LAS u8* ra = smem + wm * 4096u + lane * 16u;
        const LAS u8* rbp = smem + 32768u + wn * 4096u + lane * 16u;
        bf16x8 a0[4], b0[4], a1[4], b1[4];
        if (GV == 1) WAIT_VM(12); else WAIT_VM(9);
        __builtin_amdgcn_s_barrier();
        CFENCE();
#pragma unroll
        for (int i = 0; i < 4; ++i) {
          b0[i] = *(const LAS bf16x8*)(rbp + i * 1024);
          a0[i] = *(const LAS bf16x8*)(ra + i * 1024);
        }
        u32 sbuf = 0, pbuf = 2u * L_STAGE;
#define MM_(FA, FB, A, C) acc[A][C] = __builtin_amdgcn_mfma_f32_16x16x32_bf16(FB[C], FA[A], acc[A][C], 0, 0, 0)
#define RD(DST, PTR) DST = *(const LAS bf16x8*)(PTR)
#define X1(MODE, I) if (GV == 2 && MODE <= 1) STG_A(I, pbuf, kn - 1u)
#define KSTEP(MODE) do { \
          const LAS u8* pa = ra + sbuf; const LAS u8* pb = rbp + sbuf; \
          __builtin_amdgcn_s_setprio(0); \
          SB(); MM_(a0, b0, 0, 0); SB(); RD(b1[0], pb + 8192); SB(); MM_(a0, b0, 0, 1); SB(); RD(a1[0], pa + 16384); X1(MODE, 1); \
          SB(); MM_(a0, b0, 0, 2); SB(); RD(b1[1], pb + 9216); SB(); MM_(a0, b0, 0, 3); SB(); RD(b1[2], pb + 10240); \
          SB(); MM_(a0, b0, 1, 0); SB(); RD(b1[3], pb + 11264); SB(); MM_(a0, b0, 1, 1); SB(); RD(a1[1], pa + 17408); \
          SB(); MM_(a0, b0, 1, 2); SB(); RD(a1[2], pa + 18432); X1(MODE, 2); SB(); MM_(a0, b0, 1, 3); SB(); RD(a1[3], pa + 19456); \
          SB(); MM_(a0, b0, 2, 0); MM_(a0, b0, 2, 1); MM_(a0, b0, 2, 2); MM_(a0, b0, 2, 3); SB(); X1(MODE, 3); SB(); \
          MM_(a0, b0, 3, 0); MM_(a0, b0, 3, 1); MM_(a0, b0, 3, 2); MM_(a0, b0, 3, 3); SB(); \
          const u32 nbuf = sbuf == 2u * L_STAGE ? 0u : sbuf + L_STAGE; \
          if (MODE < 3) { \
            if (MODE < 2) asm volatile("s_waitcnt vmcnt(6) lgkmcnt(0)" ::: "memory"); \
            else asm volatile("s_waitcnt vmcnt(0) lgkmcnt(0)" ::: "memory"); \
            __builtin_amdgcn_s_barrier(); CFENCE(); \
            pa = ra + nbuf; pb = rbp + nbuf; \
          } \
          __builtin_amdgcn_s_setprio(1); \
          if (GV == 1) { \
          SB(); MM_(a1, b1, 0, 0); SB(); if (MODE < 3) RD(b0[0], pb); if (MODE == 0) STG_B(0, sbuf, kn); SB(); MM_(a1, b1, 0, 1); SB(); if (MODE < 3) RD(a0[0], pa); \
          SB(); MM_(a1, b1, 0, 2); SB(); if (MODE < 3) RD(b0[1], pb + 1024); SB(); MM_(a1, b1, 0, 3); SB(); if (MODE < 3) RD(b0[2], pb + 2048); if (MODE == 0) STG_B(1, sbuf, kn); \
          SB(); MM_(a1, b1, 1, 0); SB(); if (MODE < 3) RD(b0[3], pb + 3072); SB(); MM_(a1, b1, 1, 1); SB(); if (MODE < 3) RD(a0[1], pa + 1024); \
          SB(); MM_(a1, b1, 1, 2); SB(); if (MODE < 3) RD(a0[2], pa + 2048); if (MODE == 0) STG_A(0, sbuf, kn); SB(); MM_(a1, b1, 1, 3); SB(); if (MODE < 3) RD(a0[3], pa + 3072); \
          SB(); MM_(a1, b1, 2, 0); SB(); MM_(a1, b1, 2, 1); SB(); if (MODE == 0) STG_A(1, sbuf, kn); \
          SB(); MM_(a1, b1, 2, 2); SB(); MM_(a1, b1, 2, 3); SB(); MM_(a1, b1, 3, 0); SB(); if (MODE == 0) STG_A(2, sbuf, kn); \
          SB(); MM_(a1, b1, 3, 1); MM_(a1, b1, 3, 2); MM_(a1, b1, 3, 3); SB(); if (MODE == 0) STG_A(3, sbuf, kn); SB(); \
          } else { \
          SB(); MM_(a1, b1, 0, 0); SB(); if (MODE < 3) RD(b0[0], pb); SB(); MM_(a1, b1, 0, 1); SB(); if (MODE < 3) RD(a0[0], pa); if (MODE == 0) STG_B(0, sbuf, kn); \
          SB(); MM_(a1, b1, 0, 2); SB(); if (MODE < 3) RD(b0[1], pb + 1024); SB(); MM_(a1, b1, 0, 3); SB(); if (MODE < 3) RD(b0[2], pb + 2048); \
          SB(); MM_(a1, b1, 1, 0); SB(); if (MODE < 3) RD(b0[3], pb + 3072); SB(); MM_(a1, b1, 1, 1); SB(); if (MODE < 3) RD(a0[1], pa + 1024); \
          SB(); MM_(a1, b1, 1, 2); SB(); if (MODE < 3) RD(a0[2], pa + 2048); if (MODE == 0) STG_B(1, sbuf, kn); SB(); MM_(a1, b1, 1, 3); SB(); if (MODE < 3) RD(a0[3], pa + 3072); \
          SB(); MM_(a1, b1, 2, 0); MM_(a1, b1, 2, 1); MM_(a1, b1, 2, 2); MM_(a1, b1, 2, 3); SB(); if (MODE == 0) STG_A(0, sbuf, kn); SB(); \
          MM_(a1, b1, 3, 0); MM_(a1, b1, 3, 1); MM_(a1, b1, 3, 2); MM_(a1, b1, 3, 3); SB(); \
          } \
          ++kn; pbuf = sbuf; sbuf = nbuf; \
        } while (0)
        for (u32 ks = 3; ks < nks; ++ks) KSTEP(0);
        KSTEP(1);
        KSTEP(2);
        KSTEP(3);
        __builtin_amdgcn_s_setprio(0);
      }
      STAMP(wsl, 3 + round * 3);
      DBG(STAMPV(wsl, 8 + round, __builtin_amdgcn_s_memtime() - ck0);)
      u32 P[32];
#pragma unroll
      for (int a = 0; a < 4; ++a)
#pragma unroll
        for (int c = 0; c < 4; ++c)
#pragma unroll
          for (int j = 0; j < 2; ++j) {
            const f32x2 v = {__builtin_elementwise_maximum(acc[a][c][2 * j], 0.f), __builtin_elementwise_maximum(acc[a][c][2 * j + 1], 0.f)};
            const u32 pk = __builtin_bit_cast(u32, __builtin_convertvector(v, bf16x2));
            P[a * 8 + c * 2 + j] = dead ? 0x7fc07fc0u : pk;
          }
      WAIT_LGKM0();
      __builtin_amdgcn_s_barrier();
      CFENCE();
      const u32 ts = tile_seq(seq, pos);
      DBG(const u64 te0 = now();)
      encode_chunk<true>(P, bld, exs, lutc, wsl, OFF_EO + ts * 65536u, smem + L_SZ + (4u + round) * 32u, wid, lane,
                         ts >= 256u - TAILT ? 1u : ts >= (u32)OCT0 && ts < (u32)OCT1 ? 8u : 4u);
      DBG(const u64 te1 = now();)
      publish(smem, 4u + round, pws, rank, 96u + ts, lane, lane < 7u);
      DBG(STAMPV(wsl, 10, te1 - te0); STAMPV(wsl, 11, now() - te1);)
      STAMP(wsl, 4 + round * 3);
      {
        const u32 voff = (lane & 15u) * (NN * 2u) + (lane >> 4) * 16u;
        const u32 soff = (rb * 256u + wm * 64u) * (NN * 2u) + (rank * NSH + cb * 128u + wn * 64u) * 2u;
#pragma unroll
        for (int v = 0; v < 8; ++v) {
          const u32x4 q = {P[4 * v], P[4 * v + 1], P[4 * v + 2], P[4 * v + 3]};
          st16<OAUX>(q, ro, voff + (v & 1) * 64u, soff + (v >> 1) * 16u * (NN * 2u));
        }
      }
    }
  } else {
    if (cid >= 7u * NL) return;
    char* wsp = (char*)prm.peer[cid % 7u];
    WAIT_LGKM0();
    __builtin_amdgcn_s_barrier();
    CFENCE();
    if (pusher) {
      u32 P[32];
      WAIT_VM(0);
      DBG(STAMP(wsl, 8);)
      w_fragments(pstg, lane, P);
      const rsrc_t rp = make_rsrc(wsp, WS_NEED);
      const u32 o = OFF_BT + (pq >> 4) * (1u << 20) + (rank * 16u + (pq & 15u)) * 8192u;
#pragma unroll
      for (int v = 0; v < 8; ++v) {
        const u32x4 q = {P[4 * v], P[4 * v + 1], P[4 * v + 2], P[4 * v + 3]};
        st16<17>(q, rp, lane * 16u + v * 1024u, o);
        asm volatile("s_waitcnt vmcnt(" STR(PWIN) ")" ::: "memory");
      }
      WAIT_VM(0);
      if (lane == 0)
        __hip_atomic_fetch_add((GAS u32*)(wsp + OFF_CNT + (48u + (pq >> 4)) * 128u), 1u, __ATOMIC_RELAXED, __HIP_MEMORY_SCOPE_SYSTEM);
      DBG(STAMP(wsl, 9);)
    }
    if (wid >= 2u && wid < 6u && lane == 0)
      __hip_atomic_fetch_add((LAS u32*)(smem + L_FREE + (NREG - 1u - ((wid - 2u) >> 1)) * 4u), 1u, __ATOMIC_RELAXED, __HIP_MEMORY_SCOPE_WORKGROUP);
    if (wid == 1u) {
      GAS u32* tk = (GAS u32*)(wsl + OFF_TK + src * 128u);
      const u32 cv = __hip_atomic_load(callp, __ATOMIC_RELAXED, __HIP_MEMORY_SCOPE_AGENT);
      u32 tnext = 0;
      if (lane == 0) tnext = __hip_atomic_fetch_add(tk, 1u, __ATOMIC_RELAXED, __HIP_MEMORY_SCOPE_AGENT);
      const u32 call = rfl(cv) + 1u;
      lds_st(smem, L_CALL, call);
      __hip_atomic_store(callp, call, __ATOMIC_RELAXED, __HIP_MEMORY_SCOPE_AGENT);
      const u32 tbase = (call - 1u) * (u32)(NTK + NL);
      u32 nreq = 0, njob = 0, pieces = 0, region = 0, dead = 0;
      u32 cum = (lane == NREG - 1u || lane == NREG - 2u) ? 2u : 0u;
      u32 wbase = 0;
      u64 win = 0;
      __builtin_amdgcn_s_sleep(FIRSTSLEEP);
      DBG(u64 tw[4] = {};)
      for (;;) {
        const u32 t = rfl(tnext) - tbase;
        if (t >= (u32)NTK) break;
        u32 g, c0, cnt;
        const bool tail = t >= TK_F;
        if (t < TK_B) { g = t >> 2; c0 = (t & 3u) * 2u; cnt = 2; }
        else if (t < NTI) { g = PQUAD + ((t - TK_B) >> 1); c0 = (t & 1u) * 4u; cnt = 4; }
        else if (t < TK_D) { g = 96u + ((t - NTI) >> 1); c0 = (t & 1u) * 4u; cnt = 4; }
        else if (t < TK_E) { g = 96u + OCT0 + (t - TK_D); c0 = 0; cnt = 8; }
        else if (!tail) { g = 96u + OCT1 + ((t - TK_E) >> 1); c0 = ((t - TK_E) & 1u) * 4u; cnt = 4; }
        else { const u32 tt = (t - TK_F) * TAILC; g = 96u + (256 - TAILT) + (tt >> 3); c0 = tt & 7u; cnt = TAILC; }
        u32 kind, id;
        const u32 code = pos_code(g);
        code_inv(code, kind, id);
        const u32 ub = (kind == 0 ? OFF_EW + rank * (32u * 65536u) : kind == 1 ? OFF_EX : OFF_EO) + (id + c0) * 8192u;
        const u32 hx = t < NFU ? t : 64u + 2u * g + (c0 >> 2), two = cnt >> 3, tag = (call & 0xffffu) | 0x8000u;
        u64 rec = 0, rec2 = 0;
        auto lane64 = [](u64 v, u32 l) { return ((u64)(u32)__builtin_amdgcn_readlane((u32)(v >> 32), l) << 32) | (u32)__builtin_amdgcn_readlane((u32)v, l); };
        if (hx - wbase < 63u) { rec = lane64(win, hx - wbase); rec2 = lane64(win, hx - wbase + two); }
        if ((u32)(rec >> 48) != tag || (u32)(rec2 >> 48) != tag) {
          const u64 t0 = now();
          const GAS u64* fp = (const GAS u64*)(wsl + flag_off(src, 0)) + hx;
          wbase = hx;
          DBG(u32 polls = 0;)
          if (!dead) win = __hip_atomic_load(fp + lane, __ATOMIC_RELAXED, __HIP_MEMORY_SCOPE_SYSTEM);
          rec = lane64(win, 0u);
          rec2 = lane64(win, two);
          for (u32 it = 0; !dead && ((u32)(rec >> 48) != tag || (u32)(rec2 >> 48) != tag); ++it) {
            DBG(++polls;)
            if ((it & 7u) == 7u && (timed_out(tmo, call) || now() - t0 > DEADLINE)) {
              __hip_atomic_store(tmo, call, __ATOMIC_RELAXED, __HIP_MEMORY_SCOPE_AGENT);
              dead = 1;
            }
            if (nreq != 0u) __builtin_amdgcn_s_sleep(FSLEEP);
            const u64 f = __hip_atomic_load(fp + (lane & 1u), __ATOMIC_RELAXED, __HIP_MEMORY_SCOPE_SYSTEM);
            rec = lane64(f, 0u);
            rec2 = lane64(f, two);
          }
          if (dead) rec = rec2 = 0;
          DBG(if (polls && nreq > 0) tw[t < NTI ? 0 : 2] += now() - t0;)
        }
        auto sz = [&](u32 i) { const u32 v = (u32)(((i & 4u) ? rec2 : rec) >> (12u * (i & 3u))) & 4095u; return v > 2048u ? 2048u : v; };
        u32 tot = 0, big = 0;
        for (u32 i = c0; i < c0 + cnt; ++i) { tot += sz(i); big = sz(i) > big ? sz(i) : big; }
        if (pieces - lds_ld(smem, L_LANDED) > (u32)THRESH) {
          const u64 t0 = now();
          while (pieces - lds_ld(smem, L_LANDED) > (u32)THRESH && now() - t0 < 4 * DEADLINE) __builtin_amdgcn_s_sleep(DSLEEP);
        }
        auto post_req = [&](u32 go, u32 kb, u32 len, u32 need) {
          lds_st(smem, L_REQ + (nreq % NQ) * 8u + 4u, go | (region << 27) | (region * REGK + kb));
          lds_st(smem, L_REQ + (nreq % NQ) * 8u, ((nreq + 1u) & 255u) | (((len + 3u) >> 2) << 8) | (need << 19));
          ++nreq;
        };
        auto post_job = [&](u32 e, u32 v, u32 off, u32 nh, u32 na) {
          lds_st(smem, L_JOB + (njob % NJ) * 8u + 4u, e | (off << 12) | ((v == 0u ? 2u : v >> 11) << 25));
          lds_st(smem, L_JOB + (njob % NJ) * 8u, ((njob + 1u) & 255u) | (region << 8) | ((na - nh) << 11) | (nh << 16));
          ++njob;
        };
        if (tail && big <= 1024u && !dead) {
          const u32 need = __builtin_amdgcn_readlane(cum, region), wide = t >= (u32)(NTK - NWT) ? 0x80000000u : 0u;
          for (u32 j = 0; j < cnt; ++j) post_req(ub + j * 8192u, j * 4u, sz(c0 + j) < 256u ? sz(c0 + j) : 256u, need);
          u32 na = pieces + cnt;
          for (u32 j = 0; j < cnt; ++j) {
            const u32 v = sz(c0 + j);
            if (v > 256u) {
              post_req((ub + j * 8192u + 1024u) | wide, j * 4u + 1u, v - 256u, need);
              na += (v - 1u) >> 8;
            }
            post_job(code + c0 + j, v, j * 1024u, pieces + j + 1u, na);
          }
          pieces = na;
          cum += lane == region ? cnt : 0u;
          region = region == NREG - 1u ? 0u : region + 1u;
        } else {
          const u32 take = !tail && tot <= REGK * 256u ? cnt : 1u, h4 = kind == 2u ? HEAD_OUT / 4u : HEAD_IN / 4u;
          u32 pre = 0;
          for (u32 i = c0; i < c0 + cnt; i += take) {
            const u32 lead = pre & 31u;
            post_req(ub + (pre - lead) * 4u, 0u, lead + (take == cnt ? tot : sz(i)), __builtin_amdgcn_readlane(cum, region));
            u32 off = lead;
            for (u32 j = 0; j < take; ++j) {
              const u32 v = sz(i + j), nh = (off + (v < h4 ? v : h4) + 255u) >> 8, na = (off + v + 255u) >> 8;
              post_job(code + i + j, v, off, pieces + (nh ? nh : 1u), pieces + (na ? na : 1u));
              off += v;
            }
            cum += lane == region ? take : 0u;
            const u32 np = (off + 255u) >> 8;
            pieces += np ? np : 1u;
            pre += tail ? 2048u : off - lead;
            region = region == NREG - 1u ? 0u : region + 1u;
          }
        }
        if (lane == 0) tnext = __hip_atomic_fetch_add(tk, 1u, __ATOMIC_RELAXED, __HIP_MEMORY_SCOPE_AGENT);
      }
      lds_st(smem, L_JEND, njob + 1u);
      lds_st(smem, L_REND, nreq + 1u);
      DBG(for (int i = 0; i < 4; ++i) STAMPV(wsl, 10 + i, tw[i]);)
    } else if (wid == 0) {
      const rsrc_t rp = make_rsrc(wsp, WS_NEED);
      const u32 lbase = (u32)(size_t)smem;
      u32 issued = 0, pub = 0;
      DBG(u64 tw[4] = {}; u32 seen = 0;)
      for (u32 n = 0;; ++n) {
        u32 rq, g1;
        auto ready = [&]() {
          rq = lds_ld(smem, L_REQ + (n % NQ) * 8u);
          g1 = lds_ld(smem, L_REQ + (n % NQ) * 8u + 4u);
          return (rq & 255u) == ((n + 1u) & 255u) && lds_ld(smem, L_FREE + ((g1 >> 27) & 7u) * 4u) >= (rq >> 19);
        };
        bool stop = false;
        if (!ready()) {
          const u64 t0 = now();
          DBG(const bool wf = (rq & 255u) != ((n + 1u) & 255u);)
          for (;;) {
            if (issued > pub) {
              switch (issued - pub) {
                case 2: WAIT_VM(1); break;
                case 3: WAIT_VM(2); break;
                case 4: WAIT_VM(3); break;
                case 5: WAIT_VM(4); break;
                case 6: WAIT_VM(5); break;
                case 7: WAIT_VM(6); break;
                case 8: WAIT_VM(7); break;
                default: WAIT_VM(0); break;
              }
              ++pub;
              lds_st(smem, L_LANDED, pub);
            } else {
              const u32 ne = lds_ld(smem, L_REND);
              if ((ne != 0u && n + 1u >= ne) || now() - t0 > 4 * DEADLINE) { stop = true; break; }
              __builtin_amdgcn_s_sleep(1);
            }
            if (ready()) break;
          }
          DBG(if (n > 0 && !stop) { u64 dt = now() - t0; if (!(seen & 16)) { if (wf) tw[0] += dt; else tw[1] += dt; } else { if (wf) tw[2] += dt; else tw[3] += dt; } })
        }
        if (stop) break;
        const u32 go = g1 & 0x07ffff80u, tot = (rq >> 8) & 2047u;
        u32 np = (tot + 63u) >> 6;
        np = np ? (np > (u32)REGK ? (u32)REGK : np) : 1u;
        for (u32 pc = 0; pc < np; ++pc) {
          const u32 o16 = pc * 64u + lane;
          const u32 vo = o16 < tot ? o16 * 16u : 0x80000000u, ld = lbase + ((g1 & 127u) + pc) * 1024u;
          if ((int)g1 >= 0) {
            asm volatile("s_mov_b32 m0, %0\n\ts_nop 3\n\ts_waitcnt vmcnt(" STR(WINDOW) " - 1)\n\tbuffer_load_dwordx4 %1, %2, %3 offen sc0 sc1 lds"
                         :: "s"(ld), "v"(vo), "s"(rp), "s"(go) : "memory");
            if (issued >= pub + WINDOW) { pub = issued - (WINDOW - 1u); lds_st(smem, L_LANDED, pub); }
          } else {
            asm volatile("s_mov_b32 m0, %0\n\ts_nop 3\n\ts_waitcnt vmcnt(" STR(TWIN) " - 1)\n\tbuffer_load_dwordx4 %1, %2, %3 offen sc0 sc1 lds"
                         :: "s"(ld), "v"(vo), "s"(rp), "s"(go) : "memory");
            if (issued >= pub + TWIN) { pub = issued - (TWIN - 1u); lds_st(smem, L_LANDED, pub); }
          }
          ++issued;
        }
      }
      STAMP(wsl, 3);
      DBG(for (int i = 0; i < 4; ++i) STAMPV(wsl, 10 + i, tw[i]);)
    } else {
      const rsrc_t ro = make_rsrc(prm.out, (u32)MM * NN * 2u);
      LAS u8* hs = smem + L_HS + (wid - 2u) * L_HS_STRIDE;
      const LAS u32x2* lutx = (const LAS u32x2*)(smem + L_LUT);
      DBG(u64 td[4] = {};)
      for (u32 n = wid - 2u;; n += NDEC) {
        bool stop = false;
        u32 j0;
        {
          const u64 t0 = now();
          for (;;) {
            j0 = lds_ld(smem, L_JOB + (n % NJ) * 8u);
            if ((j0 & 255u) == ((n + 1u) & 255u)) break;
            const u32 ne = lds_ld(smem, L_JEND);
            if (ne != 0u && n + 1u >= ne) { stop = true; break; }
            if (now() - t0 > 5 * DEADLINE) {
              if (lane == 0) {
                __hip_atomic_store(tmo, lds_ld(smem, L_CALL), __ATOMIC_RELAXED, __HIP_MEMORY_SCOPE_AGENT);
                ((GAS unsigned short*)prm.out)[src * NSH] = 0x7fc0;
              }
              stop = true;
              break;
            }
            __builtin_amdgcn_s_sleep(DSLEEP);
          }
        }
        if (stop) break;
        CFENCE();
        DBG(u64 td0 = now(); if (n == wid - 2u) STAMP(wsl, 1);)
        const u32 j1 = lds_ld(smem, L_JOB + (n % NJ) * 8u + 4u);
        const u32 region = (j0 >> 8) & 7u;
        const LAS u8* ch = smem + region * (REGK * 1024u) + ((j1 >> 12) & 8191u) * 4u;
        const bool dead = (j1 >> 25) == 2u, raw = (j1 >> 25) == 1u;
        u32 kind, id;
        code_inv(j1 & 4095u, kind, id);
        bool late = false;
        auto landed = [&](u32 bytes) {
          const u32 need = (j0 >> 16) + (bytes <= (kind == 2u ? HEAD_OUT : HEAD_IN) ? 0u : (j0 >> 11) & 31u);
          if (lds_ld(smem, L_LANDED) < need) {
            const u64 t0 = now();
            while (lds_ld(smem, L_LANDED) < need) {
              if (now() - t0 > 5 * DEADLINE) { late = true; break; }
              __builtin_amdgcn_s_sleep(DSLEEP);
            }
          }
          CFENCE();
          DBG(if (bytes == 8192u) td0 = now();)
        };
        u32 P[32];
        if (dead) {
          landed(0u);
        } else if (kind == 2) {
          decode_chunk<true>(ch, hs, lutx, raw, lane, P, landed);
        } else {
          decode_chunk<false>(ch, hs, lutx, raw, lane, P, landed);
        }
        if (dead || late) {
#pragma unroll
          for (int i = 0; i < 32; ++i) P[i] = 0x7fc07fc0u;
        }
#pragma unroll
        for (int i = 0; i < 32; ++i) asm volatile("" : "+v"(P[i]));
        WAIT_LGKM0();
        if (lane == 0) __hip_atomic_fetch_add((LAS u32*)(smem + L_FREE + region * 4u), 1u, __ATOMIC_RELAXED, __HIP_MEMORY_SCOPE_WORKGROUP);
        DBG(td[kind == 2 ? 2 : 0] += now() - td0;)
        if (kind == 2) {
          u32 rb, cb;
          tile_inv(id >> 3, rb, cb);
          const u32 w = id & 7u;
          const u32 voff = (lane & 15u) * (NN * 2u) + (lane >> 4) * 16u;
          const u32 soff = (rb * 256u + (w & 3u) * 64u) * (NN * 2u) + (src * NSH + cb * 128u + (w >> 2) * 64u) * 2u;
#pragma unroll
          for (int v = 0; v < 8; ++v) {
            const u32x4 q = {P[4 * v], P[4 * v + 1], P[4 * v + 2], P[4 * v + 3]};
            st16<OAUX>(q, ro, voff + (v & 1) * 64u, soff + (v >> 1) * 16u * (NN * 2u));
          }
        } else {
          u32 o;
          if (kind == 0) o = OFF_BT + (id >> 4) * (1u << 20) + (src * 16u + (id & 15u)) * 8192u;
          else o = OFF_AT + (id >> 5) * (2u << 20) + (src * 8u + ((id >> 2) & 7u)) * 32768u + (id & 3u) * 4096u;
          const u32 hstride = kind == 0 ? 4096u : 16384u;
#pragma unroll
          for (int v = 0; v < 8; ++v) {
            const u32x4 q = {P[4 * v], P[4 * v + 1], P[4 * v + 2], P[4 * v + 3]};
            st16<16>(q, ws, lane * 16u + (v & 3) * 1024u, o + (v >> 2) * hstride);
          }
          WAIT_VM(0);
          if (lane == 0)
            __hip_atomic_fetch_add((GAS u32*)(wsl + OFF_CNT + (32u + (kind == 0 ? 16u + (id >> 4) : (id >> 5))) * 128u), 1u, __ATOMIC_RELAXED, __HIP_MEMORY_SCOPE_SYSTEM);
        }
        DBG(td[kind == 2 ? 3 : 1] += now() - td0; if (n == wid - 2u) STAMP(wsl, 2);)
      }
      STAMP(wsl, 3);
      DBG(for (int i = 0; i < 4; ++i) STAMPV(wsl, 10 + i, td[i]);)
    }
  }
}

extern "C" void kernel_launch(int rank, int nranks, ncclComm_t comm, hipStream_t stream, void* const* d_in, const int* in_sizes,
                              int n_in, void* d_out, int out_size, const int* in_dtypes, int out_dtype, void* const* d_out_all,
                              void* const* d_ws_all, size_t ws_bytes) {
  (void)comm; (void)in_sizes; (void)n_in; (void)out_size; (void)in_dtypes; (void)out_dtype; (void)d_out_all;
  if (nranks != NR || ws_bytes < WS_NEED || (unsigned)rank >= (unsigned)NR) {
    fprintf(stderr, "unsupported configuration\n");
    return;
  }
  Params p;
  p.x = d_in[0];
  p.w = d_in[1];
  p.out = d_out;
  p.ws = d_ws_all[rank];
  for (int i = 0; i < NR; ++i) p.peer[i] = d_ws_all[(rank + 1 + i) % NR];
  p.rank = rank;
  p.pad = 0;
  gemm_ar_kernel<<<NWG, NTHR, 0, stream>>>(p);
}
```

```cpp
#include <hip/hip_runtime.h>
#include <rccl/rccl.h>
#include <cstdio>

typedef unsigned u32;
typedef unsigned long long u64;
typedef unsigned char u8;
typedef u32 u32x2 __attribute__((ext_vector_type(2)));
typedef u32 u32x4 __attribute__((ext_vector_type(4)));
typedef float f32x2 __attribute__((ext_vector_type(2)));
typedef float f32x4 __attribute__((ext_vector_type(4)));
typedef short s16x2 __attribute__((ext_vector_type(2)));
typedef short s16x4 __attribute__((ext_vector_type(4)));
typedef __bf16 bf16x2 __attribute__((ext_vector_type(2)));
typedef __bf16 bf16x8 __attribute__((ext_vector_type(8)));
#define LAS __attribute__((address_space(3)))
#define GAS __attribute__((address_space(1)))
typedef __amdgpu_buffer_rsrc_t rsrc_t;

namespace {

constexpr int NR = 8;
constexpr int MM = 4096, NN = 16384, KSH = 512, NSH = 2048;
constexpr int NWG = 256, NTHR = 512;
#define NLOAD 18
constexpr int NL = NLOAD;
#define NFASTG 16
constexpr u32 NF = NFASTG;
constexpr u32 NFU = 4 * NF;
constexpr int NGRP = 32 + 96 + 256;
#define SLICE 4
#define OWNB 28
#define MINHELP 6
#define WAITHELP 24
#define GAP0 5
#define GAPC 7
#define NBAL 5
#define NHELP 9
#define BAL0 20
#define NQD 5
constexpr u32 TAILT = 7 * SLICE, NTS = 4 * SLICE - NQD, RB0 = 256 - TAILT - OWNB;
#define NPUSHG 4
constexpr u32 NP = NPUSHG;
#define PQUAD 16
#define OCT0 96
#define OCT1 190
constexpr u32 TK_B = 4 * PQUAD, NTI = TK_B + 2 * (96 - NP - PQUAD), TK_D = NTI + 2 * OCT0, TK_E = TK_D + (OCT1 - OCT0);
constexpr u32 TK_F = TK_E + 2 * (RB0 - OCT1), TK_G = TK_F + 2 * SLICE;
constexpr int NTK = TK_G + 2 * OWNB;
static_assert(PQUAD >= NF && PQUAD <= 96 - NP && OCT0 <= OCT1 && OCT1 <= RB0 && SLICE <= 4 && BAL0 <= 2 * OWNB, "units");
constexpr u32 NTASK = 320 - NF - 7 * NP;
#define SDELAY 8
#define SSTAG 8
constexpr int NREG = 4, REGK = 25, NDEC = 6, NQ = 32, NJ = 128;
#define WINDOW 3
#define THRESH 6
#define OAUX 16
#define PSLOW 127
#define PFAST 16
#define DSLEEP 3
#define PWIN 1
#define FSLEEP 8
#define FIRSTSLEEP 32
#define GV 2
constexpr u64 DEADLINE = 30000000ull;

constexpr u32 OFF_CALL = 0;
constexpr u32 OFF_TMO = 16384;
constexpr u32 OFF_TK = 17408;
constexpr u32 OFF_RC = 19456;
constexpr u32 OFF_RF = 28672;
constexpr u32 OFF_CNT = 20480;
constexpr u32 OFF_FLAG = 32768;
constexpr u32 OFF_DBG = 512u << 10;
constexpr u32 OFF_EX = 1u << 20;
constexpr u32 OFF_EW = 5u << 20;
constexpr u32 OFF_EO = 21u << 20;
constexpr u32 OFF_AT = 37u << 20;
constexpr u32 OFF_BT = 69u << 20;
constexpr u32 OFF_RL = 85u << 20;
constexpr u32 WS_NEED = 87u << 20;

constexpr int L_STAGE = 49152;
constexpr int L_HS = NREG * REGK * 1024 + 512;
constexpr int L_HS_STRIDE = 4160;
constexpr int L_MISC = 3 * L_STAGE;
constexpr int L_LUT = L_MISC;
constexpr int L_Z0 = L_MISC + 128;
constexpr int L_REQ = L_Z0;
constexpr int L_JOB = L_REQ + 8 * NQ;
constexpr int L_FREE = L_JOB + 8 * NJ;
constexpr int L_LANDED = L_FREE + 32;
constexpr int L_REND = L_LANDED + 4;
constexpr int L_JEND = L_LANDED + 8;
constexpr int L_CALL = L_LANDED + 16;
constexpr int L_SZ = L_LANDED + 32;
constexpr int L_ARR = L_SZ + 192;
constexpr int L_GST = L_ARR + 32;
constexpr int L_Z1 = L_GST + 16;
constexpr int L_EXC = L_MISC + 2048;
constexpr int L_TOTAL = L_EXC + 8 * 256;
static_assert(L_HS + NDEC * L_HS_STRIDE <= L_MISC, "lds");
static_assert(L_Z1 <= L_EXC && (L_Z1 - L_Z0) / 4 <= NTHR, "lds");
static_assert(L_TOTAL <= 163840, "lds");

constexpr u32 HEAD_OUT = 1024, HEAD_IN = 1536;
constexpr u32 HD_OFF = 512, UN_OFF = 516;

struct Params {
  const void* x;
  const void* w;
  void* out;
  void* ws;
  int rank;
  int pad;
  void* peer[NR];
};

#define CFENCE() asm volatile("" ::: "memory")
#define WAIT_VM(n) asm volatile("s_waitcnt vmcnt(" #n ")" ::: "memory")
#define WAIT_LGKM0() asm volatile("s_waitcnt lgkmcnt(0)" ::: "memory")
#define SB() __builtin_amdgcn_sched_barrier(0)
#define STR2(x) #x
#define STR(x) STR2(x)

__device__ __forceinline__ u32 rfl(u32 v) { return __builtin_amdgcn_readfirstlane(v); }
__device__ __forceinline__ u32 vperm(u32 hi, u32 lo, u32 sel) { return __builtin_amdgcn_perm(hi, lo, sel); }
__device__ __forceinline__ u32 bsum(u32 v, u32 acc) { return __builtin_amdgcn_sad_u8(v, 0u, acc); }
__device__ __forceinline__ u64 now() { return __builtin_amdgcn_s_memrealtime(); }
__device__ __forceinline__ u32 mbcnt(u64 m) {
  return __builtin_amdgcn_mbcnt_hi((u32)(m >> 32), __builtin_amdgcn_mbcnt_lo((u32)m, 0u));
}
__device__ __forceinline__ u32 scan_incl(u32 x) {
  x += __builtin_amdgcn_update_dpp(0u, x, 0x111, 0xf, 0xf, true);
  x += __builtin_amdgcn_update_dpp(0u, x, 0x112, 0xf, 0xf, true);
  x += __builtin_amdgcn_update_dpp(0u, x, 0x114, 0xf, 0xf, true);
  x += __builtin_amdgcn_update_dpp(0u, x, 0x118, 0xf, 0xf, true);
  x += __builtin_amdgcn_update_dpp(0u, x, 0x142, 0xa, 0xf, false);
  x += __builtin_amdgcn_update_dpp(0u, x, 0x143, 0xc, 0xf, false);
  return x;
}
__device__ __forceinline__ rsrc_t make_rsrc(const void* p, u32 bytes) {
  return __builtin_amdgcn_make_buffer_rsrc((void*)p, 0, bytes, 0x00020000);
}
#define GSC " sc0 sc1"
template <int IMM>
__device__ __forceinline__ void dma16(rsrc_t r, u32 lds, u32 voff, u32 soff) {
  asm volatile("s_mov_b32 m0, %0\n\ts_nop 0\n\tbuffer_load_dwordx4 %1, %2, %3 offen offset:%4" GSC " lds"
               :: "s"(lds), "v"(voff), "s"(r), "s"(soff), "i"(IMM) : "memory");
}
__device__ __forceinline__ u32 draw32(const void* p) {
  u32 r;
  asm volatile("s_mov_b64 exec, 1\n\tglobal_atomic_add %0, %1, %2, off sc0\n\ts_waitcnt vmcnt(0)\n\ts_mov_b64 exec, -1"
               : "=&v"(r) : "v"(p), "v"(1u) : "memory");
  return rfl(r);
}
__device__ __forceinline__ u64 draw64(const void* p, u64 inc) {
  u64 r;
  asm volatile("s_mov_b64 exec, 1\n\tglobal_atomic_add_x2 %0, %1, %2, off sc0\n\ts_waitcnt vmcnt(0)\n\ts_mov_b64 exec, -1"
               : "=&v"(r) : "v"(p), "v"(inc) : "memory");
  return ((u64)rfl((u32)(r >> 32)) << 32) | rfl((u32)r);
}
__device__ __forceinline__ u64 draw64x8(const void* p, u64 inc) {
  u64 r;
  asm volatile("s_mov_b64 exec, 0xff\n\tglobal_atomic_add_x2 %0, %1, %2, off sc0\n\ts_waitcnt vmcnt(0)\n\ts_mov_b64 exec, -1"
               : "=&v"(r) : "v"(p), "v"(inc) : "memory");
  return r;
}
template <int AUX>
__device__ __forceinline__ void st16(u32x4 v, rsrc_t r, u32 voff, u32 soff) {
  __builtin_amdgcn_raw_buffer_store_b128(v, r, voff, soff, AUX);
  SB();
  asm volatile("s_nop 1");
  SB();
}
__device__ __forceinline__ u32 lds_ld(LAS u8* lds, int off) {
  return rfl(*(volatile LAS u32*)(lds + off));
}
__device__ __forceinline__ void lds_st(LAS u8* lds, int off, u32 v) { *(volatile LAS u32*)(lds + off) = v; }

#define STAMP(ws, slot) do {} while (0)
#define STAMPV(ws, slot, v) do {} while (0)
#define DBG(x)

__device__ __forceinline__ u64 rank_table(u32 c) {
  const u32 k = c & 7u;
  const u64 below = (1ull << (8 * k)) - 1ull;
  const u64 t = (0x0001020304050607ull & ~below & ~(0xffull << (8 * k))) | (7ull << (8 * k)) | (0x0000010203040506ull & below);
  return (c & 8u) ? (t + 0x0707070707070707ull) & 0x0707070707070707ull : t;
}
__device__ __forceinline__ u64 len_table(u32 c) {
  const u32 k = c & 7u;
  const u64 t = (0x0001020304050607ull + ((0x0101010101010101ull >> (8 * k)) >> 8)) | ((u64)(k + 1) << 56);
  return (c & 8u) ? (t >> 8) | (t << 56) : t;
}
__device__ __forceinline__ u32 top_mask(u32 v) { return vperm(v << 8, v, 0x090b080au); }
__device__ __forceinline__ u32 bsel(u32 m, u32 a, u32 b) { return (m & a) | (~m & b); }
__device__ __forceinline__ u32 unit_offset(LAS u8* szw, u32 wid, u32 U) {
  u32 off = 0;
  for (u32 j = wid & ~(U - 1u); j < wid; ++j) {
    u32 v = lds_ld(szw, j * 4u);
    for (int spin = 0; v == 0u && spin < (1 << 22); ++spin) v = lds_ld(szw, j * 4u);
    off += v;
  }
  return off;
}
template <bool OUT>
__device__ __forceinline__ void encode_chunk(const u32 (&P)[32], LAS u8* bld, LAS u32* exs, const LAS u32* lutc, char* wsl,
                                             u32 gbase, LAS u8* szw, u32 wid, u32 lane, u32 U) {
  const u32 ub = gbase + (wid & ~(U - 1u)) * 8192u;
  u32 lo[16], len[16], nzm[16];
  u32 pl[2] = {0u, 0u};
  u32 size4 = 2048u, nexc = 0, nw = 0, lbase = 0, pbase = 0, qbase = 0, header = 0;
  {
    u32 hi[16];
    u32 sum = 0, sgn = 0;
#pragma unroll
    for (int g = 0; g < 16; ++g) {
      lo[g] = vperm(P[2 * g + 1], P[2 * g], 0x06040200u);
      hi[g] = vperm(P[2 * g + 1], P[2 * g], 0x07050301u);
      if (OUT) {
        const s16x2 z = {0, 0};
        const u32 n0 = __builtin_bit_cast(u32, z - __builtin_bit_cast(s16x2, P[2 * g]));
        const u32 n1 = __builtin_bit_cast(u32, z - __builtin_bit_cast(s16x2, P[2 * g + 1]));
        nzm[g] = vperm(n1, n0, 0x0b0a0908u);
        pl[g >> 3] |= bsum(nzm[g] & 0x08040201u, 0u) << (4 * (g & 7));
        sgn |= hi[g];
      } else {
        pl[g >> 3] |= (hi[g] >> (7 - (g & 7))) & (0x01010101u << (g & 7));
        hi[g] &= 0x7f7f7f7fu;
        nzm[g] = 0xffffffffu;
      }
      if (g & 1) sum = bsum(hi[g] + hi[g - 1], sum);
    }
    const bool signed_out = OUT && __builtin_amdgcn_ballot_w64((sgn & 0x80808080u) != 0u) != 0ull;
    const u32 cnt = OUT ? (u32)(__builtin_popcount(pl[0]) + __builtin_popcount(pl[1])) : 64u;
    const u32 sc0 = scan_incl((sum & 0x1fffu) | (cnt << 19));
    const u32 tot0 = __builtin_amdgcn_readlane(sc0, 63);
    const u32 nnz = tot0 >> 19;
    pbase = (sc0 >> 19) - cnt;
    const float mean = (float)(tot0 & 0x7ffffu) * __builtin_amdgcn_rcpf((float)(nnz ? nnz : 1u));
    int b = (int)(mean + 0.97f);
    const float fr = mean - (float)(b - 1);
    const u32 c0 = rfl((u32)(fr < 0.08f) + (fr < 0.18f) + (fr < 0.225f) + (fr < 0.32f) + (fr < 0.46f) + (fr < 0.655f) + (fr < 0.92f));
    const u32 c1 = rfl(2u + (fr < 0.975f) + (fr < 0.815f) + (fr < 0.71f) + (fr < 0.655f) + (fr < 0.555f) + 2u * (fr < 0.41f) + (fr < 0.18f));
    b = b < 6 ? 6 : b > 126 ? 126 : b;
    const u32 base = rfl((u32)(b - 6));
    const u32 cneg = 0x80808080u - base * 0x01010101u;
    const u64 inv0 = len_table(c0), inv1 = len_table(c1);
    u32 inv0_lo = (u32)inv0, inv1_lo = (u32)inv1;
    asm volatile("" : "+v"(inv0_lo), "+v"(inv1_lo));
    u32 anybad = 0;
#pragma unroll
    for (int g = 0; g < 16; ++g) {
      const u32 e4 = (hi[g] + cneg) ^ 0x80808080u;
      anybad |= e4 & nzm[g];
      len[g] = bsel(top_mask(lo[g]), vperm((u32)(inv1 >> 32), inv1_lo, e4), vperm((u32)(inv0 >> 32), inv0_lo, e4)) & nzm[g];
      if (OUT) lo[g] = vperm(0u, lo[g], lutc[(pl[g >> 3] >> (4 * (g & 7))) & 15u]);
    }
    if (__builtin_amdgcn_ballot_w64((anybad & 0xf8f8f8f8u) != 0u) != 0ull) {
      u32 bp[2] = {0u, 0u};
#pragma unroll
      for (int g = 0; g < 16; ++g) {
        const u32 bad = ((hi[g] + cneg) ^ 0x80808080u) & 0xf8f8f8f8u & nzm[g];
        const u32 t = ((bad >> 1) + 0x7c7c7c7cu) & 0x80808080u;
        const u32 fm = (t << 1) - (t >> 7);
        len[g] = (len[g] & ~fm) | (t >> 7);
        bp[g >> 3] |= bsum(fm & 0x08040201u, 0u) << (4 * (g & 7));
      }
#pragma unroll
      for (int j = 0; j < 4; ++j) {
        const u32x4 t = {hi[4 * j], hi[4 * j + 1], hi[4 * j + 2], hi[4 * j + 3]};
        *(LAS u32x4*)(bld + lane * 64u + j * 16) = t;
      }
      CFENCE();
      u64 bm = ((u64)bp[1] << 32) | bp[0];
      const u64 plm = ((u64)pl[1] << 32) | pl[0];
      for (;;) {
        const u64 m = __builtin_amdgcn_ballot_w64(bm != 0ull);
        if (m == 0ull || nexc > 63u) break;
        if (bm != 0ull) {
          const u32 j = __builtin_ctzll(bm);
          bm &= bm - 1ull;
          const u32 slot = nexc + mbcnt(m);
          const u32 idx = pbase + (OUT ? (u32)__builtin_popcountll(plm & ((1ull << j) - 1ull)) : j);
          if (slot < 64u) exs[slot] = (idx << 8) | bld[lane * 64u + j];
        }
        nexc += __builtin_popcountll(m);
      }
      CFENCE();
    }
    u32 acc = 0;
#pragma unroll
    for (int g = 0; g < 16; ++g) acc += len[g];
    const u32 bits = bsum(acc, 0u);
    const u32 sc1 = scan_incl(bits);
    qbase = sc1 - bits;
    nw = (__builtin_amdgcn_readlane(sc1, 63) + 31u) >> 5;
    lbase = UN_OFF + 4u * nw + 4u * nexc;
    size4 = (lbase + nnz + 3u) >> 2;
    if (size4 >= 2048u || nexc > 63u || signed_out) size4 = 2048u;
    header = base | (c0 << 7) | (c1 << 11) | (nw << 15) | (nexc << 26);
  }
  lds_st(szw, wid * 4u, size4);
  if (size4 == 2048u) {
    const u32 go = ub + unit_offset(szw, wid, U) * 4u;
    const rsrc_t ws = make_rsrc(wsl, WS_NEED);
#pragma unroll
    for (int v = 0; v < 8; ++v) {
      const u32x4 q = {P[4 * v], P[4 * v + 1], P[4 * v + 2], P[4 * v + 3]};
      st16<17>(q, ws, lane * 16u + v * 1024u, go);
    }
    return;
  }
  {
    const u32x4 z = {0u, 0u, 0u, 0u};
    for (u32 o = HD_OFF; o < (OUT ? size4 * 4u : lbase); o += 1024u) *(LAS u32x4*)(bld + o + lane * 16u) = z;
  }
  CFENCE();
  u32 q = qbase + ((u32)(size_t)bld + UN_OFF) * 8u;
  u32 p = pbase + (u32)(size_t)bld + lbase;
  u32 ones = 0x01010101u;
  asm volatile("" : "+v"(ones));
#pragma unroll
  for (int g0 = 0; g0 < 16; g0 += 4) {
    u64 v[4], u[4];
    u32 da[4], ea[4];
#pragma unroll
    for (int i = 0; i < 4; ++i) {
      const int g = g0 + i;
      u32 t = len[g] + (len[g] << 8);
      asm volatile("" : "+v"(t));
      t += t << 16;
      const u32 tb = t + 0x1f1f1f1fu;
      const u32 one = OUT ? (nzm[g] & 0x01010101u) : ones;
      u32 b4, b5, b6;
      asm("v_lshlrev_b32_sdwa %0, %3, %4 dst_sel:DWORD dst_unused:UNUSED_PAD src0_sel:BYTE_0 src1_sel:BYTE_0\n\t"
          "v_lshlrev_b32_sdwa %1, %3, %4 dst_sel:DWORD dst_unused:UNUSED_PAD src0_sel:BYTE_1 src1_sel:BYTE_1\n\t"
          "v_lshlrev_b32_sdwa %2, %3, %4 dst_sel:DWORD dst_unused:UNUSED_PAD src0_sel:BYTE_2 src1_sel:BYTE_2\n\t"
          "v_or3_b32 %0, %0, %1, %2\n\t"
          "v_lshlrev_b32_sdwa %1, %3, %4 dst_sel:DWORD dst_unused:UNUSED_PAD src0_sel:BYTE_3 src1_sel:BYTE_3\n\t"
          "v_or_b32 %0, %0, %1"
          : "=&v"(b4), "=&v"(b5), "=&v"(b6) : "v"(tb), "v"(one));
      v[i] = (u64)b4 << (q & 31u);
      da[i] = (q >> 3) & ~3u;
      q += t >> 24;
      if (OUT) {
        u[i] = (u64)lo[g] << ((p & 3u) * 8u);
        ea[i] = p & ~3u;
        p += __builtin_popcount(one);
      }
    }
    SB();
#pragma unroll
    for (int i = 0; i < 4; ++i) {
      LAS u32* d = (LAS u32*)(size_t)da[i];
      __hip_atomic_fetch_or(d, (u32)v[i], __ATOMIC_RELAXED, __HIP_MEMORY_SCOPE_WAVEFRONT);
      __hip_atomic_fetch_or(d + 1, (u32)(v[i] >> 32), __ATOMIC_RELAXED, __HIP_MEMORY_SCOPE_WAVEFRONT);
      if (OUT) {
        LAS u32* e = (LAS u32*)(size_t)ea[i];
        __hip_atomic_fetch_or(e, (u32)u[i], __ATOMIC_RELAXED, __HIP_MEMORY_SCOPE_WAVEFRONT);
        __hip_atomic_fetch_or(e + 1, (u32)(u[i] >> 32), __ATOMIC_RELAXED, __HIP_MEMORY_SCOPE_WAVEFRONT);
      }
    }
    SB();
  }
  if (!OUT) {
#pragma unroll
    for (int g = 0; g < 16; ++g) *(LAS u32*)(bld + lbase + g * 256 + lane * 4u) = lo[g];
  }
  *(LAS u32*)(bld + lane * 4u) = pl[0];
  *(LAS u32*)(bld + 256 + lane * 4u) = pl[1];
  CFENCE();
  *(LAS u32*)(bld + HD_OFF) = header;
  if (lane < nexc) *(LAS u32*)(bld + UN_OFF + 4u * nw + 4u * lane) = exs[lane];
  CFENCE();
  const u32 go = ub + unit_offset(szw, wid, U) * 4u;
  u32 lim = go + size4 * 4u;
  const rsrc_t wc = make_rsrc(wsl, lim < WS_NEED ? lim : WS_NEED);
  constexpr int NB = OUT ? 3 : 6;
  for (u32 o = 0; o < size4 * 4u; o += 1024u * NB) {
    u32x4 t[NB];
#pragma unroll
    for (int j = 0; j < NB; ++j) t[j] = *(const LAS u32x4*)(bld + ((o + j * 1024u) & 8191u) + lane * 16u);
#pragma unroll
    for (int j = 0; j < NB; ++j) st16<17>(t[j], wc, lane * 16u + j * 1024u, go + o);
  }
  CFENCE();
}

__device__ __forceinline__ u32 parse4(u32& w, u32& q) {
  u32 r, p, t;
  asm("v_ffbl_b32 %3, %1\n\tv_add_u32 %4, -1, %1\n\tv_sub_u32 %0, %3, %2\n\tv_and_b32 %1, %4, %1\n\t"
      "v_ffbl_b32 %2, %1\n\tv_add_u32 %4, -1, %1\n\t"
      "v_sub_u32_sdwa %0, %2, %3 dst_sel:BYTE_1 dst_unused:UNUSED_PRESERVE src0_sel:DWORD src1_sel:DWORD\n\tv_and_b32 %1, %4, %1\n\t"
      "v_ffbl_b32 %3, %1\n\tv_add_u32 %4, -1, %1\n\t"
      "v_sub_u32_sdwa %0, %3, %2 dst_sel:BYTE_2 dst_unused:UNUSED_PRESERVE src0_sel:DWORD src1_sel:DWORD\n\tv_and_b32 %1, %4, %1\n\t"
      "v_ffbl_b32 %2, %1\n\tv_add_u32 %4, -1, %1\n\t"
      "v_sub_u32_sdwa %0, %2, %3 dst_sel:BYTE_3 dst_unused:UNUSED_PRESERVE src0_sel:DWORD src1_sel:DWORD\n\tv_and_b32 %1, %4, %1"
      : "=&v"(r), "+v"(w), "+v"(q), "=&v"(p), "=&v"(t));
  return r - 0x01010101u;
}

template <bool OUT, class LF>
__device__ __forceinline__ void decode_chunk(const LAS u8* ch, LAS u8* hs, const LAS u32x2* lutx, bool raw, u32 lane,
                                             u32 (&P)[32], LF landed) {
  if (raw) {
    landed(8192u);
#pragma unroll
    for (int v = 0; v < 8; ++v) {
      const volatile LAS u32* t = (const volatile LAS u32*)(ch + v * 1024 + lane * 16u);
      P[4 * v] = t[0]; P[4 * v + 1] = t[1]; P[4 * v + 2] = t[2]; P[4 * v + 3] = t[3];
    }
    return;
  }
  landed(UN_OFF);
  const u32 tr = rfl(*(const volatile LAS u32*)(ch + HD_OFF));
  const u32x2 pl = {*(const LAS u32*)(ch + lane * 4u), *(const LAS u32*)(ch + 256 + lane * 4u)};
  const u32 nw = (tr >> 15) & 0x7ffu, nexc = tr >> 26;
  const u64 dec0 = rank_table((tr >> 7) & 15u), dec1 = rank_table((tr >> 11) & 15u);
  const u32 base4 = (tr & 127u) * 0x01010101u;
  u32 t0_lo = (u32)dec0 + base4, t1_lo = (u32)dec1 + base4;
  const u32 t0_hi = (u32)(dec0 >> 32) + base4, t1_hi = (u32)(dec1 >> 32) + base4;
  asm volatile("" : "+v"(t0_lo), "+v"(t1_lo));
  u32 p = 0;
  if (OUT) {
    const u32 c = __builtin_popcount(pl.x) + __builtin_popcount(pl.y);
    p = scan_incl(c) - c;
  }
  u32 lbase = UN_OFF + 4u * nw + 4u * nexc;
  landed(lbase);
  lbase &= 8191u;
  const LAS u32* U = (const LAS u32*)(ch + UN_OFF);
  u32 run = 0;
  for (u32 w0 = 0; w0 < nw; w0 += 64u) {
    const u32 i = w0 + lane;
    u32 w = U[i], prev = U[(int)i - 1];
    w = i < nw ? w : 0u;
    prev = i > 0u ? prev : 0x80000000u;
    const u32 c = __builtin_popcount(w);
    const u32 incl = scan_incl(c);
    const u32 dst = (u32)(size_t)hs + ((run + incl - c) & 4095u);
    run += __builtin_amdgcn_readlane(incl, 63);
    u32 q = ~(u32)__builtin_clz(prev | 1u);
    u32 R[8];
#pragma unroll
    for (int j = 0; j < 6; ++j) R[j] = parse4(w, q);
    R[6] = R[7] = 0u;
    if (__builtin_amdgcn_ballot_w64(w != 0u) != 0ull) {
      R[6] = parse4(w, q);
      if (__builtin_amdgcn_ballot_w64(w != 0u) != 0ull) R[7] = parse4(w, q);
    }
    const u32 wb = (dst - 1u) & ~3u, ca = c + dst - wb, sh = 0u - dst;
    u32 W[8];
    u64 m[8];
#pragma unroll
    for (int j = 0; j < 8; ++j) {
      W[j] = __builtin_amdgcn_alignbyte(j < 7 ? R[j + 1] : 0u, R[j], sh);
      m[j] = __builtin_amdgcn_ballot_w64(4u * (j + 1) < ca);
    }
    const u64 m0 = __builtin_amdgcn_ballot_w64(c != 0u);
    const u32 r1 = R[0] >> 8;
    asm volatile("s_mov_b64 exec, %9\n\tds_write_b32 %0, %1 offset:4\n\ts_mov_b64 exec, %10\n\tds_write_b32 %0, %2 offset:8\n\t"
                 "s_mov_b64 exec, %11\n\tds_write_b32 %0, %3 offset:12\n\ts_mov_b64 exec, %12\n\tds_write_b32 %0, %4 offset:16\n\t"
                 "s_mov_b64 exec, %13\n\tds_write_b32 %0, %5 offset:20\n\ts_mov_b64 exec, %14\n\tds_write_b32 %0, %6 offset:24\n\t"
                 "s_mov_b64 exec, %15\n\tds_write_b32 %0, %7 offset:28\n\ts_mov_b64 exec, %16\n\tds_write_b32 %0, %8 offset:32\n\t"
                 "s_mov_b64 exec, %17\n\tds_write_b8 %18, %19\n\tds_write_b8 %18, %20 offset:1\n\tds_write_b8_d16_hi %18, %19 offset:2\n\t"
                 "s_mov_b64 exec, -1"
                 :: "v"(wb), "v"(W[0]), "v"(W[1]), "v"(W[2]), "v"(W[3]), "v"(W[4]), "v"(W[5]), "v"(W[6]), "v"(W[7]),
                    "s"(m[0]), "s"(m[1]), "s"(m[2]), "s"(m[3]), "s"(m[4]), "s"(m[5]), "s"(m[6]), "s"(m[7]), "s"(m0),
                    "v"(dst), "v"(R[0]), "v"(r1) : "memory");
  }
  CFENCE();
  if (lane < nexc) {
    const u32 r = U[nw + lane];
    hs[(r >> 8) & 4095u] = (u8)(r | 0x80u);
  }
  CFENCE();
  u32 s4[16], pp[16];
  u32x2 sel[16];
  if (OUT) {
#pragma unroll
    for (int g0 = 0; g0 < 16; g0 += 8) {
      u32x2 H[8];
      const LAS u8 *sa[8], *ha[8];
#pragma unroll
      for (int i = 0; i < 8; ++i) {
        const u32 wd = g0 ? pl.y : pl.x;
        const u32 m8 = (i == 0 ? wd << 3 : wd >> (4 * i - 3)) & 0x78u;
        pp[g0 + i] = p;
        sa[i] = (const LAS u8*)lutx + m8;
        ha[i] = hs + (p & ~3u);
        p += __builtin_popcount(m8);
      }
      SB();
#pragma unroll
      for (int i = 0; i < 8; ++i) {
        sel[g0 + i] = *(const LAS u32x2*)sa[i];
        H[i].x = ((const LAS u32*)ha[i])[0]; H[i].y = ((const LAS u32*)ha[i])[1];
      }
      SB();
#pragma unroll
      for (int i = 0; i < 8; ++i) s4[g0 + i] = __builtin_amdgcn_alignbyte(H[i].y, H[i].x, pp[g0 + i]);
    }
  } else {
#pragma unroll
    for (int j = 0; j < 4; ++j) {
      const u32x4 H = *(const LAS u32x4*)(hs + lane * 64u + j * 16);
      s4[4 * j] = H.x; s4[4 * j + 1] = H.y; s4[4 * j + 2] = H.z; s4[4 * j + 3] = H.w;
    }
  }
  u32 h0[16], h1[16];
#pragma unroll
  for (int g = 0; g < 16; ++g) { h0[g] = vperm(t0_hi, t0_lo, s4[g]); h1[g] = vperm(t1_hi, t1_lo, s4[g]); }
  if (nexc) {
#pragma unroll
    for (int g = 0; g < 16; ++g) {
      const u32 xm = top_mask(s4[g]), xv = s4[g] & 0x7f7f7f7fu;
      h0[g] = bsel(xm, xv, h0[g]);
      h1[g] = bsel(xm, xv, h1[g]);
    }
  }
  if (OUT) {
    u32 la[16];
#pragma unroll
    for (int g = 0; g < 16; ++g) {
      la[g] = (u32)(size_t)ch + lbase + (pp[g] & ~3u);
      asm volatile("" : "+v"(la[g]), "+v"(h0[g]), "+v"(h1[g]), "+v"(sel[g]));
    }
    landed(8192u);
    u32x2 L[16];
#pragma unroll
    for (int g = 0; g < 16; ++g) { L[g].x = ((const LAS u32*)(size_t)la[g])[0]; L[g].y = ((const LAS u32*)(size_t)la[g])[1]; }
    SB();
#pragma unroll
    for (int g = 0; g < 16; ++g) {
      const u32 l4 = __builtin_amdgcn_alignbyte(L[g].y, L[g].x, pp[g]);
      const u32 h4 = bsel(top_mask(l4), h1[g], h0[g]);
      P[2 * g] = vperm(h4, l4, sel[g].x);
      P[2 * g + 1] = vperm(h4, l4, sel[g].y);
    }
  } else {
#pragma unroll
    for (int g = 0; g < 16; ++g) {
      const u32 sg = ((g < 8 ? pl.x : pl.y) << (7 - (g & 7))) & 0x80808080u;
      h0[g] |= sg;
      h1[g] |= sg;
      asm volatile("" : "+v"(h0[g]), "+v"(h1[g]));
    }
    landed(8192u);
    u32 L[16];
#pragma unroll
    for (int g = 0; g < 16; ++g) L[g] = *(const LAS u32*)(ch + lbase + g * 256 + lane * 4u);
    SB();
#pragma unroll
    for (int g = 0; g < 16; ++g) {
      const u32 h4 = bsel(top_mask(L[g]), h1[g], h0[g]);
      P[2 * g] = vperm(h4, L[g], 0x05010400u);
      P[2 * g + 1] = vperm(h4, L[g], 0x07030602u);
    }
  }
}

__device__ __forceinline__ u32 pos_x(u32 g) { return g + (g < NF ? 0u : g < 16u ? 8u - NP : g < 32u ? 16u - NP : 32u - NP); }
__device__ __forceinline__ u32 pos_w(u32 g) { return g - NP + (g < 8u ? NF : g < 16u ? 16u : 32u); }
__device__ __forceinline__ u32 pos_code(u32 p) {
  if (p >= 96u) return 768u + (p - 96u) * 8u;
  if (p < NF) return 256u + p * 8u;
  if (p < NF + 8u - NP) return (p + NP - NF) * 8u;
  if (p < 24u - NP) return 256u + (p + NP - 8u) * 8u;
  if (p < 32u - NP) return (p + NP - 16u) * 8u;
  if (p < 48u - NP) return 256u + (p + NP - 16u) * 8u;
  if (p < 64u - NP) return (p + NP - 32u) * 8u;
  return 256u + (p + NP - 32u) * 8u;
}
__device__ __forceinline__ void code_inv(u32 e, u32& kind, u32& id) {
  kind = e < 256u ? 0u : e < 768u ? 1u : 2u;
  id = e < 256u ? e : e < 768u ? e - 256u : e - 768u;
}
__device__ __forceinline__ void w_stage(rsrc_t rw, LAS u8* stg, u32 dest, u32 id, u32 lane) {
  const u32 cb = id >> 4, kb = id & 15u;
  const u32 voff = (lane >> 4) * (NN * 2) + (lane & 15u) * 16u;
  u32 soff = rfl(kb * 32u * (NN * 2) + (dest * NSH + cb * 128u) * 2u), lds = rfl((u32)(size_t)stg);
  asm volatile("s_nop 4" : "+s"(soff), "+s"(lds));
#pragma unroll
  for (int u = 0; u < 8; ++u) dma16<0>(rw, lds + u * 1024u, voff, soff + u * 4u * (NN * 2));
}
__device__ __forceinline__ void w_fragments(const LAS u8* stg, u32 lane, u32 (&P)[32]) {
  const LAS u8* tb = stg + ((lane >> 4) * 8u + ((lane & 15u) >> 2)) * 256u + (lane & 3u) * 16u;
#pragma unroll
  for (int nf = 0; nf < 8; ++nf)
#pragma unroll
    for (int h = 0; h < 2; ++h) {
      const s16x4 r = __builtin_amdgcn_ds_read_tr16_b64_v4i16((LAS s16x4*)(tb + h * 1024 + (nf >> 1) * 64 + (nf & 1) * 8));
      const u32x2 r2 = __builtin_bit_cast(u32x2, r);
      P[4 * nf + 2 * h] = r2.x;
      P[4 * nf + 2 * h + 1] = r2.y;
    }
}
__device__ __forceinline__ void x_stage(rsrc_t rx, LAS u8* stg, u32 id, u32 lane) {
  const u32 rb = id >> 5, ksl = (id >> 2) & 7u, mq = id & 3u;
  const u32 voff = (lane >> 3) * (KSH * 2) + (lane & 7u) * 16u;
  u32 soff = rfl((rb * 256u + mq * 64u) * (KSH * 2) + ksl * 128u), lds = rfl((u32)(size_t)stg);
  asm volatile("s_nop 4" : "+s"(soff), "+s"(lds));
#pragma unroll
  for (int u = 0; u < 8; ++u) dma16<0>(rx, lds + u * 1024u, voff, soff + u * 8u * (KSH * 2));
}
__device__ __forceinline__ void x_fragments(const LAS u8* stg, u32 lane, u32 (&P)[32]) {
  const LAS u8* tb = stg + (lane & 15u) * 128u + (lane >> 4) * 16u;
#pragma unroll
  for (int v = 0; v < 8; ++v) {
    const u32x4 r = *(const LAS u32x4*)(tb + (v & 3) * 2048 + (v >> 2) * 64);
    P[4 * v] = r.x; P[4 * v + 1] = r.y; P[4 * v + 2] = r.z; P[4 * v + 3] = r.w;
  }
}
__device__ __forceinline__ bool task_of(u32 rnd, u32 gi, u32 wid, u32 rank, u32& isw, u32& gid, u32& dest, u32& cw) {
  const u32 x = gi & 7u;
  const u32 t = rnd < 2u ? gi : rnd == 2u ? 128u + (gi >> 3) * 7u + (x - 1u) : 240u + (gi >> 3) * 4u + (x - 4u);
  isw = 1; gid = 0; dest = rank; cw = wid;
  static_assert(NTASK >= 240 && NTASK <= 304, "tasks");
  if (rnd > 3u || (rnd == 2u && x == 0u) || (rnd == 3u && (x < 4u || t >= NTASK))) return false;
  if (rnd == 0u) { isw = 0; gid = (gi - (128u - NFU)) >> 2; cw = (gi & 3u) * 2u + wid; }
  else if (t < NP) { gid = t; }
  else {
    const u32 u = t + 7u * NP + NF;
    if (u < 64u + NF) gid = u - NF;
    else if (u < 80u) { isw = 0; gid = u - 64u; }
    else if (u < 144u) gid = u - 16u;
    else if (u < 160u) { isw = 0; gid = u - 128u; }
    else if (u < 288u) gid = u - 32u;
    else { isw = 0; gid = u - 256u; }
    if (isw) { dest = gid & 7u; gid >>= 3; }
  }
  return true;
}
__device__ __forceinline__ void st_coord(u32 seq, u32& i, u32& j) {
  if (seq >= 8u) { i = seq >> 2; j = seq & 3u; }
  else if (seq < 4u) { i = seq >> 1; j = seq & 1u; }
  else { i = seq & 1u; j = seq >> 1; }
}
__device__ __forceinline__ u32 tile_seq(u32 seq, u32 pos) {
  const u32 r = pos >> 2, c = pos & 3u, x = seq & 7u, b = (seq >> 3) * 128u;
  if (x == 0u) return b + (c >> 1) * 8u + r * 2u + (c & 1u);
  if (x == 1u) return b + 16u + c * 4u + r;
  if (x < 4u) return b + 32u + r * 8u + (x & 1u) * 4u + c;
  return b + (x >> 1) * 32u + c * 8u + (x & 1u) * 4u + r;
}
__device__ __forceinline__ void tile_inv(u32 ts, u32& rb, u32& cb) {
  const u32 t = ts & 127u;
  u32 seq, r, c;
  if (t < 16u) { seq = 0u; r = (t >> 1) & 3u; c = (t >> 3) * 2u + (t & 1u); }
  else if (t < 32u) { seq = 1u; c = (t >> 2) & 3u; r = t & 3u; }
  else if (t < 64u) { seq = 2u + ((t >> 2) & 1u); r = (t >> 3) & 3u; c = t & 3u; }
  else { seq = (t >> 5) * 2u + ((t >> 2) & 1u); c = (t >> 3) & 3u; r = t & 3u; }
  seq += (ts >> 7) * 8u;
  u32 si, sj;
  st_coord(seq, si, sj);
  rb = si * 4u + r;
  cb = sj * 4u + c;
}
__device__ __forceinline__ u32 flag_off(u32 src, u32 g) { return OFF_FLAG + (src * NGRP + g) * 16u; }
__device__ __forceinline__ void publish(LAS u8* smem, u32 rnd, char* pws, u32 rank, u32 g, u32 lane, bool sel) {
  WAIT_VM(0);
  asm volatile("" : "+v"(pws));
  u32 a = 0;
  if (lane == 0) a = __hip_atomic_fetch_add((LAS u32*)(smem + L_ARR + rnd * 4u), 1u, __ATOMIC_RELAXED, __HIP_MEMORY_SCOPE_WORKGROUP);
  if (rfl(a) == (rnd ? 7u : 1u)) {
    CFENCE();
    const u32x4 s0 = *(volatile LAS u32x4*)(smem + L_SZ + rnd * 32u), s1 = *(volatile LAS u32x4*)(smem + L_SZ + rnd * 32u + 16u);
    const u32 c16 = (lds_ld(smem, L_CALL) << 16) | 0x80000000u;
    const u32x4 rec = {s0.x | (s0.y << 12) | (s0.z << 24), c16 | (s0.z >> 8) | (s0.w << 4),
                       s1.x | (s1.y << 12) | (s1.z << 24), c16 | (s1.z >> 8) | (s1.w << 4)};
    if (sel) {
      if (rnd) {
        asm volatile("global_store_dwordx4 %0, %1, off sc0 sc1\n\ts_nop 1" :: "v"(pws + flag_off(rank, 32u + g)), "v"(rec) : "memory");
      } else {
        const u32x2 h = {rec.x | rec.z, rec.y | rec.w};
        asm volatile("global_store_dwordx2 %0, %1, off sc0 sc1\n\ts_nop 1" :: "v"(pws + flag_off(rank, 0u) + g * 8u), "v"(h) : "memory");
      }
    }
  }
}

__device__ __forceinline__ bool timed_out(const GAS u32* tmo, u32 call) {
  return rfl(__hip_atomic_load(tmo, __ATOMIC_RELAXED, __HIP_MEMORY_SCOPE_AGENT)) == call;
}

}

__global__ __launch_bounds__(NTHR, 2) void gemm_ar_kernel(Params prm) {
  __shared__ __attribute__((aligned(16))) u8 smem_raw[L_TOTAL];
  LAS u8* smem = (LAS u8*)smem_raw;
  const u32 tid = threadIdx.x, lane = tid & 63u, wid = rfl(tid >> 6);
  const u32 rank = (u32)prm.rank;
  char* wsl = (char*)prm.ws;
  const rsrc_t ws = make_rsrc(wsl, WS_NEED);
  GAS u32* tmo = (GAS u32*)(wsl + OFF_TMO);
  const u32 xcd = blockIdx.x & 7u, pos = blockIdx.x >> 3;
  DBG(const u64 tstart = now();)
  const rsrc_t rx = make_rsrc(prm.x, MM * KSH * 2), rw = make_rsrc(prm.w, KSH * NN * 2);
  const bool fast = blockIdx.x >= 128u - NFU && blockIdx.x < 128u && wid < 2u;
  if (fast) x_stage(rx, smem + wid * 18432u + 10240u, (blockIdx.x - (128u - NFU)) * 2u + wid, lane);
  const u32 cid = blockIdx.x - 128u;
  constexpr u32 PW = (NP * 8 + NL - 1) / NL;
  static_assert(PW <= 4, "push");
  const u32 pq = cid / 7u + (wid - 2u) * NL;
  const bool pusher = cid < 7u * NL && wid >= 2u && wid - 2u < PW && pq < NP * 8u;
  const u32 src = (rank + 1u + cid % 7u) & 7u;
  LAS u8* pstg = smem + (NREG - 1u - ((wid - 2u) >> 1)) * (REGK * 1024u) + (wid & 1u) * 8192u;
  if (pusher) w_stage(rw, pstg, src, pq, lane);
  DBG(const u64 tiss = now(); STAMPV(wsl, 0, tstart);)

  if (tid < (L_Z1 - L_Z0) / 4u) *(LAS u32*)(smem + L_Z0 + tid * 4u) = 0u;
  if (tid >= 496u) {
    const u32 m = tid - 496u;
    u32 cs = 0x0c0c0c0cu, r = 0;
    u64 xs = 0x0c0c0c0c0c0c0c0cull;
    for (u32 i = 0; i < 4; ++i)
      if ((m >> i) & 1u) {
        cs = (cs & ~(0xffu << (8 * r))) | (i << (8 * r));
        xs = (xs & ~(0xffffull << (16 * i))) | ((u64)(r | ((4u + r) << 8)) << (16 * i));
        ++r;
      }
    if (pos < 16u) *(LAS u32*)(smem + L_LUT + 4 * m) = cs;
    else *(LAS u64*)(smem + L_LUT + 8 * m) = xs;
  }
  GAS u32* callp = (GAS u32*)(wsl + OFF_CALL + blockIdx.x * 64u);

  if (pos < 16u) {
    LAS u8* bld = smem + wid * 18432u;
    LAS u8* stg = bld + 10240;
    LAS u32* exs = (LAS u32*)(smem + L_EXC + wid * 256u);
    const LAS u32* lutc = (const LAS u32*)(smem + L_LUT);
    char* pws = (char*)prm.peer[lane & 7u];
    const u32 prank = (rank + 1u + lane) & 7u;
    const u32 cv = __hip_atomic_load(callp, __ATOMIC_RELAXED, __HIP_MEMORY_SCOPE_AGENT);
    WAIT_LGKM0();
    __builtin_amdgcn_s_barrier();
    CFENCE();
    u32 call = 0;
    const u32 gi = blockIdx.x;
    u32 isw, gid, dest, cw;
    task_of(fast ? 0u : 1u, gi, wid, rank, isw, gid, dest, cw);
    if (!fast) {
      for (u32 i = SDELAY + (pos * SSTAG) / 4u; i > 0u; --i) __builtin_amdgcn_s_sleep(8);
      if (isw) w_stage(rw, stg, dest, gid * 8u + cw, lane);
      else x_stage(rx, stg, gid * 8u + cw, lane);
    }
    bool more = true;
    for (u32 rnd = fast ? 0u : 1u; more; ++rnd) {
      task_of(rnd, gi, wid, rank, isw, gid, dest, cw);
      const u32 id = gid * 8u + cw;
      u32 P[32];
      DBG(const u64 ti = tiss;)
      WAIT_VM(0);
      if (isw) w_fragments(stg, lane, P);
      else x_fragments(stg, lane, P);
      if (wid == 0u && call == 0u) {
        call = rfl(cv) + 1u;
        lds_st(smem, L_CALL, call);
        __hip_atomic_store(callp, call, __ATOMIC_RELAXED, __HIP_MEMORY_SCOPE_AGENT);
      }
      {
        u32 isw2, gid2, dest2, cw2;
        more = task_of(rnd + 1u, gi, wid, rank, isw2, gid2, dest2, cw2);
#pragma unroll
        for (int i = 0; i < 32; ++i) asm volatile("" : "+v"(P[i]));
        WAIT_LGKM0();
        if (more && rnd != 0u) {
          if (isw2) w_stage(rw, stg, dest2, gid2 * 8u + cw2, lane);
          else x_stage(rx, stg, gid2 * 8u + cw2, lane);
        }
      }
      DBG(for (int i = 0; i < 32; ++i) asm volatile("" : "+v"(P[i])); const u64 tl = now();)
      const bool send = !isw || dest != rank, keep = !isw || dest == rank;
      if (send)
        encode_chunk<false>(P, bld, exs, lutc, wsl, isw ? OFF_EW + (dest * 32u + gid) * 65536u : OFF_EX + gid * 65536u,
                            smem + L_SZ + rnd * 32u, cw, lane, (isw ? pos_w(gid) : pos_x(gid)) < (u32)PQUAD ? 2u : 4u);
      DBG(const u64 te = now();)
      if (send)
        publish(smem, rnd, pws, rank, rnd == 0u ? id >> 1 : isw ? pos_w(gid) : pos_x(gid), lane,
                      lane < 7u && (!isw || prank == dest));
      DBG(if (rnd == (fast ? 0u : 1u)) { STAMP(wsl, 14); STAMPV(wsl, 12, tl); STAMPV(wsl, 13, te); STAMPV(wsl, 15, ti); })
      if (rnd == 0u) {
        u32 isw2, gid2, dest2, cw2;
        task_of(1u, gi, wid, rank, isw2, gid2, dest2, cw2);
        if (isw2) w_stage(rw, stg, dest2, gid2 * 8u + cw2, lane);
        else x_stage(rx, stg, gid2 * 8u + cw2, lane);
      }
      if (keep) {
        u32 o;
        if (isw) o = OFF_BT + (id >> 4) * (1u << 20) + (rank * 16u + (id & 15u)) * 8192u;
        else o = OFF_AT + (id >> 5) * (2u << 20) + (rank * 8u + ((id >> 2) & 7u)) * 32768u + (id & 3u) * 4096u;
        const u32 hstride = isw ? 4096u : 16384u;
#pragma unroll
        for (int v = 0; v < 8; ++v) {
          const u32x4 q = {P[4 * v], P[4 * v + 1], P[4 * v + 2], P[4 * v + 3]};
          st16<16>(q, ws, lane * 16u + (v & 3) * 1024u, o + (v >> 2) * hstride);
        }
        WAIT_VM(0);
        if (lane == 0)
          __hip_atomic_fetch_add((GAS u32*)(wsl + OFF_CNT + (isw ? 16u + (id >> 4) : (id >> 5)) * 128u), 1u, __ATOMIC_RELAXED, __HIP_MEMORY_SCOPE_AGENT);
      }
    }
    STAMP(wsl, 1);
    const rsrc_t ro = make_rsrc(prm.out, (u32)MM * NN * 2u);
    const u32 wm = wid & 3u, wn = wid >> 2;
    for (u32 round = 0; round < 2u; ++round) {
      const u32 seq = round * 8u + xcd;
      u32 si, sj;
      st_coord(seq, si, sj);
      const u32 rb = si * 4u + (pos >> 2), cb = sj * 4u + (pos & 3u);
      f32x4 acc[4][4];
#pragma unroll
      for (int a = 0; a < 4; ++a)
#pragma unroll
        for (int c = 0; c < 4; ++c) acc[a][c] = (f32x4){0.f, 0.f, 0.f, 0.f};
      u32 dead = 0;
      DBG(u64 ck0 = 0;)
      for (u32 seg = 0; seg < 2u; ++seg) {
        if (wid == 0) {
          const GAS u32* cp = lane < 4u ? (const GAS u32*)(wsl + OFF_CNT + ((lane >> 1) * 32u + ((lane & 1u) ? 16u + cb : rb)) * 128u) : tmo;
          const u32 per = seg ? 7u : 1u, na = call * per * 32u, nb = call * per * 16u;
          const u64 t0 = now();
          u32 st = 0;
          for (;;) {
            const u32 v = __hip_atomic_load(cp, __ATOMIC_RELAXED, __HIP_MEMORY_SCOPE_SYSTEM);
            const u32 va = __builtin_amdgcn_readlane(v, seg * 2u), vb = __builtin_amdgcn_readlane(v, seg * 2u + 1u);
            if ((u32)__builtin_amdgcn_readlane(v, 4) == call) { st = 1; break; }
            if (va >= na && vb >= nb) {
              if ((u32)__builtin_amdgcn_readlane(v, 2) >= call * 224u && (u32)__builtin_amdgcn_readlane(v, 3) >= call * 112u) st = 2;
              break;
            }
            if (now() - t0 > DEADLINE) {
              __hip_atomic_store(tmo, call, __ATOMIC_RELAXED, __HIP_MEMORY_SCOPE_AGENT);
              st = 1;
              break;
            }
            if (va + per * 16u < na || vb + per * 8u < nb) __builtin_amdgcn_s_sleep(PSLOW);
            else __builtin_amdgcn_s_sleep(PFAST);
          }
          lds_st(smem, L_GST + (round * 2u + seg) * 4u, st);
        }
        asm volatile("s_waitcnt vmcnt(0) lgkmcnt(0)" ::: "memory");
        __builtin_amdgcn_s_barrier();
        CFENCE();
        const u32 st = lds_ld(smem, L_GST + (round * 2u + seg) * 4u);
        dead |= st & 1u;
        const u32 nks = seg ? 56u : st == 2u ? 64u : 8u;
        if (nks > 8u) STAMP(wsl, 2 + round * 3);
        DBG(if (nks > 8u) ck0 = __builtin_amdgcn_s_memtime();)
        u32 kn = rank * 8u + seg * 8u;
        if (nks == 64u) seg = 1u;
        const u32 ga = OFF_AT + rb * (2u << 20) + wid * 4096u, gb = OFF_BT + cb * (1u << 20) + wid * 2048u;
        const u32 la = (u32)(size_t)smem + wid * 4096u, lb = (u32)(size_t)smem + 32768u + wid * 2048u;
        const u32 vo = lane * 16u;
#define STG_A(I, BUF, KN) dma16<(I) * 1024>(ws, la + (BUF), vo, ga + ((KN) & 63u) * 32768u)
#define STG_B(I, BUF, KN) dma16<(I) * 1024>(ws, lb + (BUF), vo, gb + ((KN) & 63u) * 16384u)
        asm volatile("s_nop 4");
#pragma unroll
        for (int i = 0; i < 3; ++i) {
          STG_B(0, i * L_STAGE, kn); STG_B(1, i * L_STAGE, kn); STG_A(0, i * L_STAGE, kn);
          if (GV == 1 || i < 2) { STG_A(1, i * L_STAGE, kn); STG_A(2, i * L_STAGE, kn); STG_A(3, i * L_STAGE, kn); }
          ++kn;
        }
        const LAS u8* ra = smem + wm * 4096u + lane * 16u;
        const LAS u8* rbp = smem + 32768u + wn * 4096u + lane * 16u;
        bf16x8 a0[4], b0[4], a1[4], b1[4];
        if (GV == 1) WAIT_VM(12); else WAIT_VM(9);
        __builtin_amdgcn_s_barrier();
        CFENCE();
#pragma unroll
        for (int i = 0; i < 4; ++i) {
          b0[i] = *(const LAS bf16x8*)(rbp + i * 1024);
          a0[i] = *(const LAS bf16x8*)(ra + i * 1024);
        }
        u32 sbuf = 0, pbuf = 2u * L_STAGE;
#define MM_(FA, FB, A, C) acc[A][C] = __builtin_amdgcn_mfma_f32_16x16x32_bf16(FB[C], FA[A], acc[A][C], 0, 0, 0)
#define RD(DST, PTR) DST = *(const LAS bf16x8*)(PTR)
#define X1(MODE, I) if (GV == 2 && MODE <= 1) STG_A(I, pbuf, kn - 1u)
#define KSTEP(MODE) do { \
          const LAS u8* pa = ra + sbuf; const LAS u8* pb = rbp + sbuf; \
          __builtin_amdgcn_s_setprio(0); \
          SB(); MM_(a0, b0, 0, 0); SB(); RD(b1[0], pb + 8192); SB(); MM_(a0, b0, 0, 1); SB(); RD(a1[0], pa + 16384); X1(MODE, 1); \
          SB(); MM_(a0, b0, 0, 2); SB(); RD(b1[1], pb + 9216); SB(); MM_(a0, b0, 0, 3); SB(); RD(b1[2], pb + 10240); \
          SB(); MM_(a0, b0, 1, 0); SB(); RD(b1[3], pb + 11264); SB(); MM_(a0, b0, 1, 1); SB(); RD(a1[1], pa + 17408); \
          SB(); MM_(a0, b0, 1, 2); SB(); RD(a1[2], pa + 18432); X1(MODE, 2); SB(); MM_(a0, b0, 1, 3); SB(); RD(a1[3], pa + 19456); \
          SB(); MM_(a0, b0, 2, 0); MM_(a0, b0, 2, 1); MM_(a0, b0, 2, 2); MM_(a0, b0, 2, 3); SB(); X1(MODE, 3); SB(); \
          MM_(a0, b0, 3, 0); MM_(a0, b0, 3, 1); MM_(a0, b0, 3, 2); MM_(a0, b0, 3, 3); SB(); \
          const u32 nbuf = sbuf == 2u * L_STAGE ? 0u : sbuf + L_STAGE; \
          if (MODE < 3) { \
            if (MODE < 2) asm volatile("s_waitcnt vmcnt(6) lgkmcnt(0)" ::: "memory"); \
            else asm volatile("s_waitcnt vmcnt(0) lgkmcnt(0)" ::: "memory"); \
            __builtin_amdgcn_s_barrier(); CFENCE(); \
            pa = ra + nbuf; pb = rbp + nbuf; \
          } \
          __builtin_amdgcn_s_setprio(1); \
          if (GV == 1) { \
          SB(); MM_(a1, b1, 0, 0); SB(); if (MODE < 3) RD(b0[0], pb); if (MODE == 0) STG_B(0, sbuf, kn); SB(); MM_(a1, b1, 0, 1); SB(); if (MODE < 3) RD(a0[0], pa); \
          SB(); MM_(a1, b1, 0, 2); SB(); if (MODE < 3) RD(b0[1], pb + 1024); SB(); MM_(a1, b1, 0, 3); SB(); if (MODE < 3) RD(b0[2], pb + 2048); if (MODE == 0) STG_B(1, sbuf, kn); \
          SB(); MM_(a1, b1, 1, 0); SB(); if (MODE < 3) RD(b0[3], pb + 3072); SB(); MM_(a1, b1, 1, 1); SB(); if (MODE < 3) RD(a0[1], pa + 1024); \
          SB(); MM_(a1, b1, 1, 2); SB(); if (MODE < 3) RD(a0[2], pa + 2048); if (MODE == 0) STG_A(0, sbuf, kn); SB(); MM_(a1, b1, 1, 3); SB(); if (MODE < 3) RD(a0[3], pa + 3072); \
          SB(); MM_(a1, b1, 2, 0); SB(); MM_(a1, b1, 2, 1); SB(); if (MODE == 0) STG_A(1, sbuf, kn); \
          SB(); MM_(a1, b1, 2, 2); SB(); MM_(a1, b1, 2, 3); SB(); MM_(a1, b1, 3, 0); SB(); if (MODE == 0) STG_A(2, sbuf, kn); \
          SB(); MM_(a1, b1, 3, 1); MM_(a1, b1, 3, 2); MM_(a1, b1, 3, 3); SB(); if (MODE == 0) STG_A(3, sbuf, kn); SB(); \
          } else { \
          SB(); MM_(a1, b1, 0, 0); SB(); if (MODE < 3) RD(b0[0], pb); SB(); MM_(a1, b1, 0, 1); SB(); if (MODE < 3) RD(a0[0], pa); if (MODE == 0) STG_B(0, sbuf, kn); \
          SB(); MM_(a1, b1, 0, 2); SB(); if (MODE < 3) RD(b0[1], pb + 1024); SB(); MM_(a1, b1, 0, 3); SB(); if (MODE < 3) RD(b0[2], pb + 2048); \
          SB(); MM_(a1, b1, 1, 0); SB(); if (MODE < 3) RD(b0[3], pb + 3072); SB(); MM_(a1, b1, 1, 1); SB(); if (MODE < 3) RD(a0[1], pa + 1024); \
          SB(); MM_(a1, b1, 1, 2); SB(); if (MODE < 3) RD(a0[2], pa + 2048); if (MODE == 0) STG_B(1, sbuf, kn); SB(); MM_(a1, b1, 1, 3); SB(); if (MODE < 3) RD(a0[3], pa + 3072); \
          SB(); MM_(a1, b1, 2, 0); MM_(a1, b1, 2, 1); MM_(a1, b1, 2, 2); MM_(a1, b1, 2, 3); SB(); if (MODE == 0) STG_A(0, sbuf, kn); SB(); \
          MM_(a1, b1, 3, 0); MM_(a1, b1, 3, 1); MM_(a1, b1, 3, 2); MM_(a1, b1, 3, 3); SB(); \
          } \
          ++kn; pbuf = sbuf; sbuf = nbuf; \
        } while (0)
        for (u32 ks = 3; ks < nks; ++ks) KSTEP(0);
        KSTEP(1);
        KSTEP(2);
        KSTEP(3);
        __builtin_amdgcn_s_setprio(0);
      }
      STAMP(wsl, 3 + round * 3);
      DBG(STAMPV(wsl, 8 + round, __builtin_amdgcn_s_memtime() - ck0);)
      u32 P[32];
#pragma unroll
      for (int a = 0; a < 4; ++a)
#pragma unroll
        for (int c = 0; c < 4; ++c)
#pragma unroll
          for (int j = 0; j < 2; ++j) {
            const f32x2 v = {__builtin_elementwise_maximum(acc[a][c][2 * j], 0.f), __builtin_elementwise_maximum(acc[a][c][2 * j + 1], 0.f)};
            const u32 pk = __builtin_bit_cast(u32, __builtin_convertvector(v, bf16x2));
            P[a * 8 + c * 2 + j] = dead ? 0x7fc07fc0u : pk;
          }
      WAIT_LGKM0();
      __builtin_amdgcn_s_barrier();
      CFENCE();
      const u32 ts = tile_seq(seq, pos);
      DBG(const u64 te0 = now();)
      encode_chunk<true>(P, bld, exs, lutc, wsl, OFF_EO + ts * 65536u, smem + L_SZ + (4u + round) * 32u, wid, lane,
                         ts - RB0 < TAILT ? 1u : ts >= (u32)OCT0 && ts < (u32)OCT1 ? 8u : 4u);
      DBG(const u64 te1 = now();)
      publish(smem, 4u + round, pws, rank, 96u + ts, lane, lane < 7u);
      DBG(STAMPV(wsl, 10, te1 - te0); STAMPV(wsl, 11, now() - te1);)
      STAMP(wsl, 4 + round * 3);
      {
        const u32 voff = (lane & 15u) * (NN * 2u) + (lane >> 4) * 16u;
        const u32 soff = (rb * 256u + wm * 64u) * (NN * 2u) + (rank * NSH + cb * 128u + wn * 64u) * 2u;
#pragma unroll
        for (int v = 0; v < 8; ++v) {
          const u32x4 q = {P[4 * v], P[4 * v + 1], P[4 * v + 2], P[4 * v + 3]};
          st16<OAUX>(q, ro, voff + (v & 1) * 64u, soff + (v >> 1) * 16u * (NN * 2u));
        }
      }
    }
  } else {
    if (cid >= 7u * NL) return;
    char* wsp = (char*)prm.peer[cid % 7u];
    WAIT_LGKM0();
    __builtin_amdgcn_s_barrier();
    CFENCE();
    if (pusher) {
      u32 P[32];
      WAIT_VM(0);
      DBG(STAMP(wsl, 8);)
      w_fragments(pstg, lane, P);
      const rsrc_t rp = make_rsrc(wsp, WS_NEED);
      const u32 o = OFF_BT + (pq >> 4) * (1u << 20) + (rank * 16u + (pq & 15u)) * 8192u;
#pragma unroll
      for (int v = 0; v < 8; ++v) {
        const u32x4 q = {P[4 * v], P[4 * v + 1], P[4 * v + 2], P[4 * v + 3]};
        st16<17>(q, rp, lane * 16u + v * 1024u, o);
        asm volatile("s_waitcnt vmcnt(" STR(PWIN) ")" ::: "memory");
      }
      WAIT_VM(0);
      if (lane == 0)
        __hip_atomic_fetch_add((GAS u32*)(wsp + OFF_CNT + (48u + (pq >> 4)) * 128u), 1u, __ATOMIC_RELAXED, __HIP_MEMORY_SCOPE_SYSTEM);
      DBG(STAMP(wsl, 9);)
    }
    if (wid >= 2u && wid < 6u && lane == 0)
      __hip_atomic_fetch_add((LAS u32*)(smem + L_FREE + (NREG - 1u - ((wid - 2u) >> 1)) * 4u), 1u, __ATOMIC_RELAXED, __HIP_MEMORY_SCOPE_WORKGROUP);
    if (wid == 1u) {
      const u32 call = rfl(__hip_atomic_load(callp, __ATOMIC_RELAXED, __HIP_MEMORY_SCOPE_AGENT)) + 1u;
      lds_st(smem, L_CALL, call);
      __hip_atomic_store(callp, call, __ATOMIC_RELAXED, __HIP_MEMORY_SCOPE_AGENT);
      char* tkb = wsl + OFF_TK + (call & 1u) * 1024u;
      if (cid < 7u) __hip_atomic_store((GAS u64*)(wsl + OFF_TK + (~call & 1u) * 1024u + src * 128u), 0ull, __ATOMIC_RELAXED, __HIP_MEMORY_SCOPE_AGENT);
      u32 nreq = 0, njob = 0, pieces = 0, region = 0, dead = 0;
      u32 cum = (lane == NREG - 1u || lane == NREG - 2u) ? 2u : 0u;
      u32 wbase = 0;
      u64 win = 0;
      auto lane64 = [](u64 v, u32 l) { return ((u64)(u32)__builtin_amdgcn_readlane((u32)(v >> 32), l) << 32) | (u32)__builtin_amdgcn_readlane((u32)v, l); };
      u32 done = 0, late = 0, hsrc = 8u, tried = 0, scans = 0;
      u32 usable = 0;
      u64 win2 = 0;
      const u32 tag = (call & 0xffffu) | 0x8000u, others = 255u & ~(1u << rank) & ~(1u << src);
      auto remain = [](u64 w) {
        const u32 t = (u32)(w >> 36);
        u32 r = t < (u32)NTK ? 2u * (NTK - t) : 0u;
#pragma unroll
        for (u32 i = 0; i < 6u; ++i) {
          const u32 a = t > NTK + i ? (t - NTK - i + 5u) / 6u : 0u, u = ((u32)(w >> (6u * i)) & 63u) + a;
          r += u < NTS ? NTS - u + (a < NQD ? NQD - a : 0u) : 0u;
        }
        return r;
      };
      __builtin_amdgcn_s_sleep(FIRSTSLEEP);
      DBG(u64 tw[4] = {};)
      for (;;) {
        u32 t = 0xffffffffu, g = 0, c0 = 0, cnt = 2, sl = 0, st = 0;
        bool tail = true, got = false;
        const u32 jsrc = hsrc != 8u ? hsrc : src;
        u32 f = 0;
        if (jsrc != src) {
          sl = (src - jsrc - 1u) & 7u;
          f = (sl + 6u - ((rank - jsrc - 1u) & 7u)) % 7u;
        }
        const u64 inc = jsrc != src ? 1ull << (6u * f) : done ? 0ull : 1ull << 36;
        u64 w, wall = 0;
        if (late) {
          u32 rf = 0;
          const u32 l = lane >> 3;
          if (usable != others) {
            rf = __hip_atomic_load((const GAS u32*)(wsl + OFF_RF + src * 128u + l * 4u), __ATOMIC_RELAXED, __HIP_MEMORY_SCOPE_SYSTEM);
            win2 = __hip_atomic_load((const GAS u64*)(wsl + OFF_FLAG) + l * (2u * NGRP) + 64u + 2u * (96u + RB0 + ((src - l - 1u) & 7u) * SLICE) + (lane & 7u),
                                     __ATOMIC_RELAXED, __HIP_MEMORY_SCOPE_SYSTEM);
          }
          const u64 li = lane == jsrc ? inc : 0ull;
          wall = draw64x8(tkb + (lane & 7u) * 128u, li);
          w = lane64(wall, jsrc);
          wall += li;
          if (usable != others) {
            const u64 bad = __builtin_amdgcn_ballot_w64(rf != call || ((lane & 7u) < 2u * SLICE && (u32)(win2 >> 48) != tag));
            for (u32 i = 0; i < 8u; ++i) usable |= ((bad >> (i * 8u)) & 255u) ? 0u : (1u << i) & others;
          }
        } else {
          w = draw64(tkb + src * 128u, inc);
        }
        if (jsrc != src) {
          const u32 h = (u32)(w >> (6u * f)) & 63u, tt = (u32)(w >> 36);
          const u32 used = tt > NTK + f ? (tt - NTK - f + 5u) / 6u : 0u;
          if (h + used + 1u >= NTS) tried |= 1u << jsrc;
          got = h + used < NTS;
          st = NTS - 1u - h;
        } else if (!done) {
          t = (u32)(w >> 36);
          got = true;
          tail = t >= TK_F && (t < TK_G || t >= (u32)NTK);
          if (t < TK_B) { g = t >> 2; c0 = (t & 3u) * 2u; }
          else if (t < NTI) { g = PQUAD + ((t - TK_B) >> 1); c0 = (t & 1u) * 4u; cnt = 4; }
          else if (t < TK_D) { g = 96u + ((t - NTI) >> 1); c0 = (t & 1u) * 4u; cnt = 4; }
          else if (t < TK_E) { g = 96u + OCT0 + (t - TK_D); c0 = 0; cnt = 8; }
          else if (t < TK_F) { g = 96u + OCT1 + ((t - TK_E) >> 1); c0 = ((t - TK_E) & 1u) * 4u; cnt = 4; }
          else if (t < TK_G) { sl = (rank - src - 1u) & 7u; }
          else if (t < (u32)NTK) { g = 96u + RB0 + TAILT + ((t - TK_G) >> 1); c0 = ((t - TK_G) & 1u) * 4u; cnt = 4; }
          else {
            const u32 d = t - NTK, fd = d % 6u;
            st = d / 6u;
            sl = (((rank - src - 1u) & 7u) + 1u + fd) % 7u;
            got = st + ((u32)(w >> (6u * fd)) & 63u) < NTS;
            if (!got && remain(w + inc) == 0u) done = 1;
          }
        }
        hsrc = 8u;
        if (late && !dead) {
          const u32 left = remain(wall), own = __builtin_amdgcn_readlane(left, src);
          if (own == 0u) done = 1;
          u32 best = done ? MINHELP - 1u : own + GAP0 + GAPC * (cid / 7u) - 1u, later = 0;
          for (u32 i = 0; i < 8u; ++i) {
            const u32 v = __builtin_amdgcn_readlane(left, i);
            if (((others & ~tried) >> i) & 1u) {
              if (!((usable >> i) & 1u)) later |= v >= (u32)WAITHELP;
              else if (v > best) { best = v; hsrc = i; }
            }
          }
          if (done && !got && hsrc == 8u) {
            if (!later || ++scans > 100u) break;
            __builtin_amdgcn_s_sleep(32);
          }
        } else if (done && !got && (dead || cid / 7u >= (u32)NHELP)) {
          break;
        }
        if (done || (t >= (u32)(NTK - BAL0) && cid / 7u < (u32)NBAL)) late = 1;
        if (!got) continue;
        u32 sc = 0;
        if (tail) {
          if (t < TK_G) { sc = 4u * (t - TK_F); cnt = 4; }
          else if (st < NQD) { sc = 4u * st; cnt = 4; }
          else sc = 2u * (st + NQD);
          g = 96u + RB0 + sl * SLICE + (sc >> 3);
          c0 = sc & 7u;
        }
        DBG(if (jsrc != src) tw[1] += 100;)
        u32 kind, id;
        const u32 code = pos_code(g);
        code_inv(code, kind, id);
        const u32 ub = jsrc != src ? OFF_RL + (jsrc * 32u + sc) * 8192u
                                   : (kind == 0 ? OFF_EW + rank * (32u * 65536u) : kind == 1 ? OFF_EX : OFF_EO) + (id + c0) * 8192u;
        const u32 keep = tail && t < (u32)NTK ? 1u : 0u;
        const u32 hx = jsrc * (2u * NGRP) + (t < NFU ? t : 64u + 2u * g + (c0 >> 2)), two = cnt >> 3;
        u64 rec = 0, rec2 = 0;
        if (jsrc != src) rec = rec2 = lane64(win2, jsrc * 8u + (sc >> 2));
        else if (hx - wbase < 63u) { rec = lane64(win, hx - wbase); rec2 = lane64(win, hx - wbase + two); }
        if ((u32)(rec >> 48) != tag || (u32)(rec2 >> 48) != tag) {
          const u64 t0 = now();
          const GAS u64* fp = (const GAS u64*)(wsl + OFF_FLAG) + hx;
          wbase = tail ? jsrc * (2u * NGRP) + 64u + 2u * (96u + RB0) : hx;
          DBG(u32 polls = 0;)
          if (!dead) win = __hip_atomic_load((const GAS u64*)(wsl + OFF_FLAG) + wbase + lane, __ATOMIC_RELAXED, __HIP_MEMORY_SCOPE_SYSTEM);
          else win = 0;
          rec = lane64(win, hx - wbase);
          rec2 = lane64(win, hx - wbase + two);
          for (u32 it = 0; !dead && ((u32)(rec >> 48) != tag || (u32)(rec2 >> 48) != tag); ++it) {
            DBG(++polls;)
            if ((it & 7u) == 7u && (timed_out(tmo, call) || now() - t0 > DEADLINE)) {
              __hip_atomic_store(tmo, call, __ATOMIC_RELAXED, __HIP_MEMORY_SCOPE_AGENT);
              dead = 1;
            }
            if (nreq != 0u) __builtin_amdgcn_s_sleep(FSLEEP);
            const u64 f = __hip_atomic_load(fp + (lane & 1u), __ATOMIC_RELAXED, __HIP_MEMORY_SCOPE_SYSTEM);
            rec = lane64(f, 0u);
            rec2 = lane64(f, two);
          }
          if (dead) rec = rec2 = 0;
          DBG(if (polls && nreq > 0) tw[t < NTI ? 0 : 2] += now() - t0;)
        }
        auto sz = [&](u32 i) { const u32 v = (u32)(((i & 4u) ? rec2 : rec) >> (12u * (i & 3u))) & 4095u; return v > 2048u ? 2048u : v; };
        u32 tot = 0, big = 0;
        for (u32 i = c0; i < c0 + cnt; ++i) { tot += sz(i); big = sz(i) > big ? sz(i) : big; }
        if (pieces - lds_ld(smem, L_LANDED) > (u32)THRESH) {
          const u64 t0 = now();
          while (pieces - lds_ld(smem, L_LANDED) > (u32)THRESH && now() - t0 < 4 * DEADLINE) __builtin_amdgcn_s_sleep(DSLEEP);
        }
        auto post_req = [&](u32 go, u32 kb, u32 len, u32 need) {
          lds_st(smem, L_REQ + (nreq % NQ) * 8u + 4u, go | (region << 27) | (region * REGK + kb));
          lds_st(smem, L_REQ + (nreq % NQ) * 8u, ((nreq + 1u) & 255u) | (((len + 3u) >> 2) << 8) | (need << 19));
          ++nreq;
        };
        auto post_job = [&](u32 e, u32 v, u32 off, u32 nh, u32 na) {
          lds_st(smem, L_JOB + (njob % NJ) * 8u + 4u, e | (off << 12) | ((v == 0u ? 2u : v >> 11) << 25) | (jsrc << 27) | (keep << (v > 1024u ? 31 : 30)));
          lds_st(smem, L_JOB + (njob % NJ) * 8u, ((njob + 1u) & 255u) | (region << 8) | ((na - nh) << 11) | (nh << 16));
          ++njob;
        };
        if (tail && big <= 1024u && !dead) {
          const u32 need = __builtin_amdgcn_readlane(cum, region);
          for (u32 j = 0; j < cnt; ++j) post_req(ub + j * 8192u, j * 4u, sz(c0 + j) < 256u ? sz(c0 + j) : 256u, need);
          u32 na = pieces + cnt;
          for (u32 j = 0; j < cnt; ++j) {
            const u32 v = sz(c0 + j);
            if (v > 256u) {
              post_req(ub + j * 8192u + 1024u, j * 4u + 1u, v - 256u, need);
              na += (v - 1u) >> 8;
            }
            post_job(code + c0 + j, v, j * 1024u, pieces + j + 1u, na);
          }
          pieces = na;
          cum += lane == region ? cnt : 0u;
          region = region == NREG - 1u ? 0u : region + 1u;
        } else {
          const u32 take = !tail && tot <= REGK * 256u ? cnt : 1u, h4 = kind == 2u ? HEAD_OUT / 4u : HEAD_IN / 4u;
          u32 pre = 0;
          for (u32 i = c0; i < c0 + cnt; i += take) {
            const u32 lead = pre & 31u;
            post_req(ub + (pre - lead) * 4u, 0u, lead + (take == cnt ? tot : sz(i)), __builtin_amdgcn_readlane(cum, region));
            u32 off = lead;
            for (u32 j = 0; j < take; ++j) {
              const u32 v = sz(i + j), nh = (off + (v < h4 ? v : h4) + 255u) >> 8, na = (off + v + 255u) >> 8;
              post_job(code + i + j, v, off, pieces + (nh ? nh : 1u), pieces + (na ? na : 1u));
              off += v;
            }
            cum += lane == region ? take : 0u;
            const u32 np = (off + 255u) >> 8;
            pieces += np ? np : 1u;
            pre += tail ? 2048u : off - lead;
            region = region == NREG - 1u ? 0u : region + 1u;
          }
        }
      }
      lds_st(smem, L_JEND, njob + 1u);
      lds_st(smem, L_REND, nreq + 1u);
      DBG(for (int i = 0; i < 4; ++i) STAMPV(wsl, 10 + i, tw[i]);)
    } else if (wid == 0) {
      const rsrc_t rp = make_rsrc(wsp, WS_NEED);
      const u32 lbase = (u32)(size_t)smem;
      u32 issued = 0, pub = 0;
      DBG(u64 tw[4] = {}; u32 seen = 0;)
      for (u32 n = 0;; ++n) {
        u32 rq, g1;
        auto ready = [&]() {
          rq = lds_ld(smem, L_REQ + (n % NQ) * 8u);
          g1 = lds_ld(smem, L_REQ + (n % NQ) * 8u + 4u);
          return (rq & 255u) == ((n + 1u) & 255u) && lds_ld(smem, L_FREE + ((g1 >> 27) & 7u) * 4u) >= (rq >> 19);
        };
        bool stop = false;
        if (!ready()) {
          const u64 t0 = now();
          DBG(const bool wf = (rq & 255u) != ((n + 1u) & 255u);)
          for (;;) {
            if (issued > pub) {
              switch (issued - pub) {
                case 2: WAIT_VM(1); break;
                case 3: WAIT_VM(2); break;
                case 4: WAIT_VM(3); break;
                case 5: WAIT_VM(4); break;
                case 6: WAIT_VM(5); break;
                case 7: WAIT_VM(6); break;
                case 8: WAIT_VM(7); break;
                default: WAIT_VM(0); break;
              }
              ++pub;
              lds_st(smem, L_LANDED, pub);
            } else {
              const u32 ne = lds_ld(smem, L_REND);
              if ((ne != 0u && n + 1u >= ne) || now() - t0 > 4 * DEADLINE) { stop = true; break; }
              __builtin_amdgcn_s_sleep(1);
            }
            if (ready()) break;
          }
          DBG(if (n > 0 && !stop) { u64 dt = now() - t0; if (!(seen & 16)) { if (wf) tw[0] += dt; else tw[1] += dt; } else { if (wf) tw[2] += dt; else tw[3] += dt; } })
        }
        if (stop) break;
        const u32 go = g1 & 0x07ffff80u, tot = (rq >> 8) & 2047u;
        u32 np = (tot + 63u) >> 6;
        np = np ? (np > (u32)REGK ? (u32)REGK : np) : 1u;
        for (u32 pc = 0; pc < np; ++pc) {
          const u32 o16 = pc * 64u + lane;
          const u32 vo = o16 < tot ? o16 * 16u : 0x80000000u, ld = lbase + ((g1 & 127u) + pc) * 1024u;
          asm volatile("s_mov_b32 m0, %0\n\ts_nop 3\n\ts_waitcnt vmcnt(" STR(WINDOW) " - 1)\n\tbuffer_load_dwordx4 %1, %2, %3 offen sc0 sc1 lds"
                       :: "s"(ld), "v"(vo), "s"(rp), "s"(go) : "memory");
          if (issued >= pub + WINDOW) { pub = issued - (WINDOW - 1u); lds_st(smem, L_LANDED, pub); }
          ++issued;
        }
      }
      STAMP(wsl, 3);
      DBG(for (int i = 0; i < 4; ++i) STAMPV(wsl, 10 + i, tw[i]);)
    } else {
      const rsrc_t ro = make_rsrc(prm.out, (u32)MM * NN * 2u);
      LAS u8* hs = smem + L_HS + (wid - 2u) * L_HS_STRIDE;
      const LAS u32x2* lutx = (const LAS u32x2*)(smem + L_LUT);
      DBG(u64 td[4] = {};)
      for (u32 n = wid - 2u;; n += NDEC) {
        bool stop = false;
        u32 j0;
        {
          const u64 t0 = now();
          for (;;) {
            j0 = lds_ld(smem, L_JOB + (n % NJ) * 8u);
            if ((j0 & 255u) == ((n + 1u) & 255u)) break;
            const u32 ne = lds_ld(smem, L_JEND);
            if (ne != 0u && n + 1u >= ne) { stop = true; break; }
            if (now() - t0 > 5 * DEADLINE) {
              if (lane == 0) {
                __hip_atomic_store(tmo, lds_ld(smem, L_CALL), __ATOMIC_RELAXED, __HIP_MEMORY_SCOPE_AGENT);
                ((GAS unsigned short*)prm.out)[src * NSH] = 0x7fc0;
              }
              stop = true;
              break;
            }
            __builtin_amdgcn_s_sleep(DSLEEP);
          }
        }
        if (stop) break;
        CFENCE();
        DBG(u64 td0 = now(); if (n == wid - 2u) STAMP(wsl, 1);)
        const u32 j1 = lds_ld(smem, L_JOB + (n % NJ) * 8u + 4u);
        const u32 region = (j0 >> 8) & 7u;
        const LAS u8* ch = smem + region * (REGK * 1024u) + ((j1 >> 12) & 8191u) * 4u;
        const bool dead = ((j1 >> 25) & 3u) == 2u, raw = ((j1 >> 25) & 3u) == 1u;
        const u32 jsrc = (j1 >> 27) & 7u, keep = j1 >> 30;
        u32 kind, id;
        code_inv(j1 & 4095u, kind, id);
        bool late = false;
        auto landed = [&](u32 bytes) {
          const u32 need = (j0 >> 16) + (bytes <= (kind == 2u ? HEAD_OUT : HEAD_IN) ? 0u : (j0 >> 11) & 31u);
          if (lds_ld(smem, L_LANDED) < need) {
            const u64 t0 = now();
            while (lds_ld(smem, L_LANDED) < need) {
              if (now() - t0 > 5 * DEADLINE) { late = true; break; }
              __builtin_amdgcn_s_sleep(DSLEEP);
            }
          }
          CFENCE();
          DBG(if (bytes == 8192u) td0 = now();)
        };
        u32 P[32];
        if (dead) {
          landed(0u);
        } else if (kind == 2) {
          decode_chunk<true>(ch, hs, lutx, raw, lane, P, landed);
        } else {
          decode_chunk<false>(ch, hs, lutx, raw, lane, P, landed);
        }
        if (dead || late) {
#pragma unroll
          for (int i = 0; i < 32; ++i) P[i] = 0x7fc07fc0u;
        }
#pragma unroll
        for (int i = 0; i < 32; ++i) asm volatile("" : "+v"(P[i]));
        if (keep) {
          const u32 o = OFF_RL + (jsrc * 32u + (id - RB0 * 8u) % (SLICE * 8u)) * 8192u;
          for (u32 h = 0; h < keep; ++h) {
            u32x4 q[4];
#pragma unroll
            for (int v = 0; v < 4; ++v) q[v] = *(const LAS u32x4*)(ch + h * 4096u + v * 1024 + lane * 16u);
#pragma unroll
            for (int v = 0; v < 4; ++v) st16<17>(q[v], ws, lane * 16u + v * 1024u, o + h * 4096u);
          }
        }
        WAIT_LGKM0();
        if (lane == 0) __hip_atomic_fetch_add((LAS u32*)(smem + L_FREE + region * 4u), 1u, __ATOMIC_RELAXED, __HIP_MEMORY_SCOPE_WORKGROUP);
        if (keep) {
          const u32 call = lds_ld(smem, L_CALL);
          if (dead || late) __hip_atomic_store(tmo, call, __ATOMIC_RELAXED, __HIP_MEMORY_SCOPE_AGENT);
          WAIT_VM(0);
          if (draw32(wsl + OFF_RC + jsrc * 128u) + 1u == call * (SLICE * 8u) && !timed_out(tmo, call) && lane < 7u && ((rank + 1u + lane) & 7u) != jsrc)
            __hip_atomic_store((GAS u32*)((char*)prm.peer[lane] + OFF_RF + rank * 128u + jsrc * 4u), call, __ATOMIC_RELAXED, __HIP_MEMORY_SCOPE_SYSTEM);
        }
        DBG(td[kind == 2 ? 2 : 0] += now() - td0;)
        if (kind == 2) {
          u32 rb, cb;
          tile_inv(id >> 3, rb, cb);
          const u32 w = id & 7u;
          const u32 voff = (lane & 15u) * (NN * 2u) + (lane >> 4) * 16u;
          const u32 soff = (rb * 256u + (w & 3u) * 64u) * (NN * 2u) + (jsrc * NSH + cb * 128u + (w >> 2) * 64u) * 2u;
#pragma unroll
          for (int v = 0; v < 8; ++v) {
            const u32x4 q = {P[4 * v], P[4 * v + 1], P[4 * v + 2], P[4 * v + 3]};
            st16<OAUX>(q, ro, voff + (v & 1) * 64u, soff + (v >> 1) * 16u * (NN * 2u));
          }
        } else {
          u32 o;
          if (kind == 0) o = OFF_BT + (id >> 4) * (1u << 20) + (src * 16u + (id & 15u)) * 8192u;
          else o = OFF_AT + (id >> 5) * (2u << 20) + (src * 8u + ((id >> 2) & 7u)) * 32768u + (id & 3u) * 4096u;
          const u32 hstride = kind == 0 ? 4096u : 16384u;
#pragma unroll
          for (int v = 0; v < 8; ++v) {
            const u32x4 q = {P[4 * v], P[4 * v + 1], P[4 * v + 2], P[4 * v + 3]};
            st16<16>(q, ws, lane * 16u + (v & 3) * 1024u, o + (v >> 2) * hstride);
          }
          WAIT_VM(0);
          if (lane == 0)
            __hip_atomic_fetch_add((GAS u32*)(wsl + OFF_CNT + (32u + (kind == 0 ? 16u + (id >> 4) : (id >> 5))) * 128u), 1u, __ATOMIC_RELAXED, __HIP_MEMORY_SCOPE_SYSTEM);
        }
        DBG(td[kind == 2 ? 3 : 1] += now() - td0; if (n == wid - 2u) STAMP(wsl, 2);)
      }
      STAMP(wsl, 3);
      DBG(for (int i = 0; i < 4; ++i) STAMPV(wsl, 10 + i, td[i]);)
    }
  }
}

extern "C" void kernel_launch(int rank, int nranks, ncclComm_t comm, hipStream_t stream, void* const* d_in, const int* in_sizes,
                              int n_in, void* d_out, int out_size, const int* in_dtypes, int out_dtype, void* const* d_out_all,
                              void* const* d_ws_all, size_t ws_bytes) {
  (void)comm; (void)in_sizes; (void)n_in; (void)out_size; (void)in_dtypes; (void)out_dtype; (void)d_out_all;
  if (nranks != NR || ws_bytes < WS_NEED || (unsigned)rank >= (unsigned)NR) {
    fprintf(stderr, "unsupported configuration\n");
    return;
  }
  Params p;
  p.x = d_in[0];
  p.w = d_in[1];
  p.out = d_out;
  p.ws = d_ws_all[rank];
  for (int i = 0; i < NR; ++i) p.peer[i] = d_ws_all[(rank + 1 + i) % NR];
  p.rank = rank;
  p.pad = 0;
  gemm_ar_kernel<<<NWG, NTHR, 0, stream>>>(p);
}
```
